```python
import numpy as np

kernel_name = "gpu_gemm_rs_m4096_k4096_n16384_e4m3e4m3_dqrelu"


ABSMAX_RTOL = 0.03

COMPUTE_DTYPE = "float32"
EPILOGUE = "dequant_relu"
IN_QUANT = None
OUT_QUANT = None
IN_FP8 = "e4m3e4m3"
IN_DTYPES = {"x": "e4m3", "w_mat": "e4m3",
             "scale_x": "float32", "scale_w": "float32"}
WORLD_SIZE = 8

def _snap_fp8(x, mant_bits, max_abs, min_normal):
    a = np.abs(x.astype(np.float64))
    _m, _e = np.frexp(a)
    q = np.where(a >= min_normal, np.ldexp(1.0, _e - 1 - mant_bits),
                 min_normal * 2.0 ** -mant_bits)
    snapped = np.minimum(np.round(a / q) * q, max_abs)
    return (np.sign(x) * snapped).astype(np.float32)

def setup_inputs():

    x = _snap_fp8(np.random.randn(4096, 4096).astype(np.float32), 3, 448.0, 2.0 ** -6)
    w_mat = _snap_fp8(np.random.randn(4096, 16384).astype(np.float32), 3, 448.0, 2.0 ** -6)
    scale_x = (0.02 * (1.0 + np.random.rand(1))).astype(np.float32)
    scale_w = (0.02 * (1.0 + np.random.rand(1))).astype(np.float32)
    return {"x": x, "w_mat": w_mat, "scale_x": scale_x, "scale_w": scale_w}

def reference(x, w_mat, scale_x, scale_w):
    acc = (x @ w_mat).astype(np.float32)
    y = (acc * (scale_x * scale_w)).astype(np.float32)
    return np.maximum(y, 0.0).astype(np.float32)
```

```cpp
#include <hip/hip_runtime.h>
#include <rccl/rccl.h>
#include <cstdio>
#include <cstdint>

#define HIP_CHECK(expr) do { hipError_t _e = (expr); if (_e != hipSuccess) { \
    fprintf(stderr, "HIP error %s at %s:%d\n", hipGetErrorString(_e), __FILE__, __LINE__); \
    return; } } while (0)

namespace {

typedef int v8i __attribute__((ext_vector_type(8)));
typedef int v4i __attribute__((ext_vector_type(4)));
typedef unsigned v4u __attribute__((ext_vector_type(4)));
typedef float v16f __attribute__((ext_vector_type(16)));
#define LAS __attribute__((address_space(3)))
#define GAS __attribute__((address_space(1)))
typedef GAS unsigned gu32;

constexpr int NR = 8;
constexpr int GM = 4096, GK = 4096, GN = 16384;
constexpr int KS = GK / NR;
constexpr int MS = GM / NR;
constexpr int NS = GN / NR;

constexpr unsigned KiB = 1024u, MiB = 1024u * 1024u;
constexpr unsigned OFF_CNT   = 0;
constexpr unsigned OFF_TMO   = 64 * KiB;
constexpr unsigned OFF_SFLAG = 68 * KiB;
constexpr unsigned OFF_OFLAG = 80 * KiB;
constexpr unsigned OFF_PROG  = 96 * KiB;
constexpr unsigned OFF_DBG   = 128 * KiB;
constexpr unsigned OFF_XG    = 1 * MiB;
constexpr unsigned OFF_WG    = OFF_XG + 16 * MiB;
constexpr unsigned OFF_WS    = OFF_WG + 8 * MiB;
constexpr unsigned OFF_OS    = OFF_WS + 8 * MiB;
constexpr unsigned WS_USED   = OFF_OS + 32 * MiB;
constexpr unsigned OUT_BYTES = MS * GN * 4;

constexpr unsigned long long DEADLINE = 50000000ull;
constexpr int AUX_SYS = 17;
constexpr int AUX_DEV = 16;

constexpr int STAGE_A = 32 * 1024, STAGE_B = 16 * 1024, STAGE_BYTES = STAGE_A + STAGE_B;
constexpr int LDS_BYTES = 3 * STAGE_BYTES;

struct Params {
  const unsigned char* x;
  const unsigned char* w;
  const float* sx;
  const float* sw;
  float* out;
  unsigned char* ws[NR];
  int rank;
};

#define WAIT_VM(n) asm volatile("s_waitcnt vmcnt(" #n ")" ::: "memory")
#define DMA4(ldsp, soff) do { \
    __builtin_amdgcn_raw_ptr_buffer_load_lds(rsrc_me, (LAS void*)(ldsp), 16, lane * 16, (soff), 0, AUX_DEV); \
    __builtin_amdgcn_raw_ptr_buffer_load_lds(rsrc_me, (LAS void*)(ldsp), 16, lane * 16, (soff), 1024, AUX_DEV); \
    __builtin_amdgcn_raw_ptr_buffer_load_lds(rsrc_me, (LAS void*)(ldsp), 16, lane * 16, (soff), 2048, AUX_DEV); \
    __builtin_amdgcn_raw_ptr_buffer_load_lds(rsrc_me, (LAS void*)(ldsp), 16, lane * 16, (soff), 3072, AUX_DEV); } while (0)
#define WAIT_LGKM0() asm volatile("s_waitcnt lgkmcnt(0)" ::: "memory")

__device__ __forceinline__ __amdgpu_buffer_rsrc_t mk_rsrc(const void* p, unsigned bytes) {
  return __builtin_amdgcn_make_buffer_rsrc((void*)p, 0, bytes, 0x00020000);
}

#define STAMP(slot) do {} while (0)

__device__ __forceinline__ unsigned long long wait_words(const gu32* addr, unsigned thr, unsigned long long need,
                                                         gu32* tmo, unsigned call, bool& dead, unsigned& val) {
  unsigned long long t0 = __builtin_amdgcn_s_memrealtime();
  unsigned long long ok;
  for (;;) {
    val = __hip_atomic_load(addr, __ATOMIC_RELAXED, __HIP_MEMORY_SCOPE_SYSTEM);
    ok = __ballot(val >= thr);
    if ((ok & need) == need) break;
    unsigned t = __hip_atomic_load(tmo, __ATOMIC_RELAXED, __HIP_MEMORY_SCOPE_AGENT);
    if (__builtin_amdgcn_readfirstlane(t) == call || __builtin_amdgcn_s_memrealtime() - t0 > DEADLINE) {
      __hip_atomic_store(tmo, call, __ATOMIC_RELAXED, __HIP_MEMORY_SCOPE_AGENT);
      dead = true;
      ok = ~0ull;
      val = 0xffffffffu;
      break;
    }
    __builtin_amdgcn_s_sleep(4);
  }
  __builtin_amdgcn_fence(__ATOMIC_ACQUIRE, "wavefront");
  return ok;
}

__global__ void __launch_bounds__(256, 1) fused_kernel(Params P) {
  __shared__ __attribute__((aligned(1024))) unsigned char lds[LDS_BYTES];
  const int lane = threadIdx.x & 63;
  const int wave = __builtin_amdgcn_readfirstlane(threadIdx.x >> 6);
  const int rank = P.rank & 7;
  unsigned char* const me = P.ws[rank];
  const int xcd = blockIdx.x & 7, slot = blockIdx.x >> 3;
  gu32* const tmo = (gu32*)(me + OFF_TMO);
  STAMP(0);

  unsigned call = 0;
  if (lane == 0)
    call = __hip_atomic_fetch_add((gu32*)(me + OFF_CNT + (blockIdx.x * 4 + wave) * 64), 1u, __ATOMIC_RELAXED, __HIP_MEMORY_SCOPE_AGENT) + 1u;
  bool dead = false;
  const int h = lane >> 5, r = lane & 31;

  if (slot < 4) {
    const int li = (xcd < 4) ? slot : 3 + slot;
    if (xcd < 4 && slot == 3) return;
    call = __builtin_amdgcn_readfirstlane(call);
    const int peer = (rank + 1 + li) & 7;
    const int w = (xcd & 3) * 4 + wave;
    const auto rsrc_peer = mk_rsrc(P.ws[peer], WS_USED);
    const auto rsrc_me = mk_rsrc(me, WS_USED);
    const auto rsrc_out = mk_rsrc(P.out, OUT_BYTES);
    const gu32* sfl = (const gu32*)(me + OFF_SFLAG) + (peer * 16 + w) * 16 + (lane & 15);
    const gu32* ofl = (const gu32*)(me + OFF_OFLAG) + (peer * 16 + w) * 16 + (lane & 15);
    gu32* prog = (gu32*)(me + OFF_PROG) + peer * 32 + w;
    const unsigned vlin = lane * 16;
    const unsigned vout = h * 4 * (GN * 4) + r * 16;
    unsigned long long have = 0;
    unsigned dummy;

    auto src_off = [&](int i) -> unsigned {
      if (i < 48) {
        int t = i >> 2; unsigned sub = (i & 3) * 4096u;
        if (t < 8 && !(t & 1)) return OFF_WS + rank * MiB + (t >> 1) * (256 * KiB) + w * (16 * KiB) + sub;
        unsigned cs = (t < 8) ? (t >> 1) : (t - 4);
        return OFF_XG + peer * (2 * MiB) + cs * (256 * KiB) + w * (16 * KiB) + sub;
      }
      int o = i - 48; int c = o >> 5;
      return OFF_OS + rank * (4 * MiB) + c * (2 * MiB) + w * (128 * KiB) + (o & 31) * 4096u;
    };
    auto ready = [&](int i) {
      int bit = (i < 48) ? (i >> 2) : 16 + ((i - 48) >> 3);
      if (!((have >> bit) & 1)) {
        if (i < 48) have |= wait_words(sfl, call, 1ull << bit, tmo, call, dead, dummy) & 0xffffull;
        else have |= (wait_words(ofl, call, 1ull << (bit - 16), tmo, call, dead, dummy) & 0xffffull) << 16;
      }
    };
    auto load = [&](int i, v4u (&v)[4]) {
      ready(i);
      unsigned so = src_off(i);
#pragma unroll
      for (int u = 0; u < 4; ++u) v[u] = __builtin_amdgcn_raw_buffer_load_b128(rsrc_peer, vlin + u * 1024, so, AUX_SYS);
    };
    auto store = [&](int i, v4u (&v)[4]) {
      if (i < 48) {
        int t = i >> 2;
        unsigned so = src_off(i);
        if (t < 8 && !(t & 1)) so = so - (OFF_WS + rank * MiB) + (OFF_WG + peer * MiB);
#pragma unroll
        for (int u = 0; u < 4; ++u) __builtin_amdgcn_raw_buffer_store_b128(v[u], rsrc_me, vlin + u * 1024, so, AUX_DEV);
      } else {
        int o = i - 48;
        unsigned so = o * 8u * (GN * 4) + (peer * NS + w * 128) * 4u;
        if (dead) { v4u nanv = {0x7fc00000u, 0x7fc00000u, 0x7fc00000u, 0x7fc00000u}; for (int u = 0; u < 4; ++u) v[u] = nanv; }
#pragma unroll
        for (int u = 0; u < 4; ++u) __builtin_amdgcn_raw_buffer_store_b128(v[u], rsrc_out, vout, so + u * (GN * 4), AUX_DEV);
      }
    };
    auto publish = [&](int i) {
      if (i < 0 || i >= 48) return;
      unsigned cnt = 0;
      if (i < 32) { if ((i & 7) == 7) cnt = (i >> 3) + 1; }
      else if ((i & 3) == 3) cnt = 5 + ((i - 32) >> 2);
      if (cnt && lane == 0) __hip_atomic_store(prog, call * 16u + cnt, __ATOMIC_RELAXED, __HIP_MEMORY_SCOPE_AGENT);
    };
    v4u va[4], vb[4];
    load(0, va);
    STAMP(1);
    for (int i = 0; i < 112; i += 2) {
      load(i + 1, vb);
      WAIT_VM(8);
      publish(i - 2);
      store(i, va);
      if (i + 2 < 112) load(i + 2, va);
      WAIT_VM(8);
      publish(i - 1);
      store(i + 1, vb);
    }
    WAIT_VM(0);
    STAMP(5);
    return;
  }

  const int g = slot - 4;
  const int wgi = g * 8 + xcd;
  {
    const int jb = wave * 224 + wgi;
    if (jb < 640) {
      const auto rsrc_me = mk_rsrc(me, WS_USED);
      const int s = jb / 160, t = jb % 160;
      if (t < 128) {
        const int q = t >> 4, j = t & 15;
        const auto rw = mk_rsrc(P.w, KS * GN);
        const unsigned voff = h * 16 * GN + r * 4;
        const unsigned so = s * 128 * GN + q * NS + j * 128;
        unsigned d[64];
#pragma unroll
        for (int e = 0; e < 64; ++e)
          d[e] = __builtin_amdgcn_raw_buffer_load_b32(rw, voff, so + ((e >> 4) * 32 + (e & 15)) * GN, 0);
        const unsigned dst = ((q == rank) ? OFF_WG + rank * MiB : OFF_WS + q * MiB) + s * (256 * KiB) + j * (16 * KiB);
#pragma unroll
        for (int ki = 0; ki < 4; ++ki) {
          v4u o[4];
#pragma unroll
          for (int gq = 0; gq < 4; ++gq) {
            unsigned a = d[ki * 16 + gq * 4], b = d[ki * 16 + gq * 4 + 1], c = d[ki * 16 + gq * 4 + 2], e = d[ki * 16 + gq * 4 + 3];
            unsigned t0 = __builtin_amdgcn_perm(b, a, 0x05010400u), t1 = __builtin_amdgcn_perm(b, a, 0x07030602u);
            unsigned t2 = __builtin_amdgcn_perm(e, c, 0x05010400u), t3 = __builtin_amdgcn_perm(e, c, 0x07030602u);
            o[0][gq] = __builtin_amdgcn_perm(t2, t0, 0x05040100u);
            o[1][gq] = __builtin_amdgcn_perm(t2, t0, 0x07060302u);
            o[2][gq] = __builtin_amdgcn_perm(t3, t1, 0x05040100u);
            o[3][gq] = __builtin_amdgcn_perm(t3, t1, 0x07060302u);
          }
#pragma unroll
          for (int f = 0; f < 4; ++f)
            __builtin_amdgcn_raw_buffer_store_b128(o[f], rsrc_me, lane * 16, dst + f * 4096 + ki * 1024, AUX_SYS);
        }
        WAIT_VM(0);
        call = __builtin_amdgcn_readfirstlane(call);
        if (lane == 0)
          __hip_atomic_store((gu32*)(P.ws[q] + OFF_SFLAG) + (rank * 16 + j) * 16 + 2 * s, call, __ATOMIC_RELAXED, __HIP_MEMORY_SCOPE_SYSTEM);
      } else {
        const int c = (t - 128) >> 4, u = (t - 128) & 15;
        const auto rx = mk_rsrc(P.x, GM * KS);
        const unsigned voff = r * KS + h * 16;
        const unsigned so = ((u >> 1) * MS + c * 256 + (u & 1) * 128) * KS + s * 128;
        v4u d[16];
#pragma unroll
        for (int e = 0; e < 16; ++e)
          d[e] = __builtin_amdgcn_raw_buffer_load_b128(rx, voff + (e & 3) * 32, so + (e >> 2) * 32 * KS, 0);
        const unsigned dst = OFF_XG + rank * (2 * MiB) + (c * 4 + s) * (256 * KiB) + u * (16 * KiB);
#pragma unroll
        for (int e = 0; e < 16; ++e)
          __builtin_amdgcn_raw_buffer_store_b128(d[e], rsrc_me, lane * 16, dst + e * 1024, AUX_SYS);
        WAIT_VM(0);
        call = __builtin_amdgcn_readfirstlane(call);
        if (lane < NR)
          __hip_atomic_store((gu32*)(P.ws[lane] + OFF_SFLAG) + (rank * 16 + u) * 16 + (c ? 8 + s : 2 * s + 1), call, __ATOMIC_RELAXED, __HIP_MEMORY_SCOPE_SYSTEM);
      }
    }
  }
  call = __builtin_amdgcn_readfirstlane(call);
  STAMP(1);

  const float scale = P.sx[0] * P.sw[0];
  const auto rsrc_me = mk_rsrc(me, WS_USED);
  const int n0 = (xcd < 4) ? 16 : 12;
  const int ntiles = (g < 4) ? 2 : 1;
  for (int it = 0; it < ntiles; ++it) {
    int c, own, j;
    if (it == 0) {
      c = (g < n0) ? 0 : 1;
      const int t = (g < n0) ? g : g - n0;
      const int oi = ((xcd < 4) == (c == 0)) ? (t >> 2) : 4 + (t >> 2);
      own = (rank + 1 + oi) & 7;
      j = (xcd & 3) * 4 + (t & 3);
    } else {
      c = xcd >> 2; own = rank; j = (xcd & 3) * 4 + g;
    }
    const int wl = (lane < 21) ? lane : 0;
    const int wli = wl / 3, wh = wl - wli * 3;
    const gu32* paddr = (const gu32*)(me + OFF_PROG) + ((rank + 1 + wli) & 7) * 32 + (wh == 0 ? j : 2 * own + wh - 1);
    const unsigned pthr = call * 16u + ((wh != 0 && c) ? 5u : 1u);
    unsigned pval = 0;

    auto issue = [&](int k) {
      int p, s;
      if (k < 4) { p = rank; s = k; } else { int kk = k - 4; s = kk / 7; p = (rank + 1 + (kk - s * 7)) & 7; }
      const unsigned sa = OFF_XG + p * (2 * MiB) + (c * 4 + s) * (256 * KiB) + own * (32 * KiB) + wave * 8192;
      const unsigned sb = OFF_WG + p * MiB + s * (256 * KiB) + j * (16 * KiB) + wave * 4096;
      unsigned char* lb = lds + (k % 3) * STAGE_BYTES;
      DMA4(lb + wave * 8192, sa);
      DMA4(lb + wave * 8192 + 4096, sa + 4096);
      DMA4(lb + STAGE_A + wave * 4096, sb);
    };
    auto frags = [&](int k, int kb, v8i (&A)[2], v8i (&B)[4]) {
      const unsigned char* lb = lds + (k % 3) * STAGE_BYTES + lane * 16 + kb * 2048;
#pragma unroll
      for (int mi = 0; mi < 2; ++mi) {
        v4i lo = *(const v4i*)(lb + wave * 8192 + mi * 4096), hi = *(const v4i*)(lb + wave * 8192 + mi * 4096 + 1024);
        A[mi] = __builtin_shufflevector(lo, hi, 0, 1, 2, 3, 4, 5, 6, 7);
      }
#pragma unroll
      for (int f = 0; f < 4; ++f) {
        v4i lo = *(const v4i*)(lb + STAGE_A + f * 4096), hi = *(const v4i*)(lb + STAGE_A + f * 4096 + 1024);
        B[f] = __builtin_shufflevector(lo, hi, 0, 1, 2, 3, 4, 5, 6, 7);
      }
    };
    v16f acc[2][4];
#pragma unroll
    for (int mi = 0; mi < 2; ++mi)
#pragma unroll
      for (int f = 0; f < 4; ++f)
#pragma unroll
        for (int e = 0; e < 16; ++e) acc[mi][f][e] = 0.f;
    auto mma = [&](v8i (&A)[2], v8i (&B)[4]) {
#pragma unroll
      for (int mi = 0; mi < 2; ++mi)
#pragma unroll
        for (int f = 0; f < 4; ++f)
          acc[mi][f] = __builtin_amdgcn_mfma_scale_f32_32x32x64_f8f6f4(A[mi], B[f], acc[mi][f], 0, 0, 0, 0, 0, 0);
    };

    if (wave == 0) {
      const int sl = (lane < 12) ? lane : 0;
      const int ss = sl / 3, sw_ = sl - ss * 3;
      const gu32* a = (const gu32*)(me + OFF_SFLAG) + (rank * 16 + (sw_ == 0 ? j : 2 * own + sw_ - 1)) * 16 + (sw_ == 0 ? 2 * ss : (c ? 8 + ss : 2 * ss + 1));
      unsigned dummy;
      wait_words(a, call, 0xfffull, tmo, call, dead, dummy);
    }
    __builtin_amdgcn_s_barrier();
    issue(0); issue(1); issue(2);
    WAIT_VM(24);
    __builtin_amdgcn_s_barrier();
    v8i A0[2], B0[4], A1[2], B1[4];
    frags(0, 0, A0, B0);
    for (int k = 0; k < 32; ++k) {
      frags(k, 1, A1, B1);
      mma(A0, B0);
      if (wave == 0 && k + 3 >= 4 && k + 3 < 32) {
        const int kk = k - 1, s = kk / 7, li = kk - s * 7;
        const unsigned long long need = 7ull << (3 * li);
        if ((__ballot(pval >= pthr + s) & need) != need) wait_words(paddr, pthr + s, need, tmo, call, dead, pval);
      }
      WAIT_LGKM0();
      if (k < 30) WAIT_VM(12); else WAIT_VM(0);
      __builtin_amdgcn_s_barrier();
      if (k + 3 < 32) issue(k + 3);
      if (k + 1 < 32) frags(k + 1, 0, A0, B0);
      mma(A1, B1);
    }
    STAMP(2 + 2 * it);
    const bool poison = __builtin_amdgcn_readfirstlane(__hip_atomic_load(tmo, __ATOMIC_RELAXED, __HIP_MEMORY_SCOPE_AGENT)) == call;
    const bool local = (own == rank);
    const auto rdst = local ? mk_rsrc(P.out, OUT_BYTES) : rsrc_me;
    const unsigned vo = local ? (unsigned)(h * 4 * (GN * 4) + r * 16) : (unsigned)(lane * 16);
    const unsigned sbase = local ? (unsigned)((c * 256 + wave * 64) * (GN * 4) + (rank * NS + j * 128) * 4)
                                 : (unsigned)(OFF_OS + own * (4 * MiB) + c * (2 * MiB) + j * (128 * KiB) + wave * (32 * KiB));
#pragma unroll
    for (int mi = 0; mi < 2; ++mi)
#pragma unroll
      for (int e = 0; e < 16; ++e) {
        v4u o;
#pragma unroll
        for (int f = 0; f < 4; ++f) o[f] = poison ? 0x7fc00000u : __float_as_uint(fmaxf(acc[mi][f][e] * scale, 0.f));
        const unsigned so = local ? (unsigned)((mi * 32 + (e >> 2) * 8 + (e & 3)) * (GN * 4)) : (unsigned)((mi * 16 + e) * 1024);
        __builtin_amdgcn_raw_buffer_store_b128(o, rdst, vo, sbase + so, AUX_SYS);
      }
    WAIT_VM(0);
    if (!local && lane == 0)
      __hip_atomic_store((gu32*)(P.ws[own] + OFF_OFLAG) + (rank * 16 + j) * 16 + c * 4 + wave, call, __ATOMIC_RELAXED, __HIP_MEMORY_SCOPE_SYSTEM);
    STAMP(3 + 2 * it);
  }
}

}

extern "C" void kernel_launch(int rank, int nranks, ncclComm_t comm,
                              hipStream_t stream,
                              void* const* d_in, const int* in_sizes, int n_in,
                              void* d_out, int out_size,
                              const int* in_dtypes, int out_dtype,
                              void* const* d_out_all,
                              void* const* d_ws_all, size_t ws_bytes) {
  if (nranks != NR || n_in < 4 || ws_bytes < WS_USED || out_size != MS * GN ||
      in_sizes[0] != GM * KS || in_sizes[1] != KS * GN) {
    fprintf(stderr, "unexpected problem shape\n");
    return;
  }
  Params P;
  P.x = (const unsigned char*)d_in[0];
  P.w = (const unsigned char*)d_in[1];
  P.sx = (const float*)d_in[2];
  P.sw = (const float*)d_in[3];
  P.out = (float*)d_out;
  for (int p = 0; p < NR; ++p) P.ws[p] = (unsigned char*)d_ws_all[p];
  P.rank = rank;
  hipLaunchKernelGGL(fused_kernel, dim3(256), dim3(256), 0, stream, P);
  HIP_CHECK(hipGetLastError());
}
```

```cpp
#include <hip/hip_runtime.h>
#include <rccl/rccl.h>
#include <cstdio>
#include <cstdint>

#define HIP_CHECK(expr) do { hipError_t _e = (expr); if (_e != hipSuccess) { \
    fprintf(stderr, "HIP error %s at %s:%d\n", hipGetErrorString(_e), __FILE__, __LINE__); \
    return; } } while (0)

namespace {

typedef int v8i __attribute__((ext_vector_type(8)));
typedef int v4i __attribute__((ext_vector_type(4)));
typedef unsigned v4u __attribute__((ext_vector_type(4)));
typedef unsigned v2u __attribute__((ext_vector_type(2)));
typedef float v16f __attribute__((ext_vector_type(16)));
typedef unsigned long long u64;
#define LAS __attribute__((address_space(3)))
#define GAS __attribute__((address_space(1)))
typedef GAS unsigned gu32;
typedef GAS u64 gu64;

constexpr int NR = 8;
constexpr int GM = 4096, GK = 4096, GN = 16384;
constexpr int KS = GK / NR;
constexpr int MS = GM / NR;
constexpr int NS = GN / NR;

constexpr unsigned KiB = 1024u, MiB = 1024u * 1024u;
constexpr unsigned OFF_CNT   = 0;
constexpr unsigned OFF_TMO   = 64 * KiB;
constexpr unsigned OFF_PROG  = 68 * KiB;
constexpr unsigned OFF_FLAG  = 80 * KiB;
constexpr unsigned OFF_DBG   = 128 * KiB;
constexpr unsigned OFF_XG    = 1 * MiB;
constexpr unsigned OFF_WG    = OFF_XG + 16 * MiB;
constexpr unsigned OFF_WS    = OFF_WG + 8 * MiB;
constexpr unsigned OFF_OS    = OFF_WS + 8 * MiB;
constexpr unsigned OSLOT     = 17 * KiB;
constexpr unsigned OS_OWNER  = 256 * OSLOT;
constexpr unsigned WS_USED   = OFF_OS + 8 * OS_OWNER;
constexpr unsigned OUT_BYTES = MS * GN * 4;

constexpr unsigned long long DEADLINE = 50000000ull;
#define TOPO_MASK 0x4Du
constexpr int AUX_SYS = 17;
constexpr int AUX_DEV = 16;
#define GEMM_AUX 16

constexpr int STAGE_A = 16 * 1024, STAGE_B = 32 * 1024, STAGE_BYTES = STAGE_A + STAGE_B;
constexpr int LDS_ORDER = 3 * STAGE_BYTES;
constexpr int LDS_BYTES = LDS_ORDER + 512;
constexpr int LDS_FLAGS = 8 * OSLOT;
static_assert(LDS_FLAGS + 2048 <= LDS_BYTES, "lds");

struct Params {
  const unsigned char* x;
  const unsigned char* w;
  const float* sx;
  const float* sw;
  float* out;
  unsigned char* ws[NR];
  int rank;
};

#define WAIT_VM(n) asm volatile("s_waitcnt vmcnt(" #n ")" ::: "memory")
#define STORE16(val, rs, voff, soff, aux) do { const v4u sv_ = (val); \
    __builtin_amdgcn_raw_buffer_store_b128(sv_, rs, voff, soff, aux); \
    asm volatile("s_nop 1" :: "v"(sv_) : "memory"); } while (0)
#define DELAY1 24
#define DELAY2 72
#define STAGE_DELAY(n) do { if ((n) == DELAY1) __builtin_amdgcn_s_sleep(DELAY1); else __builtin_amdgcn_s_sleep(DELAY2); } while (0)
template <int N> __device__ __forceinline__ void wait_vm() { asm volatile("s_waitcnt vmcnt(%0)" :: "i"(N) : "memory"); }
#define OUT_AUX 16
#define IN_WIN 4
#define OUT_WIN 4
#define TAIL_WIN 4
#define TAIL_FROM 14
__device__ __forceinline__ void wait_vm_dyn(unsigned n) {
  n = n > 31u ? 31u : n;
#define WV1(k) "s_waitcnt vmcnt(" #k ")\n\ts_branch 1f\n\t"
  asm volatile("s_getpc_b64 vcc\n\t"
               "s_lshl3_add_u32 %0, %0, 16\n\t"
               "s_add_u32 vcc_lo, vcc_lo, %0\n\t"
               "s_addc_u32 vcc_hi, vcc_hi, 0\n\t"
               "s_setpc_b64 vcc\n\t"
               WV1(0) WV1(1) WV1(2) WV1(3) WV1(4) WV1(5) WV1(6) WV1(7) WV1(8) WV1(9) WV1(10) WV1(11) WV1(12) WV1(13) WV1(14) WV1(15)
               WV1(16) WV1(17) WV1(18) WV1(19) WV1(20) WV1(21) WV1(22) WV1(23) WV1(24) WV1(25) WV1(26) WV1(27) WV1(28) WV1(29) WV1(30) WV1(31)
               "1:"
               : "+s"(n) :: "memory", "scc", "vcc");
#undef WV1
}
#define WAIT_LGKM0() asm volatile("s_waitcnt lgkmcnt(0)" ::: "memory")

__device__ __forceinline__ __amdgpu_buffer_rsrc_t mk_rsrc(const void* p, unsigned bytes) {
  return __builtin_amdgcn_make_buffer_rsrc((void*)p, 0, bytes, 0x00020000);
}
__device__ __forceinline__ unsigned wave_scan(unsigned x) {
  x += __builtin_amdgcn_update_dpp(0u, x, 0x111, 0xf, 0xf, false);
  x += __builtin_amdgcn_update_dpp(0u, x, 0x112, 0xf, 0xf, false);
  x += __builtin_amdgcn_update_dpp(0u, x, 0x114, 0xf, 0xf, false);
  x += __builtin_amdgcn_update_dpp(0u, x, 0x118, 0xf, 0xf, false);
  x += __builtin_amdgcn_update_dpp(0u, x, 0x142, 0xa, 0xf, false);
  x += __builtin_amdgcn_update_dpp(0u, x, 0x143, 0xc, 0xf, false);
  return x;
}
template <int P>
__device__ __forceinline__ void run_addr4(unsigned m, unsigned other, unsigned& q, unsigned (&a)[4]) {
  unsigned t;
  asm("v_bfe_i32 %5, %6, %8, 1\n\tv_bfi_b32 %0, %5, %7, %4\n\tv_lshl_add_u32 %4, %5, 2, %4\n\t"
      "v_bfe_i32 %5, %6, %9, 1\n\tv_bfi_b32 %1, %5, %7, %4\n\tv_lshl_add_u32 %4, %5, 2, %4\n\t"
      "v_bfe_i32 %5, %6, %10, 1\n\tv_bfi_b32 %2, %5, %7, %4\n\tv_lshl_add_u32 %4, %5, 2, %4\n\t"
      "v_bfe_i32 %5, %6, %11, 1\n\tv_bfi_b32 %3, %5, %7, %4\n\tv_lshl_add_u32 %4, %5, 2, %4"
      : "=&v"(a[0]), "=&v"(a[1]), "=&v"(a[2]), "=&v"(a[3]), "+v"(q), "=&v"(t)
      : "v"(m), "v"(other), "i"(P), "i"(P - 1), "i"(P - 2), "i"(P - 3));
}
template <int E> struct IC { static constexpr int v = E; };
template <int N, int I = 0, class F>
__device__ __forceinline__ void static_for(F&& f) { if constexpr (I < N) { f(IC<I>{}); static_for<N, I + 1>(f); } }
__device__ __forceinline__ unsigned lds_addr(const void* p) { return (unsigned)(size_t)(const LAS unsigned char*)p; }

#define STAMP(slot) do {} while (0)

__device__ __forceinline__ bool give_up(u64 t0, gu32* tmo, unsigned call) {
  unsigned t = __hip_atomic_load(tmo, __ATOMIC_RELAXED, __HIP_MEMORY_SCOPE_AGENT);
  if (__builtin_amdgcn_readfirstlane(t) == call || __builtin_amdgcn_s_memrealtime() - t0 > DEADLINE) {
    __hip_atomic_store(tmo, call, __ATOMIC_RELAXED, __HIP_MEMORY_SCOPE_AGENT);
    return true;
  }
  __builtin_amdgcn_s_sleep(4);
  return false;
}

__device__ __forceinline__ u64 wait_words(const gu32* addr, unsigned thr, u64 need,
                                          gu32* tmo, unsigned call, bool& dead, unsigned& val) {
  u64 t0 = __builtin_amdgcn_s_memrealtime();
  u64 ok;
  for (;;) {
    val = __hip_atomic_load(addr, __ATOMIC_RELAXED, __HIP_MEMORY_SCOPE_SYSTEM);
    ok = __ballot(val >= thr);
    if ((ok & need) == need) break;
    if (give_up(t0, tmo, call)) { dead = true; ok = ~0ull; val = 0xffffffffu; break; }
  }
  __builtin_amdgcn_fence(__ATOMIC_ACQUIRE, "wavefront");
  return ok;
}

#define ROUND_ITEMS 0x75634ba12980ull
#define ITEM_ROUNDS 0x6521b9a78340ull
__device__ __forceinline__ int item_of_round(int t) { return (int)((ROUND_ITEMS >> (4 * t)) & 15u); }
__device__ __forceinline__ int round_of_item(int it) { return (int)((ITEM_ROUNDS >> (4 * it)) & 15u); }
__device__ __forceinline__ int round_of_w(int s) { return round_of_item(8 + s); }
__device__ __forceinline__ int round_of_x(int xr) { return round_of_item(xr); }

__global__ void __launch_bounds__(256, 1) fused_kernel(Params P) {
  __shared__ __attribute__((aligned(1024))) unsigned char lds[LDS_BYTES];
  const int lane = threadIdx.x & 63;
  const int wave = __builtin_amdgcn_readfirstlane(threadIdx.x >> 6);
  const int rank = P.rank & 7;
  unsigned char* const me = P.ws[rank];
  const int xcd = blockIdx.x & 7, slot = blockIdx.x >> 3;
  gu32* const tmo = (gu32*)(me + OFF_TMO);
  STAMP(0);

  gu32* const cnt = (gu32*)(me + OFF_CNT + (blockIdx.x * 4 + wave) * 64);
  const unsigned call_v = __hip_atomic_load(cnt, __ATOMIC_RELAXED, __HIP_MEMORY_SCOPE_AGENT) + 1u;
  bool dead = false;
  const int h = lane >> 5, r = lane & 31;

  if (slot < 4) {
    const unsigned call = __builtin_amdgcn_readfirstlane(call_v);
    if (lane == 0) __hip_atomic_store(cnt, call, __ATOMIC_RELAXED, __HIP_MEMORY_SCOPE_AGENT);
    const int li = (xcd < 4) ? slot : 3 + slot;
    if (xcd < 4 && slot == 3) return;
    const unsigned quad = ((TOPO_MASK >> rank) & 1) ? TOPO_MASK : (~TOPO_MASK & 0xffu);
    unsigned cand = (xcd < 4) ? (quad & ~(1u << rank)) : (~quad & 0xffu);
    for (int i = 0; i < slot; ++i) cand &= cand - 1;
    const int peer = __builtin_ctz(cand) & 7;
    const int w = (xcd & 3) * 4 + wave;
    unsigned char* const pw = P.ws[peer];
    const auto rsrc_peer = mk_rsrc(pw, WS_USED);
    const auto rsrc_me = mk_rsrc(me, WS_USED);
    const auto rsrc_out = mk_rsrc(P.out, OUT_BYTES);
    const gu32* const frec = (const gu32*)(me + OFF_FLAG) + (peer * 16 + w) * 64;
    const auto rsrc_flag = mk_rsrc((const void*)frec, 256);
    gu32* const prog = (gu32*)(me + OFF_PROG) + peer * 32 + w;
    const unsigned vlin = lane * 16;
    unsigned char* const ubuf = lds + wave * (2 * OSLOT);
    unsigned char* const fbuf = lds + LDS_FLAGS + wave * 256;
    const unsigned vout = h * 4 * (GN * 4) + r * 16;
    const unsigned zaddr = lds_addr(lds + LDS_FLAGS + 1024 + wave * 256);
    *(LAS unsigned*)(size_t)(zaddr + lane * 4) = 0u;

    unsigned ops = 64;
    u64 hist = 0;
    unsigned pend_op = 0, pend_round = 0;
    unsigned fl = 0;
    auto settle = [&](unsigned done_below) {
      if (pend_round && pend_op < done_below) {
        if (lane == 0) __hip_atomic_store(prog, call * 16u + pend_round, __ATOMIC_RELAXED, __HIP_MEMORY_SCOPE_AGENT);
        ++ops; pend_round = 0;
      }
    };
    auto throttle = [&](int win) {
      const unsigned n = (ops - 1u - (unsigned)(hist >> (8 * win - 8))) & 255u;
      wait_vm_dyn(n);
      settle(ops - n);
    };
    auto piece = [&](const __amdgpu_buffer_rsrc_t rs, unsigned char* dst, unsigned voff, unsigned soff) {
      __builtin_amdgcn_raw_ptr_buffer_load_lds(rs, (LAS void*)dst, 16, voff, soff, 0, AUX_SYS);
      hist = (hist << 8) | (ops & 255u); ++ops;
    };
    auto refresh = [&]() { __builtin_amdgcn_raw_ptr_buffer_load_lds(rsrc_flag, (LAS void*)fbuf, 4, lane * 4, 0, 0, AUX_SYS); ++ops; };
    auto await_flag = [&](int idx) {
      const u64 t0 = __builtin_amdgcn_s_memrealtime();
      while (!dead) {
        fl = __hip_atomic_load(frec + lane, __ATOMIC_RELAXED, __HIP_MEMORY_SCOPE_SYSTEM);
        if (__builtin_amdgcn_readlane(fl, idx) == call) break;
        if (give_up(t0, tmo, call)) dead = true;
      }
      __builtin_amdgcn_s_waitcnt(0x0F70);
      __builtin_amdgcn_fence(__ATOMIC_ACQUIRE, "wavefront");
      settle(ops);
    };

    unsigned char* const ring = ubuf + OSLOT;
    auto put = [&](int c) {
      const int it = item_of_round(c >> 4);
      const unsigned so = ((it >= 8) ? OFF_WG + peer * MiB + (it - 8) * (256 * KiB) : OFF_XG + peer * (2 * MiB) + it * (256 * KiB)) + w * (16 * KiB) + (c & 15) * 1024u;
      const v4u t = *(const v4u*)(ring + (c & 15) * 1024 + vlin);
      STORE16(t, rsrc_me, vlin, so, AUX_DEV);
      if ((c & 15) == 15) { pend_op = ops; pend_round = (c >> 4) + 1; }
      ++ops;
    };
    for (int d = 0; d < 192; ++d) {
      const int t = d >> 4, it = item_of_round(t);
      throttle(IN_WIN);
      if ((d & 15) == 0 || (d < 16 && (d & 3) == 0)) {
        const int idx = (d < 16) ? 12 + (d >> 2) : t;
        if (d >= 16) fl = *(const unsigned*)(fbuf + lane * 4);
        if (__builtin_amdgcn_readlane(fl, idx) != call) await_flag(idx);
      }
      piece(rsrc_peer, ring + (d & 15) * 1024, vlin,
            ((it >= 8) ? OFF_WS + rank * MiB + (it - 8) * (256 * KiB) : OFF_XG + peer * (2 * MiB) + it * (256 * KiB)) + w * (16 * KiB) + (d & 15) * 1024u);
      if ((d & 15) == 7) refresh();
      if (d >= IN_WIN) put(d - IN_WIN);
    }

    int c = 192 - IN_WIN;
    unsigned last_op = 0;
    for (int hu = -1; hu < 16; ++hu) {
      const int nx = hu + 1;
      const int win = (hu < 0) ? IN_WIN : (nx >= TAIL_FROM ? TAIL_WIN : OUT_WIN);
      unsigned nb = 0;
      if (nx < 16) {
        fl = *(const unsigned*)(fbuf + lane * 4);
        if (__builtin_amdgcn_readlane(fl, 33 + 2 * nx) != call) {
          if (c < 192) { WAIT_VM(0); for (; c < 192; ++c) put(c); }
          await_flag(33 + 2 * nx);
        }
        const unsigned cn = __builtin_amdgcn_readlane(fl, 32 + 2 * nx);
        nb = dead ? 0u : ((512u + (cn > 4096u ? 4096u : cn) * 4u + 63u) & ~63u);
      }
      const int n = (int)((nb + 1023u) >> 10);
      const auto rs = mk_rsrc(pw + OFF_OS + rank * OS_OWNER + (w * 16 + (nx & 15)) * OSLOT, nb);
      unsigned char* const nbuf = ubuf + (nx & 1) * OSLOT;
      int i = 0;
      refresh();
      const unsigned refresh_op = ops - 1u;
      if (hu >= 0) {
        for (; i < n && i < win; ++i) { throttle(win); piece(rs, nbuf + i * 1024, vlin + i * 1024, 0); }
        wait_vm_dyn(ops - 1u - last_op);
        const unsigned char* const cbuf = ubuf + (hu & 1) * OSLOT;
        v2u mk = *(const v2u*)(cbuf + lane * 8);
        const unsigned so = hu * 32u * (GN * 4) + (peer * NS + w * 128) * 4u;
        if (dead) { mk = v2u{~0u, ~0u}; *(LAS unsigned*)(size_t)(zaddr + lane * 4) = 0x7fc00000u; }
        const unsigned np = 64u - __popc(mk[0]) - __popc(mk[1]);
        unsigned qa = lds_addr(cbuf) + 512u + (wave_scan(np) - np) * 4u;
        static_for<4>([&](auto qc) {
          constexpr int q = decltype(qc)::v;
          v4u o[4];
          static_for<4>([&](auto ec) {
            constexpr int e = decltype(ec)::v, idx = (q * 4 + e) * 4;
            unsigned a[4];
            run_addr4<31 - (idx & 31)>(mk[idx >> 5], zaddr, qa, a);
#pragma unroll
            for (int f = 0; f < 4; ++f) o[e][f] = *(const LAS unsigned*)(size_t)(a[f] + (idx + f) * 4);
          });
#pragma unroll
          for (int e = 0; e < 4; ++e) STORE16(o[e], rsrc_out, vout, so + (q * 8 + e) * (GN * 4), OUT_AUX);
          ops += 4;
          if (i < n) { throttle(win); piece(rs, nbuf + i * 1024, vlin + i * 1024, 0); ++i; }
        });
      }
      for (; i < n; ++i) {
        throttle(win); piece(rs, nbuf + i * 1024, vlin + i * 1024, 0);
        if (c < 192) put(c++);
      }
      last_op = (unsigned)(ops - 1u - ((ops - 1u - (unsigned)hist) & 255u));
      if (hu < 0) {
        if (c < 192) { WAIT_VM(0); for (; c < 192; ++c) put(c); }
        if (pend_round) { wait_vm_dyn(ops - 1u - pend_op); settle(pend_op + 1u); }
        WAIT_LGKM0();
      }
      wait_vm_dyn(ops - 1u - refresh_op);
    }
    WAIT_VM(0);
    STAMP(5);
    return;
  }

  const int g = slot - 4;
  const int wgi = g * 8 + xcd;
  const auto rsrc_me = mk_rsrc(me, WS_USED);
  {
    const int jb = wave * 224 + wgi;
    if (jb >= 64 && jb < 576) {
      const int s = (jb - 64) >> 7, q = ((jb - 64) >> 4) & 7, j = jb & 15;
      STAGE_DELAY(s == 0 ? DELAY1 : DELAY2);
      const auto rw = mk_rsrc(P.w, KS * GN);
      const unsigned voff = h * 16 * GN + r * 4;
      const unsigned so = s * 128 * GN + q * NS + j * 128;
      unsigned d[64];
#pragma unroll
      for (int e = 0; e < 64; ++e)
        d[e] = __builtin_amdgcn_raw_buffer_load_b32(rw, voff, so + ((e >> 4) * 32 + (e & 15)) * GN, 0);
      const unsigned dst = ((q == rank) ? OFF_WG + rank * MiB : OFF_WS + q * MiB) + s * (256 * KiB) + j * (16 * KiB);
#pragma unroll
      for (int ki = 0; ki < 4; ++ki) {
        v4u o[4];
#pragma unroll
        for (int gq = 0; gq < 4; ++gq) {
          unsigned a = d[ki * 16 + gq * 4], b = d[ki * 16 + gq * 4 + 1], c = d[ki * 16 + gq * 4 + 2], e = d[ki * 16 + gq * 4 + 3];
          unsigned t0 = __builtin_amdgcn_perm(b, a, 0x05010400u), t1 = __builtin_amdgcn_perm(b, a, 0x07030602u);
          unsigned t2 = __builtin_amdgcn_perm(e, c, 0x05010400u), t3 = __builtin_amdgcn_perm(e, c, 0x07030602u);
          o[0][gq] = __builtin_amdgcn_perm(t2, t0, 0x05040100u);
          o[1][gq] = __builtin_amdgcn_perm(t2, t0, 0x07060302u);
          o[2][gq] = __builtin_amdgcn_perm(t3, t1, 0x05040100u);
          o[3][gq] = __builtin_amdgcn_perm(t3, t1, 0x07060302u);
        }
#pragma unroll
        for (int f = 0; f < 4; ++f)
          STORE16(o[f], rsrc_me, lane * 16, dst + f * 4096 + ki * 1024, AUX_SYS);
      }
      WAIT_VM(0);
      if (lane == 0)
        __hip_atomic_store((gu32*)(P.ws[q] + OFF_FLAG) + (rank * 16 + j) * 64 + round_of_w(s), call_v, __ATOMIC_RELAXED, __HIP_MEMORY_SCOPE_SYSTEM);
    } else if (jb < 688) {
      const bool quarter = jb < 64;
      const int un = quarter ? (jb & 15) : jb - 576 + 16;
      const int rb0 = quarter ? (jb >> 4) : 0, nrb = quarter ? 1 : 4;
      if (!quarter) STAGE_DELAY(DELAY2);
      const auto rx = mk_rsrc(P.x, GM * KS);
      const unsigned voff = r * KS + h * 16;
      const unsigned so = ((un & 7) * MS + (un >> 5) * 128) * KS + ((un >> 3) & 3) * 128;
      unsigned char* const dws = P.ws[lane & 7];
      const unsigned dst = OFF_XG + rank * (2 * MiB) + un * (16 * KiB);
      for (int rb = rb0; rb < rb0 + nrb; ++rb) {
        v4u d[4];
#pragma unroll
        for (int e = 0; e < 4; ++e) d[e] = __builtin_amdgcn_raw_buffer_load_b128(rx, voff + e * 32, so + rb * 32 * KS, 0);
#pragma unroll
        for (int e = 0; e < 4; ++e) STORE16(d[e], rsrc_me, lane * 16 + e * 1024, dst + rb * 4096, AUX_SYS);
      }
      WAIT_VM(0);
      if (lane < NR)
        __hip_atomic_store((gu32*)(dws + OFF_FLAG) + (rank * 16 + (un & 15)) * 64 + (quarter ? 12 + rb0 : round_of_x(un >> 4)), call_v, __ATOMIC_RELAXED, __HIP_MEMORY_SCOPE_SYSTEM);
    }
  }
  const unsigned call = __builtin_amdgcn_readfirstlane(call_v);
  if (lane == 0) __hip_atomic_store(cnt, call, __ATOMIC_RELAXED, __HIP_MEMORY_SCOPE_AGENT);
  STAMP(1);

  const float scale = P.sx[0] * P.sw[0];
  const int ntiles = (g < 4) ? 2 : 1;
  const int wm = wave >> 1, wn = wave & 1;
  for (int it = 0; it < ntiles; ++it) {
    const int t14 = (g >= 14) ? g - 14 : g;
    const int ch = (xcd & 1) + (it ? (g & 2) : (g >= 14 ? 2 : 0));
    const int own = it ? rank : ((rank + 1 + (t14 >> 1)) & 7);
    const int jj = (xcd >> 1) * 2 + (g & 1);
    const int wh = lane & 3;
    const int wsel = (wh < 2) ? 2 * jj + wh : own + (wh - 2) * 8;
    const gu32* paddr = (const gu32*)(me + OFF_PROG) + ((rank + 1 + ((lane >> 2) & 7)) & 7) * 32 + wsel;
    v2u* const order = (v2u*)(lds + LDS_ORDER);
    unsigned nxt = (lane < 28) ? 0u : 4u;
    int nq = 4;
    auto refill = [&](int want) {
      const u64 t0 = __builtin_amdgcn_s_memrealtime();
      for (;;) {
        const unsigned pval = dead ? 0xffffffffu : __hip_atomic_load(paddr, __ATOMIC_RELAXED, __HIP_MEMORY_SCOPE_AGENT);
        for (int pass = 0; pass < 4; ++pass) {
          const unsigned thr = call * 16u + 1u + round_of_item((wh < 2) ? 8 + (nxt & 3u) : ch * 2 + ((nxt >> 1) & 1u));
          const bool ok = nxt < 4u && (pval >= thr || (wh >= 2 && (unsigned)(wh - 2) != (nxt & 1u)));
          const unsigned m = (unsigned)__ballot(ok);
          const unsigned rdy = m & (m >> 1) & (m >> 2) & (m >> 3) & 0x01111111u;
          if (!rdy) break;
          const bool mine = lane < 32 && ((rdy >> (lane & 28)) & 1u);
          if (mine && wh == 0) {
            const unsigned p = (rank + 1 + (lane >> 2)) & 7;
            order[nq + __builtin_amdgcn_mbcnt_lo(rdy, 0u)] = v2u{p * (2 * MiB) + nxt * (128 * KiB), p * MiB + nxt * (256 * KiB)};
          }
          nxt += mine ? 1u : 0u;
          nq += __popc(rdy);
        }
        if (nq > want) break;
        if (give_up(t0, tmo, call)) dead = true;
      }
      __builtin_amdgcn_s_waitcnt(0x0F70);
      __builtin_amdgcn_fence(__ATOMIC_ACQUIRE, "wavefront");
    };

    const unsigned sa0 = OFF_XG + ch * (512 * KiB) + own * (16 * KiB) + wave * 4096;
    const unsigned sb0 = OFF_WG + jj * (32 * KiB) + wave * 8192;
    v16f acc[2][4];
#pragma unroll
    for (int mi = 0; mi < 2; ++mi)
#pragma unroll
      for (int f = 0; f < 4; ++f)
#pragma unroll
        for (int e = 0; e < 16; ++e) acc[mi][f][e] = 0.f;

    if (wave == 0) {
      if (lane < 8) order[32 + lane] = v2u{0u, 0u};
      const int wq = lane >> 4;
      const gu32* a = (const gu32*)(me + OFF_FLAG) + (rank * 16 + ((wq < 2) ? 2 * jj + wq : own + (wq - 2) * 8)) * 64 + (lane & 15);
      const u64 wbits = (1u << round_of_w(0)) | (1u << round_of_w(1)) | (1u << round_of_w(2)) | (1u << round_of_w(3));
      const u64 xbits = (ch ? (1u << round_of_x(ch * 2)) : 0xf000u) | (1u << round_of_x(ch * 2 + 1));
      unsigned dummy;
      wait_words(a, call, wbits | (wbits << 16) | (xbits << 32) | (xbits << 48), tmo, call, dead, dummy);
    }
    __builtin_amdgcn_s_barrier();
#define MF(A, B, mi, f) acc[mi][f] = __builtin_amdgcn_mfma_scale_f32_32x32x64_f8f6f4(A[mi], B[f], acc[mi][f], 0, 0, 0, 0, 0, 0)
#define DM(l, so, o) __builtin_amdgcn_raw_ptr_buffer_load_lds(rs, (LAS void*)(l), 16, lane * 16, (so), (o), GEMM_AUX)
#define FR(dst, ptr) dst = __builtin_shufflevector(*(const v4i*)(ptr), *(const v4i*)((ptr) + 1024), 0, 1, 2, 3, 4, 5, 6, 7)
#define SB() __builtin_amdgcn_sched_barrier(0)
#pragma unroll
    for (int s = 0; s < 3; ++s) {
      const auto rs = rsrc_me;
      const unsigned sa = sa0 + rank * (2 * MiB) + s * (128 * KiB), sb = sb0 + rank * MiB + s * (256 * KiB);
      unsigned char* const la = lds + s * STAGE_BYTES + wave * 4096;
      unsigned char* const lb = lds + s * STAGE_BYTES + STAGE_A + wave * 8192;
      DM(la, sa, 0); DM(la, sa, 1024); DM(lb, sb, 0); DM(lb, sb, 1024); DM(lb + 4096, sb + 4096, 0); DM(lb + 4096, sb + 4096, 1024); SB();
      DM(la, sa, 2048); DM(la, sa, 3072); DM(lb, sb, 2048); DM(lb, sb, 3072); DM(lb + 4096, sb + 4096, 2048); DM(lb + 4096, sb + 4096, 3072); SB();
    }
    WAIT_VM(30);
    __builtin_amdgcn_s_barrier();
    v8i A0[2], B0[4], A1[2], B1[4];
    STAMP(6 + it);
    const unsigned char* const ra0 = lds + lane * 16 + wm * 8192;
    const unsigned char* const rb0 = lds + lane * 16 + STAGE_A + wn * 16384;
    FR(A0[0], ra0); FR(A0[1], ra0 + 4096); FR(B0[0], rb0); FR(B0[1], rb0 + 4096); FR(B0[2], rb0 + 8192); FR(B0[3], rb0 + 12288);
    int buf = 0;
    v2u ent = {rank * (2 * MiB) + 3 * (128 * KiB), rank * MiB + 3 * (256 * KiB)};
    for (int k = 0; k < 32; ++k) {
      const auto rs = mk_rsrc(me, k < 29 ? WS_USED : 0u);
      const unsigned sa = sa0 + __builtin_amdgcn_readfirstlane(ent[0]);
      const unsigned sb = sb0 + __builtin_amdgcn_readfirstlane(ent[1]);
      unsigned char* const la = lds + buf + wave * 4096;
      unsigned char* const lb = lds + buf + STAGE_A + wave * 8192;
      const unsigned char* ra = ra0 + buf + 2048;
      const unsigned char* rb = rb0 + buf + 2048;
      WAIT_LGKM0();
      WAIT_VM(24);
      __builtin_amdgcn_s_barrier();
      SB();
      MF(A0, B0, 0, 0); SB(); FR(A1[0], ra); DM(la, sa, 0); SB();
      MF(A0, B0, 0, 1); SB(); FR(B1[0], rb); DM(la, sa, 1024); SB();
      MF(A0, B0, 0, 2); SB(); FR(B1[1], rb + 4096); DM(lb, sb, 0); SB();
      MF(A0, B0, 0, 3); SB(); FR(B1[2], rb + 8192); DM(lb, sb, 1024); SB();
      MF(A0, B0, 1, 0); SB(); FR(B1[3], rb + 12288); DM(lb + 4096, sb + 4096, 0); SB();
      MF(A0, B0, 1, 1); SB(); FR(A1[1], ra + 4096); DM(lb + 4096, sb + 4096, 1024); SB();
      MF(A0, B0, 1, 2); SB();
      buf = (buf == 2 * STAGE_BYTES) ? 0 : buf + STAGE_BYTES;
      ra = ra0 + buf; rb = rb0 + buf;
      MF(A0, B0, 1, 3); SB();
      if (wave == 0 && k < 28 && nq <= k + 4) refill(k + 4);
      WAIT_LGKM0();
      WAIT_VM(24);
      __builtin_amdgcn_s_barrier();
      SB();
      MF(A1, B1, 0, 0); SB(); ent = order[k + 4]; FR(A0[0], ra); DM(la, sa, 2048); SB();
      MF(A1, B1, 0, 1); SB(); FR(B0[0], rb); DM(la, sa, 3072); SB();
      MF(A1, B1, 0, 2); SB(); FR(B0[1], rb + 4096); DM(lb, sb, 2048); SB();
      MF(A1, B1, 0, 3); SB(); FR(B0[2], rb + 8192); DM(lb, sb, 3072); SB();
      MF(A1, B1, 1, 0); SB(); FR(B0[3], rb + 12288); DM(lb + 4096, sb + 4096, 2048); SB();
      MF(A1, B1, 1, 1); SB(); FR(A0[1], ra + 4096); DM(lb + 4096, sb + 4096, 3072); SB();
      MF(A1, B1, 1, 2); SB();
      MF(A1, B1, 1, 3); SB();
    }
    WAIT_VM(0);
    WAIT_LGKM0();
    __builtin_amdgcn_s_barrier();
    STAMP(2 + 2 * it);
    const bool poison = __builtin_amdgcn_readfirstlane(__hip_atomic_load(tmo, __ATOMIC_RELAXED, __HIP_MEMORY_SCOPE_AGENT)) == call;
    const float sc = poison ? __uint_as_float(0x7fc00000u) : scale;
    if (own == rank) {
      const auto rdst = mk_rsrc(P.out, OUT_BYTES);
      const unsigned vo = h * 4 * (GN * 4) + r * 16;
      const unsigned sbase = (ch * 128 + wm * 64) * (GN * 4) + (rank * NS + jj * 256 + wn * 128) * 4;
#pragma unroll
      for (int mi = 0; mi < 2; ++mi)
#pragma unroll
        for (int e = 0; e < 16; ++e) {
          v4u o;
#pragma unroll
          for (int f = 0; f < 4; ++f) { const float y = acc[mi][f][e] * sc; const unsigned b = __float_as_uint(y); o[f] = b & ~(unsigned)((int)b >> 31); }
          STORE16(o, rdst, vo, sbase + (mi * 32 + (e >> 2) * 8 + (e & 3)) * (GN * 4), AUX_DEV);
        }
    } else {
      unsigned cnt0 = 0;
#pragma unroll
      for (int mi = 0; mi < 2; ++mi) {
        unsigned char* const cb = lds + (wave * 2 + mi) * OSLOT;
        float y[64];
        v2u mk = {0u, 0u};
#pragma unroll
        for (int i = 0; i < 64; ++i) {
          y[i] = acc[mi][i & 3][i >> 2] * sc;
          mk[i >> 5] = __builtin_amdgcn_alignbit(mk[i >> 5], __float_as_uint(y[i]), 31);
        }
        *(v2u*)(cb + lane * 8) = mk;
        const unsigned np = 64u - __popc(mk[0]) - __popc(mk[1]);
        const unsigned inc = wave_scan(np);
        const unsigned base = __builtin_amdgcn_readlane(inc, 63);
        unsigned qa = lds_addr(cb) + 512u + (inc - np) * 4u;
        const unsigned junk = lds_addr(lds + LDS_FLAGS + wave * 512) + lane * 4;
        static_for<16>([&](auto ec) {
          constexpr int idx = decltype(ec)::v * 4;
          unsigned a[4];
          run_addr4<31 - (idx & 31)>(mk[idx >> 5], junk, qa, a);
#pragma unroll
          for (int f = 0; f < 4; ++f) *(LAS float*)(size_t)(a[f] + (idx + f) * 4) = y[idx + f];
        });
        gu64* const oflag = (gu64*)(P.ws[own] + OFF_FLAG) + (rank * 16 + jj * 2 + wn) * 32 + 16 + ch * 4 + wm * 2;
        if (mi == 1) {
          WAIT_VM(0);
          if (lane == 0) __hip_atomic_store(oflag, ((u64)call << 32) | cnt0, __ATOMIC_RELAXED, __HIP_MEMORY_SCOPE_SYSTEM);
        }
        const unsigned dst = OFF_OS + own * OS_OWNER + ((jj * 2 + wn) * 16 + ch * 4 + wm * 2 + mi) * OSLOT;
        const int n = (int)((512u + base * 4u + 1023u) >> 10);
        for (int i = 0; i < n; i += 3) {
          const auto rslot = mk_rsrc(me + dst, OSLOT);
          v4u t[3];
#pragma unroll
          for (int u = 0; u < 3; ++u) t[u] = *(const v4u*)(cb + (i + u) * 1024 + lane * 16);
#pragma unroll
          for (int u = 0; u < 3; ++u) STORE16(t[u], rslot, lane * 16 + (i + u) * 1024, 0, AUX_SYS);
        }
        if (mi == 0) cnt0 = base;
        else {
          WAIT_VM(0);
          if (lane == 0) __hip_atomic_store(oflag + 1, ((u64)call << 32) | base, __ATOMIC_RELAXED, __HIP_MEMORY_SCOPE_SYSTEM);
        }
      }
      WAIT_LGKM0();
    }
    STAMP(3 + 2 * it);
  }
}

}

extern "C" void kernel_launch(int rank, int nranks, ncclComm_t comm,
                              hipStream_t stream,
                              void* const* d_in, const int* in_sizes, int n_in,
                              void* d_out, int out_size,
                              const int* in_dtypes, int out_dtype,
                              void* const* d_out_all,
                              void* const* d_ws_all, size_t ws_bytes) {
  if (nranks != NR || n_in < 4 || ws_bytes < WS_USED || out_size != MS * GN ||
      in_sizes[0] != GM * KS || in_sizes[1] != KS * GN) {
    fprintf(stderr, "unexpected problem shape\n");
    return;
  }
  Params P;
  P.x = (const unsigned char*)d_in[0];
  P.w = (const unsigned char*)d_in[1];
  P.sx = (const float*)d_in[2];
  P.sw = (const float*)d_in[3];
  P.out = (float*)d_out;
  for (int p = 0; p < NR; ++p) P.ws[p] = (unsigned char*)d_ws_all[p];
  P.rank = rank;
  hipLaunchKernelGGL(fused_kernel, dim3(256), dim3(256), 0, stream, P);
  HIP_CHECK(hipGetLastError());
}
```

```cpp
#include <hip/hip_runtime.h>
#include <rccl/rccl.h>
#include <cstdio>
#include <cstdint>

#define HIP_CHECK(expr) do { hipError_t _e = (expr); if (_e != hipSuccess) { \
    fprintf(stderr, "HIP error %s at %s:%d\n", hipGetErrorString(_e), __FILE__, __LINE__); \
    return; } } while (0)

namespace {

typedef int v8i __attribute__((ext_vector_type(8)));
typedef int v4i __attribute__((ext_vector_type(4)));
typedef unsigned v4u __attribute__((ext_vector_type(4)));
typedef unsigned v2u __attribute__((ext_vector_type(2)));
typedef float v16f __attribute__((ext_vector_type(16)));
typedef unsigned long long u64;
#define LAS __attribute__((address_space(3)))
#define GAS __attribute__((address_space(1)))
typedef GAS unsigned gu32;
typedef GAS u64 gu64;

constexpr int NR = 8;
constexpr int GM = 4096, GK = 4096, GN = 16384;
constexpr int KS = GK / NR;
constexpr int MS = GM / NR;
constexpr int NS = GN / NR;

constexpr unsigned KiB = 1024u, MiB = 1024u * 1024u;
constexpr unsigned OFF_CNT   = 0;
constexpr unsigned OFF_TMO   = 64 * KiB;
constexpr unsigned OFF_PROG  = 68 * KiB;
constexpr unsigned OFF_TICK  = 72 * KiB;
constexpr unsigned OFF_SFLAG = 80 * KiB;
constexpr unsigned OFF_OFLAG = 96 * KiB;
constexpr unsigned OFF_DBG   = 128 * KiB;
constexpr unsigned OFF_XG    = 1 * MiB;
constexpr unsigned OFF_WG    = OFF_XG + 16 * MiB;
constexpr unsigned OFF_WS    = OFF_WG + 8 * MiB;
constexpr unsigned OFF_OS    = OFF_WS + 8 * MiB;
constexpr unsigned OSLOT     = 17 * KiB;
constexpr unsigned OS_OWNER  = 256 * OSLOT;
constexpr unsigned WS_USED   = OFF_OS + 8 * OS_OWNER;
constexpr unsigned OUT_BYTES = MS * GN * 4;

constexpr unsigned long long DEADLINE = 50000000ull;
#define TOPO_MASK 0x4Du
constexpr int AUX_SYS = 17;
constexpr int AUX_DEV = 16;
#define GEMM_AUX 16

constexpr int STAGE_A = 16 * 1024, STAGE_B = 32 * 1024, STAGE_BYTES = STAGE_A + STAGE_B;
constexpr int LDS_ORDER = 3 * STAGE_BYTES;
constexpr int LDS_BYTES = LDS_ORDER + 512;
constexpr int LDS_FLAGS = 8 * OSLOT;
static_assert(LDS_FLAGS + 5120 <= LDS_BYTES, "lds");

struct Params {
  const unsigned char* x;
  const unsigned char* w;
  const float* sx;
  const float* sw;
  float* out;
  unsigned char* ws[NR];
  int rank;
};

#define WAIT_VM(n) asm volatile("s_waitcnt vmcnt(" #n ")" ::: "memory")
#define STORE16(val, rs, voff, soff, aux) do { const v4u sv_ = (val); \
    __builtin_amdgcn_raw_buffer_store_b128(sv_, rs, voff, soff, aux); \
    asm volatile("s_nop 1" :: "v"(sv_) : "memory"); } while (0)
#define DELAY1 24
#define DELAY2 72
#define STAGE_DELAY(n) do { if ((n) == DELAY1) __builtin_amdgcn_s_sleep(DELAY1); else __builtin_amdgcn_s_sleep(DELAY2); } while (0)
template <int N> __device__ __forceinline__ void wait_vm() { asm volatile("s_waitcnt vmcnt(%0)" :: "i"(N) : "memory"); }
#define OUT_AUX 16
#define DRAW_AT 0
#define DYN_FROM 208u
#define POLL_SLEEP 4
#define IN_WIN 4
#define OUT_WIN 4
#define TAIL_WIN 4
#define TAIL_FROM 224
__device__ __forceinline__ void wait_vm_dyn(unsigned n) {
  n = n > 31u ? 31u : n;
#define WV1(k) "s_waitcnt vmcnt(" #k ")\n\ts_branch 1f\n\t"
  asm volatile("s_getpc_b64 vcc\n\t"
               "s_lshl3_add_u32 %0, %0, 16\n\t"
               "s_add_u32 vcc_lo, vcc_lo, %0\n\t"
               "s_addc_u32 vcc_hi, vcc_hi, 0\n\t"
               "s_setpc_b64 vcc\n\t"
               WV1(0) WV1(1) WV1(2) WV1(3) WV1(4) WV1(5) WV1(6) WV1(7) WV1(8) WV1(9) WV1(10) WV1(11) WV1(12) WV1(13) WV1(14) WV1(15)
               WV1(16) WV1(17) WV1(18) WV1(19) WV1(20) WV1(21) WV1(22) WV1(23) WV1(24) WV1(25) WV1(26) WV1(27) WV1(28) WV1(29) WV1(30) WV1(31)
               "1:"
               : "+s"(n) :: "memory", "scc", "vcc");
#undef WV1
}
#define WAIT_LGKM0() asm volatile("s_waitcnt lgkmcnt(0)" ::: "memory")

__device__ __forceinline__ __amdgpu_buffer_rsrc_t mk_rsrc(const void* p, unsigned bytes) {
  return __builtin_amdgcn_make_buffer_rsrc((void*)p, 0, bytes, 0x00020000);
}
__device__ __forceinline__ unsigned wave_scan(unsigned x) {
  x += __builtin_amdgcn_update_dpp(0u, x, 0x111, 0xf, 0xf, false);
  x += __builtin_amdgcn_update_dpp(0u, x, 0x112, 0xf, 0xf, false);
  x += __builtin_amdgcn_update_dpp(0u, x, 0x114, 0xf, 0xf, false);
  x += __builtin_amdgcn_update_dpp(0u, x, 0x118, 0xf, 0xf, false);
  x += __builtin_amdgcn_update_dpp(0u, x, 0x142, 0xa, 0xf, false);
  x += __builtin_amdgcn_update_dpp(0u, x, 0x143, 0xc, 0xf, false);
  return x;
}
template <int P>
__device__ __forceinline__ void run_addr4(unsigned m, unsigned other, unsigned& q, unsigned (&a)[4]) {
  unsigned t;
  asm("v_bfe_i32 %5, %6, %8, 1\n\tv_bfi_b32 %0, %5, %7, %4\n\tv_lshl_add_u32 %4, %5, 2, %4\n\t"
      "v_bfe_i32 %5, %6, %9, 1\n\tv_bfi_b32 %1, %5, %7, %4\n\tv_lshl_add_u32 %4, %5, 2, %4\n\t"
      "v_bfe_i32 %5, %6, %10, 1\n\tv_bfi_b32 %2, %5, %7, %4\n\tv_lshl_add_u32 %4, %5, 2, %4\n\t"
      "v_bfe_i32 %5, %6, %11, 1\n\tv_bfi_b32 %3, %5, %7, %4\n\tv_lshl_add_u32 %4, %5, 2, %4"
      : "=&v"(a[0]), "=&v"(a[1]), "=&v"(a[2]), "=&v"(a[3]), "+v"(q), "=&v"(t)
      : "v"(m), "v"(other), "i"(P), "i"(P - 1), "i"(P - 2), "i"(P - 3));
}
template <int E> struct IC { static constexpr int v = E; };
template <int N, int I = 0, class F>
__device__ __forceinline__ void static_for(F&& f) { if constexpr (I < N) { f(IC<I>{}); static_for<N, I + 1>(f); } }
__device__ __forceinline__ unsigned lds_addr(const void* p) { return (unsigned)(size_t)(const LAS unsigned char*)p; }

#define STAMP(slot) do {} while (0)

__device__ __forceinline__ bool give_up(u64 t0, gu32* tmo, unsigned call) {
  unsigned t = __hip_atomic_load(tmo, __ATOMIC_RELAXED, __HIP_MEMORY_SCOPE_AGENT);
  if (__builtin_amdgcn_readfirstlane(t) == call || __builtin_amdgcn_s_memrealtime() - t0 > DEADLINE) {
    __hip_atomic_store(tmo, call, __ATOMIC_RELAXED, __HIP_MEMORY_SCOPE_AGENT);
    return true;
  }
  __builtin_amdgcn_s_sleep(POLL_SLEEP);
  return false;
}

__device__ __forceinline__ u64 wait_words(const gu32* addr, unsigned thr, u64 need,
                                          gu32* tmo, unsigned call, bool& dead, unsigned& val) {
  u64 t0 = __builtin_amdgcn_s_memrealtime();
  u64 ok;
  for (;;) {
    val = __hip_atomic_load(addr, __ATOMIC_RELAXED, __HIP_MEMORY_SCOPE_SYSTEM);
    ok = __ballot(val >= thr);
    if ((ok & need) == need) break;
    if (give_up(t0, tmo, call)) { dead = true; ok = ~0ull; val = 0xffffffffu; break; }
  }
  __builtin_amdgcn_fence(__ATOMIC_ACQUIRE, "wavefront");
  return ok;
}

#define ROUND_ITEMS 0x75634ba12980ull
#define ITEM_ROUNDS 0x6521b9a78340ull
__device__ __forceinline__ int item_of_round(int t) { return (int)((ROUND_ITEMS >> (4 * t)) & 15u); }
__device__ __forceinline__ int round_of_item(int it) { return (int)((ITEM_ROUNDS >> (4 * it)) & 15u); }
__device__ __forceinline__ int round_of_w(int s) { return round_of_item(8 + s); }
__device__ __forceinline__ int round_of_x(int xr) { return round_of_item(xr); }

__global__ void __launch_bounds__(256, 1) fused_kernel(Params P) {
  __shared__ __attribute__((aligned(1024))) unsigned char lds[LDS_BYTES];
  const int lane = threadIdx.x & 63;
  const int wave = __builtin_amdgcn_readfirstlane(threadIdx.x >> 6);
  const int rank = P.rank & 7;
  unsigned char* const me = P.ws[rank];
  const int xcd = blockIdx.x & 7, slot = blockIdx.x >> 3;
  gu32* const tmo = (gu32*)(me + OFF_TMO);
  STAMP(0);

  gu32* const cnt = (gu32*)(me + OFF_CNT + (blockIdx.x * 4 + wave) * 64);
  const unsigned call_v = __hip_atomic_load(cnt, __ATOMIC_RELAXED, __HIP_MEMORY_SCOPE_AGENT) + 1u;
  bool dead = false;
  const int h = lane >> 5, r = lane & 31;

  if (slot < 4) {
    const unsigned call = __builtin_amdgcn_readfirstlane(call_v);
    if (lane == 0) __hip_atomic_store(cnt, call, __ATOMIC_RELAXED, __HIP_MEMORY_SCOPE_AGENT);
    const int xcc = __builtin_amdgcn_s_getreg((20 << 0) | (0 << 6) | (3 << 11)) & 7;
    if (xcc < 4 && slot == 3) return;
    const unsigned quad = ((TOPO_MASK >> rank) & 1) ? TOPO_MASK : (~TOPO_MASK & 0xffu);
    unsigned cand = (xcc < 4) ? (quad & ~(1u << rank)) : (~quad & 0xffu);
    for (int i = 0; i < slot; ++i) cand &= cand - 1;
    const int peer = __builtin_ctz(cand) & 7;
    const int w = (xcc & 3) * 4 + wave;
    unsigned char* const pw = P.ws[peer];
    const auto rsrc_peer = mk_rsrc(pw, WS_USED);
    const auto rsrc_me = mk_rsrc(me, WS_USED);
    const auto rsrc_out = mk_rsrc(P.out, OUT_BYTES);
    const gu32* const sfl = (const gu32*)(me + OFF_SFLAG) + (peer * 16 + w) * 16;
    const gu32* const ofl = (const gu32*)(me + OFF_OFLAG) + peer * 512;
    const auto rsrc_sfl = mk_rsrc((const void*)sfl, 64);
    const auto rsrc_ofl = mk_rsrc((const void*)ofl, 2048);
    gu32* const prog = (gu32*)(me + OFF_PROG) + peer * 32 + w;
    const unsigned vlin = lane * 16;
    unsigned char* const ubuf = lds + wave * (2 * OSLOT);
    unsigned char* const fbuf = lds + LDS_FLAGS + wave * 1024;
    const unsigned vout = h * 4 * (GN * 4) + r * 16;
    const unsigned zaddr = lds_addr(lds + LDS_FLAGS + 4096 + wave * 256);
    *(LAS unsigned*)(size_t)(zaddr + lane * 4) = 0u;

    unsigned ops = 64;
    u64 hist = 0;
    unsigned pend_op = 0, pend_round = 0;
    unsigned fl = 0;
    auto settle = [&](unsigned done_below) {
      if (pend_round && pend_op < done_below) {
        if (lane == 0) __hip_atomic_store(prog, call * 16u + pend_round, __ATOMIC_RELAXED, __HIP_MEMORY_SCOPE_AGENT);
        ++ops; pend_round = 0;
      }
    };
    auto throttle = [&](int win) {
      const unsigned n = (ops - 1u - (unsigned)(hist >> (8 * win - 8))) & 255u;
      wait_vm_dyn(n);
      settle(ops - n);
    };
    auto piece = [&](const __amdgpu_buffer_rsrc_t rs, unsigned char* dst, unsigned voff, unsigned soff) {
      __builtin_amdgcn_raw_ptr_buffer_load_lds(rs, (LAS void*)dst, 16, voff, soff, 0, AUX_SYS);
      hist = (hist << 8) | (ops & 255u); ++ops;
    };
    auto await_flag = [&](const gu32* f, int idx) {
      const u64 t0 = __builtin_amdgcn_s_memrealtime();
      while (!dead) {
        fl = __hip_atomic_load(f + (lane & 15), __ATOMIC_RELAXED, __HIP_MEMORY_SCOPE_SYSTEM);
        if ((unsigned)__builtin_amdgcn_readlane(fl, idx) >= call) break;
        if (give_up(t0, tmo, call)) dead = true;
      }
      __builtin_amdgcn_s_waitcnt(0x0F70);
      __builtin_amdgcn_fence(__ATOMIC_ACQUIRE, "wavefront");
      settle(ops);
    };
    unsigned fbase = 0;
    auto refresh_out = [&](unsigned from) {
      fbase = from & ~1u;
      __builtin_amdgcn_raw_ptr_buffer_load_lds(rsrc_ofl, (LAS void*)fbuf, 16, vlin + fbase * 8u, 0, 0, AUX_SYS); ++ops;
    };

    unsigned char* const ring = ubuf + OSLOT;
    auto put = [&](int c) {
      const int it = item_of_round(c >> 4);
      const unsigned so = ((it >= 8) ? OFF_WG + peer * MiB + (it - 8) * (256 * KiB) : OFF_XG + peer * (2 * MiB) + it * (256 * KiB)) + w * (16 * KiB) + (c & 15) * 1024u;
      const v4u t = *(const v4u*)(ring + (c & 15) * 1024 + vlin);
      STORE16(t, rsrc_me, vlin, so, AUX_DEV);
      if ((c & 15) == 15) { pend_op = ops; pend_round = (c >> 4) + 1; }
      ++ops;
    };
    for (int d = 0; d < 192; ++d) {
      const int t = d >> 4, it = item_of_round(t);
      throttle(IN_WIN);
      if ((d & 15) == 0 || (d < 16 && (d & 3) == 0)) {
        const int idx = (d < 16) ? 12 + (d >> 2) : t;
        if (d >= 16) fl = *(const unsigned*)(fbuf + (lane & 15) * 4);
        if ((unsigned)__builtin_amdgcn_readlane(fl, idx) < call) await_flag(sfl, idx);
      }
      piece(rsrc_peer, ring + (d & 15) * 1024, vlin,
            ((it >= 8) ? OFF_WS + rank * MiB + (it - 8) * (256 * KiB) : OFF_XG + peer * (2 * MiB) + it * (256 * KiB)) + w * (16 * KiB) + (d & 15) * 1024u);
      if ((d & 15) == 7) {
        if (d < 176) { __builtin_amdgcn_raw_ptr_buffer_load_lds(rsrc_sfl, (LAS void*)fbuf, 4, lane * 4, 0, 0, AUX_SYS); ++ops; }
        else refresh_out(0);
      }
      if (d >= IN_WIN) put(d - IN_WIN);
    }

    const gu32* const tick = (const gu32*)(me + OFF_TICK + peer * 128);
    int c = 192 - IN_WIN;
    unsigned last_op = 0;
    unsigned nx = w, par = 0;
    for (int cur = -1;;) {
      const int win = (cur < 0) ? IN_WIN : (nx >= TAIL_FROM ? TAIL_WIN : OUT_WIN);
      unsigned nb = 0;
      if (nx < 256u) {
        unsigned cn = 0, tag = 0;
        if (nx - fbase < 128u) {
          const v2u f = *(const v2u*)(fbuf + (nx - fbase) * 8u);
          cn = __builtin_amdgcn_readfirstlane(f[0]); tag = __builtin_amdgcn_readfirstlane(f[1]);
        }
        if (tag < call) {
          if (c < 192) { WAIT_VM(0); for (; c < 192; ++c) put(c); }
          await_flag(ofl + (nx & ~7u) * 2u, (nx & 7u) * 2u + 1u);
          cn = __builtin_amdgcn_readlane(fl, (nx & 7u) * 2u);
        }
        nb = dead ? 0u : ((512u + (cn > 4096u ? 4096u : cn) * 4u + 63u) & ~63u);
      }
      const int n = (int)((nb + 1023u) >> 10);
      const auto rs = mk_rsrc(pw + OFF_OS + rank * OS_OWNER + (nx & 255u) * OSLOT, nb);
      unsigned char* const nbuf = ubuf + par * OSLOT;
      int i = 0;
      unsigned draw_op = 0;
      auto draw = [&]() {
        if (nx < 256u && nx + 16u >= DYN_FROM) {
          asm volatile("s_mov_b64 exec, 1\n\tglobal_atomic_add v250, %0, %1, %2 sc0\n\ts_mov_b64 exec, -1"
                       :: "v"(0u), "v"(1u), "s"(tick) : "memory", "v250");
          draw_op = ops; ++ops;
        }
      };
      refresh_out(nx);
      draw_op = ops - 1u;
      if (cur >= 0) {
        for (; i < n && i < win; ++i) { throttle(win); piece(rs, nbuf + i * 1024, vlin + i * 1024, 0); }
        wait_vm_dyn(ops - 1u - last_op);
        const unsigned char* const cbuf = ubuf + (par ^ 1u) * OSLOT;
        v2u mk = *(const v2u*)(cbuf + lane * 8);
        const unsigned so = (cur >> 4) * 32u * (GN * 4) + (peer * NS + (cur & 15) * 128) * 4u;
        if (dead) { mk = v2u{~0u, ~0u}; *(LAS unsigned*)(size_t)(zaddr + lane * 4) = 0x7fc00000u; }
        const unsigned np = 64u - __popc(mk[0]) - __popc(mk[1]);
        unsigned qa = lds_addr(cbuf) + 512u + (wave_scan(np) - np) * 4u;
        static_for<4>([&](auto qc) {
          constexpr int q = decltype(qc)::v;
          v4u o[4];
          static_for<4>([&](auto ec) {
            constexpr int e = decltype(ec)::v, idx = (q * 4 + e) * 4;
            unsigned a[4];
            run_addr4<31 - (idx & 31)>(mk[idx >> 5], zaddr, qa, a);
#pragma unroll
            for (int f = 0; f < 4; ++f) o[e][f] = *(const LAS unsigned*)(size_t)(a[f] + (idx + f) * 4);
          });
          if (q == DRAW_AT) draw();
#pragma unroll
          for (int e = 0; e < 4; ++e) STORE16(o[e], rsrc_out, vout, so + (q * 8 + e) * (GN * 4), OUT_AUX);
          ops += 4;
          if (i < n) { throttle(win); piece(rs, nbuf + i * 1024, vlin + i * 1024, 0); ++i; }
        });
      } else {
        draw();
      }
      if (nx >= 256u) break;
      for (; i < n; ++i) {
        throttle(win); piece(rs, nbuf + i * 1024, vlin + i * 1024, 0);
        if (c < 192) put(c++);
      }
      last_op = (unsigned)(ops - 1u - ((ops - 1u - (unsigned)hist) & 255u));
      if (cur < 0) {
        if (c < 192) { WAIT_VM(0); for (; c < 192; ++c) put(c); }
        if (pend_round) { wait_vm_dyn(ops - 1u - pend_op); settle(pend_op + 1u); }
        WAIT_LGKM0();
      }
      wait_vm_dyn(ops - 1u - draw_op);
      cur = (int)nx;
      if (nx + 16u >= DYN_FROM) {
        asm volatile("v_readfirstlane_b32 %0, v250" : "=s"(nx) :: "memory");
        nx = nx - (call - 1u) * (272u - DYN_FROM) + DYN_FROM;
      } else nx += 16u;
      par ^= 1u;
    }
    WAIT_VM(0);
    STAMP(5);
    return;
  }

  const int g = slot - 4;
  const int wgi = g * 8 + xcd;
  const auto rsrc_me = mk_rsrc(me, WS_USED);
  {
    const int jb = wave * 224 + wgi;
    if (jb >= 64 && jb < 576) {
      const int s = (jb - 64) >> 7, q = ((jb - 64) >> 4) & 7, j = jb & 15;
      STAGE_DELAY(s == 0 ? DELAY1 : DELAY2);
      const auto rw = mk_rsrc(P.w, KS * GN);
      const unsigned voff = h * 16 * GN + r * 4;
      const unsigned so = s * 128 * GN + q * NS + j * 128;
      unsigned d[64];
#pragma unroll
      for (int e = 0; e < 64; ++e)
        d[e] = __builtin_amdgcn_raw_buffer_load_b32(rw, voff, so + ((e >> 4) * 32 + (e & 15)) * GN, 0);
      const unsigned dst = ((q == rank) ? OFF_WG + rank * MiB : OFF_WS + q * MiB) + s * (256 * KiB) + j * (16 * KiB);
#pragma unroll
      for (int ki = 0; ki < 4; ++ki) {
        v4u o[4];
#pragma unroll
        for (int gq = 0; gq < 4; ++gq) {
          unsigned a = d[ki * 16 + gq * 4], b = d[ki * 16 + gq * 4 + 1], c = d[ki * 16 + gq * 4 + 2], e = d[ki * 16 + gq * 4 + 3];
          unsigned t0 = __builtin_amdgcn_perm(b, a, 0x05010400u), t1 = __builtin_amdgcn_perm(b, a, 0x07030602u);
          unsigned t2 = __builtin_amdgcn_perm(e, c, 0x05010400u), t3 = __builtin_amdgcn_perm(e, c, 0x07030602u);
          o[0][gq] = __builtin_amdgcn_perm(t2, t0, 0x05040100u);
          o[1][gq] = __builtin_amdgcn_perm(t2, t0, 0x07060302u);
          o[2][gq] = __builtin_amdgcn_perm(t3, t1, 0x05040100u);
          o[3][gq] = __builtin_amdgcn_perm(t3, t1, 0x07060302u);
        }
#pragma unroll
        for (int f = 0; f < 4; ++f)
          STORE16(o[f], rsrc_me, lane * 16, dst + f * 4096 + ki * 1024, AUX_SYS);
      }
      WAIT_VM(0);
      if (lane == 0)
        __hip_atomic_store((gu32*)(P.ws[q] + OFF_SFLAG) + (rank * 16 + j) * 16 + round_of_w(s), call_v, __ATOMIC_RELAXED, __HIP_MEMORY_SCOPE_SYSTEM);
    } else if (jb < 688) {
      const bool quarter = jb < 64;
      const int un = quarter ? (jb & 15) : jb - 576 + 16;
      const int rb0 = quarter ? (jb >> 4) : 0, nrb = quarter ? 1 : 4;
      if (!quarter) STAGE_DELAY(DELAY2);
      const auto rx = mk_rsrc(P.x, GM * KS);
      const unsigned voff = r * KS + h * 16;
      const unsigned so = ((un & 7) * MS + (un >> 5) * 128) * KS + ((un >> 3) & 3) * 128;
      unsigned char* const dws = P.ws[lane & 7];
      const unsigned dst = OFF_XG + rank * (2 * MiB) + un * (16 * KiB);
      for (int rb = rb0; rb < rb0 + nrb; ++rb) {
        v4u d[4];
#pragma unroll
        for (int e = 0; e < 4; ++e) d[e] = __builtin_amdgcn_raw_buffer_load_b128(rx, voff + e * 32, so + rb * 32 * KS, 0);
#pragma unroll
        for (int e = 0; e < 4; ++e) STORE16(d[e], rsrc_me, lane * 16 + e * 1024, dst + rb * 4096, AUX_SYS);
      }
      WAIT_VM(0);
      if (lane < NR)
        __hip_atomic_store((gu32*)(dws + OFF_SFLAG) + (rank * 16 + (un & 15)) * 16 + (quarter ? 12 + rb0 : round_of_x(un >> 4)), call_v, __ATOMIC_RELAXED, __HIP_MEMORY_SCOPE_SYSTEM);
    }
  }
  const unsigned call = __builtin_amdgcn_readfirstlane(call_v);
  if (lane == 0) __hip_atomic_store(cnt, call, __ATOMIC_RELAXED, __HIP_MEMORY_SCOPE_AGENT);
  STAMP(1);

  const float scale = P.sx[0] * P.sw[0];
  const int ntiles = (g < 4) ? 2 : 1;
  const int wm = wave >> 1, wn = wave & 1;
  for (int it = 0; it < ntiles; ++it) {
    const int t14 = (g >= 14) ? g - 14 : g;
    const int ch = (xcd & 1) + (it ? (g & 2) : (g >= 14 ? 2 : 0));
    const int own = it ? rank : ((rank + 1 + (t14 >> 1)) & 7);
    const int jj = (xcd >> 1) * 2 + (g & 1);
    const int wh = lane & 3;
    const int wsel = (wh < 2) ? 2 * jj + wh : own + (wh - 2) * 8;
    const gu32* paddr = (const gu32*)(me + OFF_PROG) + ((rank + 1 + ((lane >> 2) & 7)) & 7) * 32 + wsel;
    v2u* const order = (v2u*)(lds + LDS_ORDER);
    unsigned nxt = (lane < 28) ? 0u : 4u;
    int nq = 4;
    auto refill = [&](int want) {
      const u64 t0 = __builtin_amdgcn_s_memrealtime();
      for (;;) {
        const unsigned pval = dead ? 0xffffffffu : __hip_atomic_load(paddr, __ATOMIC_RELAXED, __HIP_MEMORY_SCOPE_AGENT);
        for (int pass = 0; pass < 4; ++pass) {
          const unsigned thr = call * 16u + 1u + round_of_item((wh < 2) ? 8 + (nxt & 3u) : ch * 2 + ((nxt >> 1) & 1u));
          const bool ok = nxt < 4u && (pval >= thr || (wh >= 2 && (unsigned)(wh - 2) != (nxt & 1u)));
          const unsigned m = (unsigned)__ballot(ok);
          const unsigned rdy = m & (m >> 1) & (m >> 2) & (m >> 3) & 0x01111111u;
          if (!rdy) break;
          const bool mine = lane < 32 && ((rdy >> (lane & 28)) & 1u);
          if (mine && wh == 0) {
            const unsigned p = (rank + 1 + (lane >> 2)) & 7;
            order[nq + __builtin_amdgcn_mbcnt_lo(rdy, 0u)] = v2u{p * (2 * MiB) + nxt * (128 * KiB), p * MiB + nxt * (256 * KiB)};
          }
          nxt += mine ? 1u : 0u;
          nq += __popc(rdy);
        }
        if (nq > want) break;
        if (give_up(t0, tmo, call)) dead = true;
      }
      __builtin_amdgcn_s_waitcnt(0x0F70);
      __builtin_amdgcn_fence(__ATOMIC_ACQUIRE, "wavefront");
    };

    const unsigned sa0 = OFF_XG + ch * (512 * KiB) + own * (16 * KiB) + wave * 4096;
    const unsigned sb0 = OFF_WG + jj * (32 * KiB) + wave * 8192;
    v16f acc[2][4];
#pragma unroll
    for (int mi = 0; mi < 2; ++mi)
#pragma unroll
      for (int f = 0; f < 4; ++f)
#pragma unroll
        for (int e = 0; e < 16; ++e) acc[mi][f][e] = 0.f;

    if (wave == 0) {
      if (lane < 8) order[32 + lane] = v2u{0u, 0u};
      const int wq = lane >> 4;
      const gu32* a = (const gu32*)(me + OFF_SFLAG) + (rank * 16 + ((wq < 2) ? 2 * jj + wq : own + (wq - 2) * 8)) * 16 + (lane & 15);
      const u64 wbits = (1u << round_of_w(0)) | (1u << round_of_w(1)) | (1u << round_of_w(2)) | (1u << round_of_w(3));
      const u64 xbits = (ch ? (1u << round_of_x(ch * 2)) : 0xf000u) | (1u << round_of_x(ch * 2 + 1));
      unsigned dummy;
      wait_words(a, call, wbits | (wbits << 16) | (xbits << 32) | (xbits << 48), tmo, call, dead, dummy);
    }
    __builtin_amdgcn_s_barrier();
#define MF(A, B, mi, f) acc[mi][f] = __builtin_amdgcn_mfma_scale_f32_32x32x64_f8f6f4(A[mi], B[f], acc[mi][f], 0, 0, 0, 0, 0, 0)
#define DM(l, so, o) __builtin_amdgcn_raw_ptr_buffer_load_lds(rs, (LAS void*)(l), 16, lane * 16, (so), (o), GEMM_AUX)
#define FR(dst, ptr) dst = __builtin_shufflevector(*(const v4i*)(ptr), *(const v4i*)((ptr) + 1024), 0, 1, 2, 3, 4, 5, 6, 7)
#define SB() __builtin_amdgcn_sched_barrier(0)
#pragma unroll
    for (int s = 0; s < 3; ++s) {
      const auto rs = rsrc_me;
      const unsigned sa = sa0 + rank * (2 * MiB) + s * (128 * KiB), sb = sb0 + rank * MiB + s * (256 * KiB);
      unsigned char* const la = lds + s * STAGE_BYTES + wave * 4096;
      unsigned char* const lb = lds + s * STAGE_BYTES + STAGE_A + wave * 8192;
      DM(la, sa, 0); DM(la, sa, 1024); DM(lb, sb, 0); DM(lb, sb, 1024); DM(lb + 4096, sb + 4096, 0); DM(lb + 4096, sb + 4096, 1024); SB();
      DM(la, sa, 2048); DM(la, sa, 3072); DM(lb, sb, 2048); DM(lb, sb, 3072); DM(lb + 4096, sb + 4096, 2048); DM(lb + 4096, sb + 4096, 3072); SB();
    }
    WAIT_VM(30);
    __builtin_amdgcn_s_barrier();
    v8i A0[2], B0[4], A1[2], B1[4];
    STAMP(6 + it);
    const unsigned char* const ra0 = lds + lane * 16 + wm * 8192;
    const unsigned char* const rb0 = lds + lane * 16 + STAGE_A + wn * 16384;
    FR(A0[0], ra0); FR(A0[1], ra0 + 4096); FR(B0[0], rb0); FR(B0[1], rb0 + 4096); FR(B0[2], rb0 + 8192); FR(B0[3], rb0 + 12288);
    int buf = 0;
    v2u ent = {rank * (2 * MiB) + 3 * (128 * KiB), rank * MiB + 3 * (256 * KiB)};
    for (int k = 0; k < 32; ++k) {
      const auto rs = mk_rsrc(me, k < 29 ? WS_USED : 0u);
      const unsigned sa = sa0 + __builtin_amdgcn_readfirstlane(ent[0]);
      const unsigned sb = sb0 + __builtin_amdgcn_readfirstlane(ent[1]);
      unsigned char* const la = lds + buf + wave * 4096;
      unsigned char* const lb = lds + buf + STAGE_A + wave * 8192;
      const unsigned char* ra = ra0 + buf + 2048;
      const unsigned char* rb = rb0 + buf + 2048;
      WAIT_LGKM0();
      WAIT_VM(24);
      __builtin_amdgcn_s_barrier();
      SB();
      MF(A0, B0, 0, 0); SB(); FR(A1[0], ra); DM(la, sa, 0); SB();
      MF(A0, B0, 0, 1); SB(); FR(B1[0], rb); DM(la, sa, 1024); SB();
      MF(A0, B0, 0, 2); SB(); FR(B1[1], rb + 4096); DM(lb, sb, 0); SB();
      MF(A0, B0, 0, 3); SB(); FR(B1[2], rb + 8192); DM(lb, sb, 1024); SB();
      MF(A0, B0, 1, 0); SB(); FR(B1[3], rb + 12288); DM(lb + 4096, sb + 4096, 0); SB();
      MF(A0, B0, 1, 1); SB(); FR(A1[1], ra + 4096); DM(lb + 4096, sb + 4096, 1024); SB();
      MF(A0, B0, 1, 2); SB();
      buf = (buf == 2 * STAGE_BYTES) ? 0 : buf + STAGE_BYTES;
      ra = ra0 + buf; rb = rb0 + buf;
      MF(A0, B0, 1, 3); SB();
      if (wave == 0 && k < 28 && nq <= k + 4) refill(k + 4);
      WAIT_LGKM0();
      WAIT_VM(24);
      __builtin_amdgcn_s_barrier();
      SB();
      MF(A1, B1, 0, 0); SB(); ent = order[k + 4]; FR(A0[0], ra); DM(la, sa, 2048); SB();
      MF(A1, B1, 0, 1); SB(); FR(B0[0], rb); DM(la, sa, 3072); SB();
      MF(A1, B1, 0, 2); SB(); FR(B0[1], rb + 4096); DM(lb, sb, 2048); SB();
      MF(A1, B1, 0, 3); SB(); FR(B0[2], rb + 8192); DM(lb, sb, 3072); SB();
      MF(A1, B1, 1, 0); SB(); FR(B0[3], rb + 12288); DM(lb + 4096, sb + 4096, 2048); SB();
      MF(A1, B1, 1, 1); SB(); FR(A0[1], ra + 4096); DM(lb + 4096, sb + 4096, 3072); SB();
      MF(A1, B1, 1, 2); SB();
      MF(A1, B1, 1, 3); SB();
    }
    WAIT_VM(0);
    WAIT_LGKM0();
    __builtin_amdgcn_s_barrier();
    STAMP(2 + 2 * it);
    const bool poison = __builtin_amdgcn_readfirstlane(__hip_atomic_load(tmo, __ATOMIC_RELAXED, __HIP_MEMORY_SCOPE_AGENT)) == call;
    const float sc = poison ? __uint_as_float(0x7fc00000u) : scale;
    if (own == rank) {
      const auto rdst = mk_rsrc(P.out, OUT_BYTES);
      const unsigned vo = h * 4 * (GN * 4) + r * 16;
      const unsigned sbase = (ch * 128 + wm * 64) * (GN * 4) + (rank * NS + jj * 256 + wn * 128) * 4;
#pragma unroll
      for (int mi = 0; mi < 2; ++mi)
#pragma unroll
        for (int e = 0; e < 16; ++e) {
          v4u o;
#pragma unroll
          for (int f = 0; f < 4; ++f) { const float y = acc[mi][f][e] * sc; const unsigned b = __float_as_uint(y); o[f] = b & ~(unsigned)((int)b >> 31); }
          STORE16(o, rdst, vo, sbase + (mi * 32 + (e >> 2) * 8 + (e & 3)) * (GN * 4), AUX_DEV);
        }
    } else {
      unsigned cnt0 = 0;
#pragma unroll
      for (int mi = 0; mi < 2; ++mi) {
        unsigned char* const cb = lds + (wave * 2 + mi) * OSLOT;
        float y[64];
        v2u mk = {0u, 0u};
#pragma unroll
        for (int i = 0; i < 64; ++i) {
          y[i] = acc[mi][i & 3][i >> 2] * sc;
          mk[i >> 5] = __builtin_amdgcn_alignbit(mk[i >> 5], __float_as_uint(y[i]), 31);
        }
        *(v2u*)(cb + lane * 8) = mk;
        const unsigned np = 64u - __popc(mk[0]) - __popc(mk[1]);
        const unsigned inc = wave_scan(np);
        const unsigned base = __builtin_amdgcn_readlane(inc, 63);
        unsigned qa = lds_addr(cb) + 512u + (inc - np) * 4u;
        const unsigned junk = lds_addr(lds + LDS_FLAGS + wave * 512) + lane * 4;
        static_for<16>([&](auto ec) {
          constexpr int idx = decltype(ec)::v * 4;
          unsigned a[4];
          run_addr4<31 - (idx & 31)>(mk[idx >> 5], junk, qa, a);
#pragma unroll
          for (int f = 0; f < 4; ++f) *(LAS float*)(size_t)(a[f] + (idx + f) * 4) = y[idx + f];
        });
        gu64* const oflag = (gu64*)(P.ws[own] + OFF_OFLAG) + rank * 256 + (ch * 4 + wm * 2) * 16 + jj * 2 + wn;
        if (mi == 1) {
          WAIT_VM(0);
          if (lane == 0) __hip_atomic_store(oflag, ((u64)call << 32) | cnt0, __ATOMIC_RELAXED, __HIP_MEMORY_SCOPE_SYSTEM);
        }
        const unsigned dst = OFF_OS + own * OS_OWNER + ((ch * 4 + wm * 2 + mi) * 16 + jj * 2 + wn) * OSLOT;
        const int n = (int)((512u + base * 4u + 1023u) >> 10);
        for (int i = 0; i < n; i += 3) {
          const auto rslot = mk_rsrc(me + dst, OSLOT);
          v4u t[3];
#pragma unroll
          for (int u = 0; u < 3; ++u) t[u] = *(const v4u*)(cb + (i + u) * 1024 + lane * 16);
#pragma unroll
          for (int u = 0; u < 3; ++u) STORE16(t[u], rslot, lane * 16 + (i + u) * 1024, 0, AUX_SYS);
        }
        if (mi == 0) cnt0 = base;
        else {
          WAIT_VM(0);
          if (lane == 0) __hip_atomic_store(oflag + 16, ((u64)call << 32) | base, __ATOMIC_RELAXED, __HIP_MEMORY_SCOPE_SYSTEM);
        }
      }
      WAIT_LGKM0();
    }
    STAMP(3 + 2 * it);
  }
}

}

extern "C" void kernel_launch(int rank, int nranks, ncclComm_t comm,
                              hipStream_t stream,
                              void* const* d_in, const int* in_sizes, int n_in,
                              void* d_out, int out_size,
                              const int* in_dtypes, int out_dtype,
                              void* const* d_out_all,
                              void* const* d_ws_all, size_t ws_bytes) {
  if (nranks != NR || n_in < 4 || ws_bytes < WS_USED || out_size != MS * GN ||
      in_sizes[0] != GM * KS || in_sizes[1] != KS * GN) {
    fprintf(stderr, "unexpected problem shape\n");
    return;
  }
  Params P;
  P.x = (const unsigned char*)d_in[0];
  P.w = (const unsigned char*)d_in[1];
  P.sx = (const float*)d_in[2];
  P.sw = (const float*)d_in[3];
  P.out = (float*)d_out;
  for (int p = 0; p < NR; ++p) P.ws[p] = (unsigned char*)d_ws_all[p];
  P.rank = rank;
  hipLaunchKernelGGL(fused_kernel, dim3(256), dim3(256), 0, stream, P);
  HIP_CHECK(hipGetLastError());
}
```

```cpp
#include <hip/hip_runtime.h>
#include <rccl/rccl.h>
#include <cstdio>
#include <cstdint>

#define HIP_CHECK(expr) do { hipError_t _e = (expr); if (_e != hipSuccess) { \
    fprintf(stderr, "HIP error %s at %s:%d\n", hipGetErrorString(_e), __FILE__, __LINE__); \
    return; } } while (0)

namespace {

typedef int v8i __attribute__((ext_vector_type(8)));
typedef int v4i __attribute__((ext_vector_type(4)));
typedef unsigned v4u __attribute__((ext_vector_type(4)));
typedef unsigned v2u __attribute__((ext_vector_type(2)));
typedef float v16f __attribute__((ext_vector_type(16)));
typedef unsigned long long u64;
#define LAS __attribute__((address_space(3)))
#define GAS __attribute__((address_space(1)))
typedef GAS unsigned gu32;
typedef GAS u64 gu64;

constexpr int NR = 8;
constexpr int GM = 4096, GK = 4096, GN = 16384;
constexpr int KS = GK / NR;
constexpr int MS = GM / NR;
constexpr int NS = GN / NR;

constexpr unsigned KiB = 1024u, MiB = 1024u * 1024u;
constexpr unsigned OFF_CNT   = 0;
constexpr unsigned OFF_TMO   = 64 * KiB;
constexpr unsigned OFF_PROG  = 112 * KiB;
constexpr unsigned OFF_TICK  = 72 * KiB;
constexpr unsigned OFF_SFLAG = 80 * KiB;
constexpr unsigned OFF_OFLAG = 96 * KiB;
constexpr unsigned OFF_DBG   = 128 * KiB;
constexpr unsigned OFF_XG    = 1 * MiB;
constexpr unsigned OFF_WG    = OFF_XG + 16 * MiB;
constexpr unsigned OFF_WS    = OFF_WG + 8 * MiB;
constexpr unsigned OFF_OS    = OFF_WS + 8 * MiB;
constexpr unsigned OSLOT     = 17 * KiB;
constexpr unsigned OS_OWNER  = 256 * OSLOT;
constexpr unsigned OFF_XALT  = OFF_OS + 8 * OS_OWNER;
constexpr unsigned WS_USED   = OFF_XALT + 2 * MiB;
constexpr unsigned OUT_BYTES = MS * GN * 4;

constexpr unsigned long long DEADLINE = 50000000ull;
#define TOPO_MASK 0x4Du
constexpr int AUX_SYS = 17;
constexpr int AUX_DEV = 16;
#define GEMM_AUX 16

constexpr int STAGE_A = 16 * 1024, STAGE_B = 32 * 1024, STAGE_BYTES = STAGE_A + STAGE_B;
constexpr int LDS_ORDER = 3 * STAGE_BYTES;
constexpr int LDS_BYTES = LDS_ORDER + 512;
constexpr int LDS_FLAGS = 8 * OSLOT;
static_assert(LDS_FLAGS + 5120 <= LDS_BYTES, "lds");

struct Params {
  const unsigned char* x;
  const unsigned char* w;
  unsigned char* me;
  int rank, pad;
  const float* sx;
  const float* sw;
  float* out;
  unsigned char* ws[NR];
};

#define WAIT_VM(n) asm volatile("s_waitcnt vmcnt(" #n ")" ::: "memory")
#define STORE16(val, rs, voff, soff, aux) do { const v4u sv_ = (val); \
    __builtin_amdgcn_raw_buffer_store_b128(sv_, rs, voff, soff, aux); \
    asm volatile("s_nop 1" :: "v"(sv_) : "memory"); } while (0)
#define DELAY1 24
#define DELAY2 72
#define STAGE_DELAY(n) do { if ((n) == DELAY1) __builtin_amdgcn_s_sleep(DELAY1); else __builtin_amdgcn_s_sleep(DELAY2); } while (0)
template <int N> __device__ __forceinline__ void wait_vm() { asm volatile("s_waitcnt vmcnt(%0)" :: "i"(N) : "memory"); }
#define OUT_AUX 16
#define DRAW_AT 0
#define DYN_FROM 208u
#define POLL_SLEEP 4
#define PUSHQ 1
#define PROG_STRIDE 32
#define GEMM_POLL_SLEEP 0
#define IN_WIN 3
#define OUT_WIN 3
#define TAIL_WIN 3
#define TAIL_FROM 224
__device__ __forceinline__ void wait_vm_dyn(unsigned n) {
  n = n > 31u ? 31u : n;
#define WV1(k) "s_waitcnt vmcnt(" #k ")\n\ts_branch 1f\n\t"
  asm volatile("s_getpc_b64 vcc\n\t"
               "s_lshl3_add_u32 %0, %0, 16\n\t"
               "s_add_u32 vcc_lo, vcc_lo, %0\n\t"
               "s_addc_u32 vcc_hi, vcc_hi, 0\n\t"
               "s_setpc_b64 vcc\n\t"
               WV1(0) WV1(1) WV1(2) WV1(3) WV1(4) WV1(5) WV1(6) WV1(7) WV1(8) WV1(9) WV1(10) WV1(11) WV1(12) WV1(13) WV1(14) WV1(15)
               WV1(16) WV1(17) WV1(18) WV1(19) WV1(20) WV1(21) WV1(22) WV1(23) WV1(24) WV1(25) WV1(26) WV1(27) WV1(28) WV1(29) WV1(30) WV1(31)
               "1:"
               : "+s"(n) :: "memory", "scc", "vcc");
#undef WV1
}
#define WAIT_LGKM0() asm volatile("s_waitcnt lgkmcnt(0)" ::: "memory")

__device__ __forceinline__ __amdgpu_buffer_rsrc_t mk_rsrc(const void* p, unsigned bytes) {
  return __builtin_amdgcn_make_buffer_rsrc((void*)p, 0, bytes, 0x00020000);
}
__device__ __forceinline__ unsigned wave_scan(unsigned x) {
  x += __builtin_amdgcn_update_dpp(0u, x, 0x111, 0xf, 0xf, false);
  x += __builtin_amdgcn_update_dpp(0u, x, 0x112, 0xf, 0xf, false);
  x += __builtin_amdgcn_update_dpp(0u, x, 0x114, 0xf, 0xf, false);
  x += __builtin_amdgcn_update_dpp(0u, x, 0x118, 0xf, 0xf, false);
  x += __builtin_amdgcn_update_dpp(0u, x, 0x142, 0xa, 0xf, false);
  x += __builtin_amdgcn_update_dpp(0u, x, 0x143, 0xc, 0xf, false);
  return x;
}
template <int P>
__device__ __forceinline__ void run_addr4(unsigned m, unsigned other, unsigned& q, unsigned (&a)[4]) {
  unsigned t;
  asm("v_bfe_i32 %5, %6, %8, 1\n\tv_bfi_b32 %0, %5, %7, %4\n\tv_lshl_add_u32 %4, %5, 2, %4\n\t"
      "v_bfe_i32 %5, %6, %9, 1\n\tv_bfi_b32 %1, %5, %7, %4\n\tv_lshl_add_u32 %4, %5, 2, %4\n\t"
      "v_bfe_i32 %5, %6, %10, 1\n\tv_bfi_b32 %2, %5, %7, %4\n\tv_lshl_add_u32 %4, %5, 2, %4\n\t"
      "v_bfe_i32 %5, %6, %11, 1\n\tv_bfi_b32 %3, %5, %7, %4\n\tv_lshl_add_u32 %4, %5, 2, %4"
      : "=&v"(a[0]), "=&v"(a[1]), "=&v"(a[2]), "=&v"(a[3]), "+v"(q), "=&v"(t)
      : "v"(m), "v"(other), "i"(P), "i"(P - 1), "i"(P - 2), "i"(P - 3));
}
template <int E> struct IC { static constexpr int v = E; };
template <int N, int I = 0, class F>
__device__ __forceinline__ void static_for(F&& f) { if constexpr (I < N) { f(IC<I>{}); static_for<N, I + 1>(f); } }
__device__ __forceinline__ unsigned lds_addr(const void* p) { return (unsigned)(size_t)(const LAS unsigned char*)p; }

#define STAMP(slot) do {} while (0)

__device__ __forceinline__ bool give_up(u64 t0, gu32* tmo, unsigned call) {
  unsigned t = __hip_atomic_load(tmo, __ATOMIC_RELAXED, __HIP_MEMORY_SCOPE_AGENT);
  if (__builtin_amdgcn_readfirstlane(t) == call || __builtin_amdgcn_s_memrealtime() - t0 > DEADLINE) {
    __hip_atomic_store(tmo, call, __ATOMIC_RELAXED, __HIP_MEMORY_SCOPE_AGENT);
    return true;
  }
  __builtin_amdgcn_s_sleep(POLL_SLEEP);
  return false;
}

__device__ __forceinline__ u64 wait_words(const gu32* addr, unsigned thr, u64 need,
                                          gu32* tmo, unsigned call, bool& dead, unsigned& val) {
  u64 t0 = __builtin_amdgcn_s_memrealtime();
  u64 ok;
  for (;;) {
    val = __hip_atomic_load(addr, __ATOMIC_RELAXED, __HIP_MEMORY_SCOPE_SYSTEM);
    ok = __ballot(val >= thr);
    if ((ok & need) == need) break;
    if (give_up(t0, tmo, call)) { dead = true; ok = ~0ull; val = 0xffffffffu; break; }
  }
  __builtin_amdgcn_fence(__ATOMIC_ACQUIRE, "wavefront");
  return ok;
}

#define ROUND_ITEMS 0x75634ba12980ull
#define ITEM_ROUNDS 0x6521b9a78340ull
__device__ __forceinline__ int item_of_round(int t) { return (int)((ROUND_ITEMS >> (4 * t)) & 15u); }
__device__ __forceinline__ int round_of_item(int it) { return (int)((ITEM_ROUNDS >> (4 * it)) & 15u); }
__device__ __forceinline__ int round_of_w(int s) { return round_of_item(8 + s); }
__device__ __forceinline__ int round_of_x(int xr) { return round_of_item(xr); }

__device__ __forceinline__ unsigned x_item_off(int p, int it, unsigned call) {
  return (it == 0 && (call & 1u)) ? OFF_XALT + p * (256 * KiB) : OFF_XG + p * (2 * MiB) + it * (256 * KiB);
}

__global__ void __launch_bounds__(256, 1) fused_kernel(Params P) {
  __shared__ __attribute__((aligned(1024))) unsigned char lds[LDS_BYTES];
  const int lane = threadIdx.x & 63;
  const int wave = __builtin_amdgcn_readfirstlane(threadIdx.x >> 6);
  const int rank = P.rank & 7;
  unsigned char* const me = P.me;
  const unsigned char* xp = P.x;
  const unsigned char* wp = P.w;
  u64 w0 = (u64)P.ws[0], w1 = (u64)P.ws[1], w2 = (u64)P.ws[2], w3 = (u64)P.ws[3];
  u64 w4 = (u64)P.ws[4], w5 = (u64)P.ws[5], w6 = (u64)P.ws[6], w7 = (u64)P.ws[7];
  asm volatile("" : "+s"(xp), "+s"(wp), "+s"(w0), "+s"(w1), "+s"(w2), "+s"(w3), "+s"(w4), "+s"(w5), "+s"(w6), "+s"(w7));
  auto ws_of = [=](int p) {
    const u64 a = (p & 1) ? w1 : w0, b = (p & 1) ? w3 : w2, c = (p & 1) ? w5 : w4, d = (p & 1) ? w7 : w6;
    const u64 e = (p & 2) ? b : a, f = (p & 2) ? d : c;
    return (unsigned char*)((p & 4) ? f : e);
  };
  const int xcd = blockIdx.x & 7, slot = blockIdx.x >> 3;
  gu32* const tmo = (gu32*)(me + OFF_TMO);
  const int xcc = __builtin_amdgcn_s_getreg((20 << 0) | (0 << 6) | (3 << 11)) & 7;
  const unsigned quad = ((TOPO_MASK >> rank) & 1) ? TOPO_MASK : (~TOPO_MASK & 0xffu);
  unsigned cand = (xcc < 4) ? (quad & ~(1u << rank)) : (~quad & 0xffu);
  for (int i = 0; i < (slot & 3); ++i) cand &= cand - 1;
  const int peer = __builtin_ctz(cand | 0x80u) & 7;
  const int jb = wave * 224 + (slot - 4) * 8 + xcd;
  const int tgt = (slot < 4) ? peer : (jb < 576) ? (((jb - 64) >> 4) & 7) : (jb >= 688) ? ((rank + 1 + ((jb - 688) >> 4)) & 7) : rank;
  unsigned char* const tws = ws_of(tgt);
  unsigned char* const dws = ws_of(lane & 7);
  STAMP(0);

  gu32* const cnt = (gu32*)(me + OFF_CNT + (blockIdx.x * 4 + wave) * 64);
  const unsigned prev_v = __hip_atomic_load(cnt, __ATOMIC_RELAXED, __HIP_MEMORY_SCOPE_AGENT);
#define CALL_NOW() ((unsigned)__builtin_amdgcn_readfirstlane(prev_v) + 1u)
  bool dead = false;
  const int h = lane >> 5, r = lane & 31;

  if (slot < 4) {
    const int w = (xcc & 3) * 4 + wave;
    unsigned fl0 = __hip_atomic_load((const gu32*)(me + OFF_SFLAG) + (peer * 16 + w) * 16 + (lane & 15), __ATOMIC_RELAXED, __HIP_MEMORY_SCOPE_SYSTEM);
    asm volatile("" : "+v"(fl0));
    const unsigned call = CALL_NOW();
    if (lane == 0) __hip_atomic_store(cnt, call, __ATOMIC_RELAXED, __HIP_MEMORY_SCOPE_AGENT);
    if (xcc < 4 && slot == 3) return;
    unsigned char* const pw = tws;
    const auto rsrc_peer = mk_rsrc(pw, WS_USED);
    const auto rsrc_me = mk_rsrc(me, WS_USED);
    const auto rsrc_out = mk_rsrc(P.out, OUT_BYTES);
    const gu32* const sfl = (const gu32*)(me + OFF_SFLAG) + (peer * 16 + w) * 16;
    const gu32* const ofl = (const gu32*)(me + OFF_OFLAG) + peer * 512;
    const auto rsrc_sfl = mk_rsrc((const void*)sfl, 64);
    const auto rsrc_ofl = mk_rsrc((const void*)ofl, 2048);
    gu32* const prog = (gu32*)(me + OFF_PROG) + peer * (PROG_STRIDE > 2 ? 16 * PROG_STRIDE : 32) + w * PROG_STRIDE;
    const unsigned vlin = lane * 16;
    unsigned char* const ubuf = lds + wave * (2 * OSLOT);
    unsigned char* const fbuf = lds + LDS_FLAGS + wave * 1024;
    const unsigned vout = h * 4 * (GN * 4) + r * 16;
    const unsigned zaddr = lds_addr(lds + LDS_FLAGS + 4096 + wave * 256);
    *(LAS unsigned*)(size_t)(zaddr + lane * 4) = 0u;

    unsigned ops = 64;
    u64 hist = 0;
    unsigned pend_op = 0, pend_round = 0;
    unsigned fl = fl0;
    auto settle = [&](unsigned done_below) {
      if (pend_round && pend_op < done_below) {
        if (lane == 0) __hip_atomic_store(prog, call * 16u + pend_round, __ATOMIC_RELAXED, __HIP_MEMORY_SCOPE_AGENT);
        ++ops; pend_round = 0;
      }
    };
    auto throttle = [&](int win) {
      const unsigned n = (ops - 1u - (unsigned)(hist >> (8 * win - 8))) & 255u;
      wait_vm_dyn(n);
      settle(ops - n);
    };
    auto piece = [&](const __amdgpu_buffer_rsrc_t rs, unsigned char* dst, unsigned voff, unsigned soff) {
      __builtin_amdgcn_raw_ptr_buffer_load_lds(rs, (LAS void*)dst, 16, voff, soff, 0, AUX_SYS);
      hist = (hist << 8) | (ops & 255u); ++ops;
    };
    auto await_flag = [&](const gu32* f, int idx) {
      const u64 t0 = __builtin_amdgcn_s_memrealtime();
      while (!dead) {
        fl = __hip_atomic_load(f + (lane & 15), __ATOMIC_RELAXED, __HIP_MEMORY_SCOPE_SYSTEM);
        if ((unsigned)__builtin_amdgcn_readlane(fl, idx) >= call) break;
        if (give_up(t0, tmo, call)) dead = true;
      }
      __builtin_amdgcn_s_waitcnt(0x0F70);
      __builtin_amdgcn_fence(__ATOMIC_ACQUIRE, "wavefront");
      settle(ops);
    };
    unsigned fbase = 0;
    auto refresh_out = [&](unsigned from) {
      fbase = from & ~1u;
      __builtin_amdgcn_raw_ptr_buffer_load_lds(rsrc_ofl, (LAS void*)fbuf, 16, vlin + fbase * 8u, 0, 0, AUX_SYS); ++ops;
    };

    unsigned char* const ring = ubuf + OSLOT;
    auto put = [&](int c) {
      const int it = item_of_round(c >> 4);
      const unsigned so = ((it >= 8) ? OFF_WG + peer * MiB + (it - 8) * (256 * KiB) : x_item_off(peer, it, call)) + w * (16 * KiB) + (c & 15) * 1024u;
      const v4u t = *(const v4u*)(ring + (c & 15) * 1024 + vlin);
      STORE16(t, rsrc_me, vlin, so, AUX_DEV);
      if ((c & 15) == 15) { pend_op = ops; pend_round = (c >> 4) + 1; }
      ++ops;
    };
    {
      v4u hold = {0u, 0u, 0u, 0u};
      unsigned dprev = 0;
      for (int t = 0; t < 12; ++t) {
        const int it = item_of_round(t);
        const unsigned xo = x_item_off(peer, it & 7, call);
        const unsigned src = ((it >= 8) ? OFF_WS + rank * MiB + (it - 8) * (256 * KiB) : xo) + w * (16 * KiB);
        const unsigned dcur = ((it >= 8) ? OFF_WG + peer * MiB + (it - 8) * (256 * KiB) : xo) + w * (16 * KiB);
        const auto rs_early = mk_rsrc(me, t ? WS_USED : 0u);
        static_for<4>([&](auto qc) {
          constexpr int jq = decltype(qc)::v;
          if (jq < PUSHQ && t == 0) return;
          static_for<4>([&](auto uc) {
            constexpr int j = jq * 4 + decltype(uc)::v;
            constexpr int REFRESH_AT = 15 - IN_WIN - 1, PUBLISH_AT = 2 * IN_WIN + 1 < 16 ? 2 * IN_WIN + 1 : 15;
            static_assert(2 * IN_WIN + 1 <= 15 && REFRESH_AT >= 4 * PUSHQ, "window");
            wait_vm<2 * IN_WIN - 1 + ((j > REFRESH_AT && j - IN_WIN <= REFRESH_AT) ? 1 : 0)>();
            const v4u got = *(const v4u*)(ring + ((j - IN_WIN) & 15) * 1024 + vlin);
            if (j == 0 || (j & 3) == 0) {
              if (j == 0 || t == 0) {
                const int idx = t ? t : 12 + jq;
                if (t) fl = *(const unsigned*)(fbuf + (lane & 15) * 4);
                if ((unsigned)__builtin_amdgcn_readlane(fl, idx) < call) await_flag(sfl, idx);
                if (PUSHQ && t == 1 && (unsigned)__builtin_amdgcn_readlane(fl, 12) < call) await_flag(sfl, 12);
              }
            }
            if (j == PUBLISH_AT && t && lane == 0) __hip_atomic_store(prog, call * 16u + t, __ATOMIC_RELAXED, __HIP_MEMORY_SCOPE_AGENT);
            __builtin_amdgcn_raw_ptr_buffer_load_lds(rsrc_peer, (LAS void*)(ring + jq * 4096), 16, vlin, src + jq * 4096, (j & 3) * 1024, AUX_SYS);
            asm volatile("" ::: "memory");
            if (j <= IN_WIN) STORE16(hold, rs_early, vlin, dprev + (j - IN_WIN - 1 + 16) * 1024, AUX_DEV);
            else if (j - IN_WIN - 1 < 4 * PUSHQ) STORE16(hold, rs_early, vlin, dcur + (j - IN_WIN - 1) * 1024, AUX_DEV);
            else STORE16(hold, rsrc_me, vlin, dcur + (j - IN_WIN - 1) * 1024, AUX_DEV);
            hold = got;
            if (j == REFRESH_AT) {
              if (t < 11) __builtin_amdgcn_raw_ptr_buffer_load_lds(rsrc_sfl, (LAS void*)fbuf, 4, lane * 4, 0, 0, AUX_SYS);
              else refresh_out(0);
            }
          });
        });
        dprev = dcur;
      }
      ops = 128;
#pragma unroll
      for (int i = 7; i >= 0; --i) hist = (hist << 8) | (ops - 2u - 2u * i);
      STORE16(hold, rsrc_me, vlin, dprev + (15 - IN_WIN) * 1024, AUX_DEV);
      ++ops;
    }

    const gu32* const tick = (const gu32*)(me + OFF_TICK + peer * 128);
    int c = 192 - IN_WIN;
    unsigned last_op = 0;
    unsigned nx = w, par = 0;
    for (int cur = -1;;) {
      const int win = (cur < 0) ? IN_WIN : (nx >= TAIL_FROM ? TAIL_WIN : OUT_WIN);
      unsigned nb = 0;
      if (nx < 256u) {
        unsigned cn = 0, tag = 0;
        if (nx - fbase < 128u) {
          const v2u f = *(const v2u*)(fbuf + (nx - fbase) * 8u);
          cn = __builtin_amdgcn_readfirstlane(f[0]); tag = __builtin_amdgcn_readfirstlane(f[1]);
        }
        if (tag < call) {
          if (c < 192) { WAIT_VM(0); for (; c < 192; ++c) put(c); }
          await_flag(ofl + (nx & ~7u) * 2u, (nx & 7u) * 2u + 1u);
          cn = __builtin_amdgcn_readlane(fl, (nx & 7u) * 2u);
        }
        nb = dead ? 0u : ((512u + (cn > 4096u ? 4096u : cn) * 4u + 63u) & ~63u);
      }
      const int n = (int)((nb + 1023u) >> 10);
      const auto rs = mk_rsrc(pw + OFF_OS + rank * OS_OWNER + (nx & 255u) * OSLOT, nb);
      unsigned char* const nbuf = ubuf + par * OSLOT;
      int i = 0;
      unsigned draw_op = 0;
      auto draw = [&]() {
        if (nx < 256u && nx + 16u >= DYN_FROM) {
          asm volatile("s_mov_b64 exec, 1\n\tglobal_atomic_add v250, %0, %1, %2 sc0\n\ts_mov_b64 exec, -1"
                       :: "v"(0u), "v"(1u), "s"(tick) : "memory", "v250");
          draw_op = ops; ++ops;
        }
      };
      refresh_out(nx);
      draw_op = ops - 1u;
      if (cur >= 0) {
        for (; i < n && i < win; ++i) { throttle(win); piece(rs, nbuf + i * 1024, vlin + i * 1024, 0); }
        wait_vm_dyn(ops - 1u - last_op);
        const unsigned char* const cbuf = ubuf + (par ^ 1u) * OSLOT;
        v2u mk = *(const v2u*)(cbuf + lane * 8);
        const unsigned so = (cur >> 4) * 32u * (GN * 4) + (peer * NS + (cur & 15) * 128) * 4u;
        if (dead) { mk = v2u{~0u, ~0u}; *(LAS unsigned*)(size_t)(zaddr + lane * 4) = 0x7fc00000u; }
        const unsigned np = 64u - __popc(mk[0]) - __popc(mk[1]);
        unsigned qa = lds_addr(cbuf) + 512u + (wave_scan(np) - np) * 4u;
        static_for<4>([&](auto qc) {
          constexpr int q = decltype(qc)::v;
          v4u o[4];
          static_for<4>([&](auto ec) {
            constexpr int e = decltype(ec)::v, idx = (q * 4 + e) * 4;
            unsigned a[4];
            run_addr4<31 - (idx & 31)>(mk[idx >> 5], zaddr, qa, a);
#pragma unroll
            for (int f = 0; f < 4; ++f) o[e][f] = *(const LAS unsigned*)(size_t)(a[f] + (idx + f) * 4);
          });
          if (q == DRAW_AT) draw();
#pragma unroll
          for (int e = 0; e < 4; ++e) STORE16(o[e], rsrc_out, vout, so + (q * 8 + e) * (GN * 4), OUT_AUX);
          ops += 4;
          if (i < n) { throttle(win); piece(rs, nbuf + i * 1024, vlin + i * 1024, 0); ++i; }
        });
      } else {
        draw();
      }
      if (nx >= 256u) break;
      for (; i < n; ++i) {
        throttle(win); piece(rs, nbuf + i * 1024, vlin + i * 1024, 0);
        if (c < 192) put(c++);
      }
      last_op = (unsigned)(ops - 1u - ((ops - 1u - (unsigned)hist) & 255u));
      if (cur < 0) {
        if (c < 192) { WAIT_VM(0); for (; c < 192; ++c) put(c); }
        if (pend_round) { wait_vm_dyn(ops - 1u - pend_op); settle(pend_op + 1u); }
        WAIT_LGKM0();
      }
      wait_vm_dyn(ops - 1u - draw_op);
      cur = (int)nx;
      if (nx + 16u >= DYN_FROM) {
        asm volatile("v_readfirstlane_b32 %0, v250" : "=s"(nx) :: "memory");
        nx = nx - (call - 1u) * (272u - DYN_FROM) + DYN_FROM;
      } else nx += 16u;
      par ^= 1u;
    }
    WAIT_VM(0);
    STAMP(5);
    return;
  }

  const int g = slot - 4;
  const auto rsrc_me = mk_rsrc(me, WS_USED);
  {
    if (jb >= 64 && jb < 576) {
      const int s = (jb - 64) >> 7, q = ((jb - 64) >> 4) & 7, j = jb & 15;
      STAGE_DELAY(s == 0 ? DELAY1 : DELAY2);
      const auto rw = mk_rsrc(wp, KS * GN);
      const unsigned voff = h * 16 * GN + r * 4;
      const unsigned so = s * 128 * GN + q * NS + j * 128;
      unsigned d[64];
#pragma unroll
      for (int e = 0; e < 64; ++e)
        d[e] = __builtin_amdgcn_raw_buffer_load_b32(rw, voff, so + ((e >> 4) * 32 + (e & 15)) * GN, 0);
      const unsigned dst = ((q == rank) ? OFF_WG + rank * MiB : OFF_WS + q * MiB) + s * (256 * KiB) + j * (16 * KiB);
#pragma unroll
      for (int ki = 0; ki < 4; ++ki) {
        v4u o[4];
#pragma unroll
        for (int gq = 0; gq < 4; ++gq) {
          unsigned a = d[ki * 16 + gq * 4], b = d[ki * 16 + gq * 4 + 1], c = d[ki * 16 + gq * 4 + 2], e = d[ki * 16 + gq * 4 + 3];
          unsigned t0 = __builtin_amdgcn_perm(b, a, 0x05010400u), t1 = __builtin_amdgcn_perm(b, a, 0x07030602u);
          unsigned t2 = __builtin_amdgcn_perm(e, c, 0x05010400u), t3 = __builtin_amdgcn_perm(e, c, 0x07030602u);
          o[0][gq] = __builtin_amdgcn_perm(t2, t0, 0x05040100u);
          o[1][gq] = __builtin_amdgcn_perm(t2, t0, 0x07060302u);
          o[2][gq] = __builtin_amdgcn_perm(t3, t1, 0x05040100u);
          o[3][gq] = __builtin_amdgcn_perm(t3, t1, 0x07060302u);
        }
#pragma unroll
        for (int f = 0; f < 4; ++f)
          STORE16(o[f], rsrc_me, lane * 16, dst + f * 4096 + ki * 1024, AUX_SYS);
      }
      WAIT_VM(0);
      if (lane == 0)
        __hip_atomic_store((gu32*)(tws + OFF_SFLAG) + (rank * 16 + j) * 16 + round_of_w(s), CALL_NOW(), __ATOMIC_RELAXED, __HIP_MEMORY_SCOPE_SYSTEM);
    } else if (jb < 688 + (PUSHQ ? 112 : 0)) {
      const bool quarter = jb < 64, push = jb >= 688;
      const int un = (quarter || push) ? (jb & 15) : jb - 576 + 16;
      const int rb0 = quarter ? (jb >> 4) : 0, nrb = quarter ? 1 : (push ? PUSHQ : 4);
      const int dest = push ? tgt : rank;
      if (!quarter && !push) STAGE_DELAY(DELAY2);
      const auto rx = mk_rsrc(xp, GM * KS);
      const auto rd = mk_rsrc(push ? tws : me, WS_USED);
      const unsigned voff = r * KS + h * 16;
      const unsigned so = ((un & 7) * MS + (un >> 5) * 128) * KS + ((un >> 3) & 3) * 128;
      for (int rb = rb0; rb < rb0 + nrb; rb += 2) {
        v4u d[8];
#pragma unroll
        for (int e = 0; e < 8; ++e) if (e < 4 || nrb > 1) d[e] = __builtin_amdgcn_raw_buffer_load_b128(rx, voff + (e & 3) * 32, so + (rb + (e >> 2)) * 32 * KS, 0);
        const unsigned dst = x_item_off(rank, un >> 4, CALL_NOW()) + (un & 15) * (16 * KiB) + rb * 4096;
#pragma unroll
        for (int e = 0; e < 8; ++e) if (e < 4 || nrb > 1) STORE16(d[e], rd, lane * 16 + (e & 3) * 1024, dst + (e >> 2) * 4096, AUX_SYS);
      }
      WAIT_VM(0);
      const bool tell = push ? (lane == dest) : ((quarter && rb0 < PUSHQ) ? (lane == rank) : (lane < NR));
      if (tell)
        __hip_atomic_store((gu32*)(dws + OFF_SFLAG) + (rank * 16 + (un & 15)) * 16 + ((quarter || push) ? 12 + rb0 : round_of_x(un >> 4)), CALL_NOW(), __ATOMIC_RELAXED, __HIP_MEMORY_SCOPE_SYSTEM);
    }
  }
  const unsigned call = CALL_NOW();
  if (lane == 0) __hip_atomic_store(cnt, call, __ATOMIC_RELAXED, __HIP_MEMORY_SCOPE_AGENT);
  STAMP(1);

  const float scale = P.sx[0] * P.sw[0];
  const int ntiles = (g < 4) ? 2 : 1;
  const int wm = wave >> 1, wn = wave & 1;
  for (int it = 0; it < ntiles; ++it) {
    const int t14 = (g >= 14) ? g - 14 : g;
    const int ch = (xcd & 1) + (it ? (g & 2) : (g >= 14 ? 2 : 0));
    const int own = it ? rank : ((rank + 1 + (t14 >> 1)) & 7);
    const int jj = (xcd >> 1) * 2 + (g & 1);
    const int wh = lane & 3;
    const int wsel = (wh < 2) ? 2 * jj + wh : own + (wh - 2) * 8;
    const gu32* paddr = (const gu32*)(me + OFF_PROG) + ((rank + 1 + ((lane >> 2) & 7)) & 7) * (PROG_STRIDE > 2 ? 16 * PROG_STRIDE : 32) + wsel * PROG_STRIDE;
    v2u* const order = (v2u*)(lds + LDS_ORDER);
    unsigned nxt = (lane < 28) ? 0u : 4u;
    int nq = 4;
    auto refill = [&](int want) {
      const u64 t0 = __builtin_amdgcn_s_memrealtime();
      for (;;) {
        const unsigned pval = dead ? 0xffffffffu : __hip_atomic_load(paddr, __ATOMIC_RELAXED, __HIP_MEMORY_SCOPE_AGENT);
        for (int pass = 0; pass < 4; ++pass) {
          const unsigned thr = call * 16u + 1u + round_of_item((wh < 2) ? 8 + (nxt & 3u) : ch * 2 + ((nxt >> 1) & 1u));
          const bool ok = nxt < 4u && (pval >= thr || (wh >= 2 && (unsigned)(wh - 2) != (nxt & 1u)));
          const unsigned m = (unsigned)__ballot(ok);
          const unsigned rdy = m & (m >> 1) & (m >> 2) & (m >> 3) & 0x01111111u;
          if (!rdy) break;
          const bool mine = lane < 32 && ((rdy >> (lane & 28)) & 1u);
          if (mine && wh == 0) {
            const unsigned p = (rank + 1 + (lane >> 2)) & 7;
            order[nq + __builtin_amdgcn_mbcnt_lo(rdy, 0u)] = v2u{x_item_off(p, ch * 2 + (nxt >> 1), call) + (nxt & 1u) * (128 * KiB), p * MiB + nxt * (256 * KiB)};
          }
          nxt += mine ? 1u : 0u;
          nq += __popc(rdy);
        }
        if (nq > want) break;
        if (give_up(t0, tmo, call)) dead = true;
        __builtin_amdgcn_s_sleep(GEMM_POLL_SLEEP);
      }
      __builtin_amdgcn_s_waitcnt(0x0F70);
      __builtin_amdgcn_fence(__ATOMIC_ACQUIRE, "wavefront");
    };

    const unsigned sa0 = own * (16 * KiB) + wave * 4096;
    const unsigned sb0 = OFF_WG + jj * (32 * KiB) + wave * 8192;
    v16f acc[2][4];
#pragma unroll
    for (int mi = 0; mi < 2; ++mi)
#pragma unroll
      for (int f = 0; f < 4; ++f)
#pragma unroll
        for (int e = 0; e < 16; ++e) acc[mi][f][e] = 0.f;

    if (wave == 0) {
      if (lane < 8) order[32 + lane] = v2u{0u, 0u};
      const int wq = lane >> 4;
      const gu32* a = (const gu32*)(me + OFF_SFLAG) + (rank * 16 + ((wq < 2) ? 2 * jj + wq : own + (wq - 2) * 8)) * 16 + (lane & 15);
      const u64 wbits = (1u << round_of_w(0)) | (1u << round_of_w(1)) | (1u << round_of_w(2)) | (1u << round_of_w(3));
      const u64 xbits = (ch ? (1u << round_of_x(ch * 2)) : 0xf000u) | (1u << round_of_x(ch * 2 + 1));
      unsigned dummy;
      wait_words(a, call, wbits | (wbits << 16) | (xbits << 32) | (xbits << 48), tmo, call, dead, dummy);
    }
    __builtin_amdgcn_s_barrier();
#define MF(A, B, mi, f) acc[mi][f] = __builtin_amdgcn_mfma_scale_f32_32x32x64_f8f6f4(A[mi], B[f], acc[mi][f], 0, 0, 0, 0, 0, 0)
#define DM(l, so, o) __builtin_amdgcn_raw_ptr_buffer_load_lds(rs, (LAS void*)(l), 16, lane * 16, (so), (o), GEMM_AUX)
#define FR(dst, ptr) dst = __builtin_shufflevector(*(const v4i*)(ptr), *(const v4i*)((ptr) + 1024), 0, 1, 2, 3, 4, 5, 6, 7)
#define SB() __builtin_amdgcn_sched_barrier(0)
#pragma unroll
    for (int s = 0; s < 3; ++s) {
      const auto rs = rsrc_me;
      const unsigned sa = sa0 + x_item_off(rank, ch * 2 + (s >> 1), call) + (s & 1) * (128 * KiB), sb = sb0 + rank * MiB + s * (256 * KiB);
      unsigned char* const la = lds + s * STAGE_BYTES + wave * 4096;
      unsigned char* const lb = lds + s * STAGE_BYTES + STAGE_A + wave * 8192;
      DM(la, sa, 0); DM(la, sa, 1024); DM(lb, sb, 0); DM(lb, sb, 1024); DM(lb + 4096, sb + 4096, 0); DM(lb + 4096, sb + 4096, 1024); SB();
      DM(la, sa, 2048); DM(la, sa, 3072); DM(lb, sb, 2048); DM(lb, sb, 3072); DM(lb + 4096, sb + 4096, 2048); DM(lb + 4096, sb + 4096, 3072); SB();
    }
    WAIT_VM(30);
    __builtin_amdgcn_s_barrier();
    v8i A0[2], B0[4], A1[2], B1[4];
    STAMP(6 + it);
    const unsigned char* const ra0 = lds + lane * 16 + wm * 8192;
    const unsigned char* const rb0 = lds + lane * 16 + STAGE_A + wn * 16384;
    FR(A0[0], ra0); FR(A0[1], ra0 + 4096); FR(B0[0], rb0); FR(B0[1], rb0 + 4096); FR(B0[2], rb0 + 8192); FR(B0[3], rb0 + 12288);
    int buf = 0;
    v2u ent = {x_item_off(rank, ch * 2 + 1, call) + 128 * KiB, rank * MiB + 3 * (256 * KiB)};
    for (int k = 0; k < 32; ++k) {
      const auto rs = mk_rsrc(me, k < 29 ? WS_USED : 0u);
      const unsigned sa = sa0 + __builtin_amdgcn_readfirstlane(ent[0]);
      const unsigned sb = sb0 + __builtin_amdgcn_readfirstlane(ent[1]);
      unsigned char* const la = lds + buf + wave * 4096;
      unsigned char* const lb = lds + buf + STAGE_A + wave * 8192;
      const unsigned char* ra = ra0 + buf + 2048;
      const unsigned char* rb = rb0 + buf + 2048;
      WAIT_LGKM0();
      WAIT_VM(24);
      __builtin_amdgcn_s_barrier();
      SB();
      MF(A0, B0, 0, 0); SB(); FR(A1[0], ra); DM(la, sa, 0); SB();
      MF(A0, B0, 0, 1); SB(); FR(B1[0], rb); DM(la, sa, 1024); SB();
      MF(A0, B0, 0, 2); SB(); FR(B1[1], rb + 4096); DM(lb, sb, 0); SB();
      MF(A0, B0, 0, 3); SB(); FR(B1[2], rb + 8192); DM(lb, sb, 1024); SB();
      MF(A0, B0, 1, 0); SB(); FR(B1[3], rb + 12288); DM(lb + 4096, sb + 4096, 0); SB();
      MF(A0, B0, 1, 1); SB(); FR(A1[1], ra + 4096); DM(lb + 4096, sb + 4096, 1024); SB();
      MF(A0, B0, 1, 2); SB();
      buf = (buf == 2 * STAGE_BYTES) ? 0 : buf + STAGE_BYTES;
      ra = ra0 + buf; rb = rb0 + buf;
      MF(A0, B0, 1, 3); SB();
      if (wave == 0 && k < 28 && nq <= k + 4) refill(k + 4);
      WAIT_LGKM0();
      WAIT_VM(24);
      __builtin_amdgcn_s_barrier();
      SB();
      MF(A1, B1, 0, 0); SB(); ent = order[k + 4]; FR(A0[0], ra); DM(la, sa, 2048); SB();
      MF(A1, B1, 0, 1); SB(); FR(B0[0], rb); DM(la, sa, 3072); SB();
      MF(A1, B1, 0, 2); SB(); FR(B0[1], rb + 4096); DM(lb, sb, 2048); SB();
      MF(A1, B1, 0, 3); SB(); FR(B0[2], rb + 8192); DM(lb, sb, 3072); SB();
      MF(A1, B1, 1, 0); SB(); FR(B0[3], rb + 12288); DM(lb + 4096, sb + 4096, 2048); SB();
      MF(A1, B1, 1, 1); SB(); FR(A0[1], ra + 4096); DM(lb + 4096, sb + 4096, 3072); SB();
      MF(A1, B1, 1, 2); SB();
      MF(A1, B1, 1, 3); SB();
    }
    WAIT_VM(0);
    WAIT_LGKM0();
    __builtin_amdgcn_s_barrier();
    STAMP(2 + 2 * it);
    const bool poison = __builtin_amdgcn_readfirstlane(__hip_atomic_load(tmo, __ATOMIC_RELAXED, __HIP_MEMORY_SCOPE_AGENT)) == call;
    const float sc = poison ? __uint_as_float(0x7fc00000u) : scale;
    if (own == rank) {
      const auto rdst = mk_rsrc(P.out, OUT_BYTES);
      const unsigned vo = h * 4 * (GN * 4) + r * 16;
      const unsigned sbase = (ch * 128 + wm * 64) * (GN * 4) + (rank * NS + jj * 256 + wn * 128) * 4;
#pragma unroll
      for (int mi = 0; mi < 2; ++mi)
#pragma unroll
        for (int e = 0; e < 16; ++e) {
          v4u o;
#pragma unroll
          for (int f = 0; f < 4; ++f) { const float y = acc[mi][f][e] * sc; const unsigned b = __float_as_uint(y); o[f] = b & ~(unsigned)((int)b >> 31); }
          STORE16(o, rdst, vo, sbase + (mi * 32 + (e >> 2) * 8 + (e & 3)) * (GN * 4), AUX_DEV);
        }
    } else {
      unsigned cnt0 = 0;
#pragma unroll
      for (int mi = 0; mi < 2; ++mi) {
        unsigned char* const cb = lds + (wave * 2 + mi) * OSLOT;
        float y[64];
        v2u mk = {0u, 0u};
#pragma unroll
        for (int i = 0; i < 64; ++i) {
          y[i] = acc[mi][i & 3][i >> 2] * sc;
          mk[i >> 5] = __builtin_amdgcn_alignbit(mk[i >> 5], __float_as_uint(y[i]), 31);
        }
        *(v2u*)(cb + lane * 8) = mk;
        const unsigned np = 64u - __popc(mk[0]) - __popc(mk[1]);
        const unsigned inc = wave_scan(np);
        const unsigned base = __builtin_amdgcn_readlane(inc, 63);
        unsigned qa = lds_addr(cb) + 512u + (inc - np) * 4u;
        const unsigned junk = lds_addr(lds + LDS_FLAGS + wave * 512) + lane * 4;
        static_for<16>([&](auto ec) {
          constexpr int idx = decltype(ec)::v * 4;
          unsigned a[4];
          run_addr4<31 - (idx & 31)>(mk[idx >> 5], junk, qa, a);
#pragma unroll
          for (int f = 0; f < 4; ++f) *(LAS float*)(size_t)(a[f] + (idx + f) * 4) = y[idx + f];
        });
        gu64* const oflag = (gu64*)(P.ws[own] + OFF_OFLAG) + rank * 256 + (ch * 4 + wm * 2) * 16 + jj * 2 + wn;
        if (mi == 1) {
          WAIT_VM(0);
          if (lane == 0) __hip_atomic_store(oflag, ((u64)call << 32) | cnt0, __ATOMIC_RELAXED, __HIP_MEMORY_SCOPE_SYSTEM);
        }
        const unsigned dst = OFF_OS + own * OS_OWNER + ((ch * 4 + wm * 2 + mi) * 16 + jj * 2 + wn) * OSLOT;
        const int n = (int)((512u + base * 4u + 1023u) >> 10);
        for (int i = 0; i < n; i += 3) {
          const auto rslot = mk_rsrc(me + dst, OSLOT);
          v4u t[3];
#pragma unroll
          for (int u = 0; u < 3; ++u) t[u] = *(const v4u*)(cb + (i + u) * 1024 + lane * 16);
#pragma unroll
          for (int u = 0; u < 3; ++u) STORE16(t[u], rslot, lane * 16 + (i + u) * 1024, 0, AUX_SYS);
        }
        if (mi == 0) cnt0 = base;
        else {
          WAIT_VM(0);
          if (lane == 0) __hip_atomic_store(oflag + 16, ((u64)call << 32) | base, __ATOMIC_RELAXED, __HIP_MEMORY_SCOPE_SYSTEM);
        }
      }
      WAIT_LGKM0();
    }
    STAMP(3 + 2 * it);
  }
}

}

extern "C" void kernel_launch(int rank, int nranks, ncclComm_t comm,
                              hipStream_t stream,
                              void* const* d_in, const int* in_sizes, int n_in,
                              void* d_out, int out_size,
                              const int* in_dtypes, int out_dtype,
                              void* const* d_out_all,
                              void* const* d_ws_all, size_t ws_bytes) {
  if (nranks != NR || n_in < 4 || ws_bytes < WS_USED || out_size != MS * GN ||
      in_sizes[0] != GM * KS || in_sizes[1] != KS * GN) {
    fprintf(stderr, "unexpected problem shape\n");
    return;
  }
  Params P;
  P.x = (const unsigned char*)d_in[0];
  P.w = (const unsigned char*)d_in[1];
  P.sx = (const float*)d_in[2];
  P.sw = (const float*)d_in[3];
  P.out = (float*)d_out;
  for (int p = 0; p < NR; ++p) P.ws[p] = (unsigned char*)d_ws_all[p];
  P.rank = rank; P.pad = 0;
  P.me = P.ws[rank];
  hipLaunchKernelGGL(fused_kernel, dim3(256), dim3(256), 0, stream, P);
  HIP_CHECK(hipGetLastError());
}
```

```cpp
#include <hip/hip_runtime.h>
#include <rccl/rccl.h>
#include <cstdio>
#include <cstdint>

#define HIP_CHECK(expr) do { hipError_t _e = (expr); if (_e != hipSuccess) { \
    fprintf(stderr, "HIP error %s at %s:%d\n", hipGetErrorString(_e), __FILE__, __LINE__); \
    return; } } while (0)

namespace {

typedef int v8i __attribute__((ext_vector_type(8)));
typedef int v4i __attribute__((ext_vector_type(4)));
typedef unsigned v4u __attribute__((ext_vector_type(4)));
typedef unsigned v2u __attribute__((ext_vector_type(2)));
typedef float v16f __attribute__((ext_vector_type(16)));
typedef unsigned long long u64;
#define LAS __attribute__((address_space(3)))
#define GAS __attribute__((address_space(1)))
typedef GAS unsigned gu32;
typedef GAS u64 gu64;

constexpr int NR = 8;
constexpr int GM = 4096, GK = 4096, GN = 16384;
constexpr int KS = GK / NR;
constexpr int MS = GM / NR;
constexpr int NS = GN / NR;

constexpr unsigned KiB = 1024u, MiB = 1024u * 1024u;
constexpr unsigned OFF_CNT   = 0;
constexpr unsigned OFF_TMO   = 64 * KiB;
constexpr unsigned OFF_PROG  = 112 * KiB;
constexpr unsigned OFF_TS    = 76 * KiB;
constexpr unsigned OFF_TICK  = 72 * KiB;
constexpr unsigned OFF_SFLAG = 80 * KiB;
constexpr unsigned OFF_OFLAG = 96 * KiB;
constexpr unsigned OFF_DBG   = 128 * KiB;
constexpr unsigned OFF_XG    = 1 * MiB;
constexpr unsigned OFF_WG    = OFF_XG + 16 * MiB;
constexpr unsigned OFF_WS    = OFF_WG + 8 * MiB;
constexpr unsigned OFF_OS    = OFF_WS + 8 * MiB;
constexpr unsigned OSLOT     = 17 * KiB;
constexpr unsigned OS_OWNER  = 256 * OSLOT;
constexpr unsigned OFF_XALT  = OFF_OS + 8 * OS_OWNER;
constexpr unsigned WS_USED   = OFF_XALT + 2 * MiB;
constexpr unsigned OUT_BYTES = MS * GN * 4;

constexpr unsigned long long DEADLINE = 50000000ull;
#define TOPO_MASK 0x4Du
constexpr int AUX_SYS = 17;
constexpr int AUX_DEV = 16;
#define GEMM_AUX 16

constexpr int STAGE_A = 16 * 1024, STAGE_B = 32 * 1024, STAGE_BYTES = STAGE_A + STAGE_B;
constexpr int LDS_ORDER = 3 * STAGE_BYTES;
constexpr int LDS_BYTES = LDS_ORDER + 512;
constexpr int LDS_FLAGS = 8 * OSLOT;
static_assert(LDS_FLAGS + 5120 <= LDS_BYTES, "lds");

struct Params {
  const unsigned char* x;
  const unsigned char* w;
  unsigned char* me;
  int rank, pad;
  const float* sx;
  const float* sw;
  float* out;
  unsigned char* ws[NR];
};

#define WAIT_VM(n) asm volatile("s_waitcnt vmcnt(" #n ")" ::: "memory")
#define STORE16(val, rs, voff, soff, aux) do { const v4u sv_ = (val); \
    __builtin_amdgcn_raw_buffer_store_b128(sv_, rs, voff, soff, aux); \
    asm volatile("s_nop 1" :: "v"(sv_) : "memory"); } while (0)
#define DELAY1 24
#define DELAY2 72
#define STAGE_DELAY(n) do { if ((n) == DELAY1) __builtin_amdgcn_s_sleep(DELAY1); else __builtin_amdgcn_s_sleep(DELAY2); } while (0)
template <int N> __device__ __forceinline__ void wait_vm() { asm volatile("s_waitcnt vmcnt(%0)" :: "i"(N) : "memory"); }
#define OUT_AUX 16
#define DRAW_AT 0
#define DYN_FROM 208u
#define POLL_SLEEP 4
#define PUSHQ 1
#define TS_ROUND 4
#define RELAY 1
#define RELAY_GAP 800
#define GEMM_POLL_SLEEP 0
#define PUSH_WIN 2
#define IN_WIN 3
#define OUT_WIN 3
#define TAIL_WIN 3
#define TAIL_FROM 224
__device__ __forceinline__ void wait_vm_dyn(unsigned n) {
  n = n > 31u ? 31u : n;
#define WV1(k) "s_waitcnt vmcnt(" #k ")\n\ts_branch 1f\n\t"
  asm volatile("s_getpc_b64 vcc\n\t"
               "s_lshl3_add_u32 %0, %0, 16\n\t"
               "s_add_u32 vcc_lo, vcc_lo, %0\n\t"
               "s_addc_u32 vcc_hi, vcc_hi, 0\n\t"
               "s_setpc_b64 vcc\n\t"
               WV1(0) WV1(1) WV1(2) WV1(3) WV1(4) WV1(5) WV1(6) WV1(7) WV1(8) WV1(9) WV1(10) WV1(11) WV1(12) WV1(13) WV1(14) WV1(15)
               WV1(16) WV1(17) WV1(18) WV1(19) WV1(20) WV1(21) WV1(22) WV1(23) WV1(24) WV1(25) WV1(26) WV1(27) WV1(28) WV1(29) WV1(30) WV1(31)
               "1:"
               : "+s"(n) :: "memory", "scc", "vcc");
#undef WV1
}
#define WAIT_LGKM0() asm volatile("s_waitcnt lgkmcnt(0)" ::: "memory")

__device__ __forceinline__ __amdgpu_buffer_rsrc_t mk_rsrc(const void* p, unsigned bytes) {
  return __builtin_amdgcn_make_buffer_rsrc((void*)p, 0, bytes, 0x00020000);
}
__device__ __forceinline__ unsigned wave_scan(unsigned x) {
  x += __builtin_amdgcn_update_dpp(0u, x, 0x111, 0xf, 0xf, false);
  x += __builtin_amdgcn_update_dpp(0u, x, 0x112, 0xf, 0xf, false);
  x += __builtin_amdgcn_update_dpp(0u, x, 0x114, 0xf, 0xf, false);
  x += __builtin_amdgcn_update_dpp(0u, x, 0x118, 0xf, 0xf, false);
  x += __builtin_amdgcn_update_dpp(0u, x, 0x142, 0xa, 0xf, false);
  x += __builtin_amdgcn_update_dpp(0u, x, 0x143, 0xc, 0xf, false);
  return x;
}
template <int P>
__device__ __forceinline__ void run_addr4(unsigned m, unsigned other, unsigned& q, unsigned (&a)[4]) {
  unsigned t;
  asm("v_bfe_i32 %5, %6, %8, 1\n\tv_bfi_b32 %0, %5, %7, %4\n\tv_lshl_add_u32 %4, %5, 2, %4\n\t"
      "v_bfe_i32 %5, %6, %9, 1\n\tv_bfi_b32 %1, %5, %7, %4\n\tv_lshl_add_u32 %4, %5, 2, %4\n\t"
      "v_bfe_i32 %5, %6, %10, 1\n\tv_bfi_b32 %2, %5, %7, %4\n\tv_lshl_add_u32 %4, %5, 2, %4\n\t"
      "v_bfe_i32 %5, %6, %11, 1\n\tv_bfi_b32 %3, %5, %7, %4\n\tv_lshl_add_u32 %4, %5, 2, %4"
      : "=&v"(a[0]), "=&v"(a[1]), "=&v"(a[2]), "=&v"(a[3]), "+v"(q), "=&v"(t)
      : "v"(m), "v"(other), "i"(P), "i"(P - 1), "i"(P - 2), "i"(P - 3));
}
template <int E> struct IC { static constexpr int v = E; };
template <int N, int I = 0, class F>
__device__ __forceinline__ void static_for(F&& f) { if constexpr (I < N) { f(IC<I>{}); static_for<N, I + 1>(f); } }
__device__ __forceinline__ unsigned lds_addr(const void* p) { return (unsigned)(size_t)(const LAS unsigned char*)p; }

#define STAMP(slot) do {} while (0)

__device__ __forceinline__ bool give_up(u64 t0, gu32* tmo, unsigned call) {
  unsigned t = __hip_atomic_load(tmo, __ATOMIC_RELAXED, __HIP_MEMORY_SCOPE_AGENT);
  if (__builtin_amdgcn_readfirstlane(t) == call || __builtin_amdgcn_s_memrealtime() - t0 > DEADLINE) {
    __hip_atomic_store(tmo, call, __ATOMIC_RELAXED, __HIP_MEMORY_SCOPE_AGENT);
    return true;
  }
  __builtin_amdgcn_s_sleep(POLL_SLEEP);
  return false;
}

__device__ __forceinline__ u64 wait_words(const gu32* addr, unsigned thr, u64 need,
                                          gu32* tmo, unsigned call, bool& dead, unsigned& val) {
  u64 t0 = __builtin_amdgcn_s_memrealtime();
  u64 ok;
  for (;;) {
    val = __hip_atomic_load(addr, __ATOMIC_RELAXED, __HIP_MEMORY_SCOPE_SYSTEM);
    ok = __ballot(val >= thr);
    if ((ok & need) == need) break;
    if (give_up(t0, tmo, call)) { dead = true; ok = ~0ull; val = 0xffffffffu; break; }
  }
  __builtin_amdgcn_fence(__ATOMIC_ACQUIRE, "wavefront");
  return ok;
}

#define ROUND_ITEMS 0x75634ba12980ull
#define ITEM_ROUNDS 0x6521b9a78340ull
__device__ __forceinline__ int item_of_round(int t) { return (int)((ROUND_ITEMS >> (4 * t)) & 15u); }
__device__ __forceinline__ int round_of_item(int it) { return (int)((ITEM_ROUNDS >> (4 * it)) & 15u); }
__device__ __forceinline__ int round_of_w(int s) { return round_of_item(8 + s); }
__device__ __forceinline__ int round_of_x(int xr) { return round_of_item(xr); }

__device__ __forceinline__ int x3_class(int q, int w) { return __popc(q & (w % 7 + 1)) & 1; }

__device__ __forceinline__ unsigned x_item_off(int p, int it, unsigned call) {
  return (it == 0 && (call & 1u)) ? OFF_XALT + p * (256 * KiB) : OFF_XG + p * (2 * MiB) + it * (256 * KiB);
}

__global__ void __launch_bounds__(256, 1) fused_kernel(Params P) {
  __shared__ __attribute__((aligned(1024))) unsigned char lds[LDS_BYTES];
  const int lane = threadIdx.x & 63;
  const int wave = __builtin_amdgcn_readfirstlane(threadIdx.x >> 6);
  const int rank = P.rank & 7;
  unsigned char* const me = P.me;
  const unsigned char* xp = P.x;
  const unsigned char* wp = P.w;
  u64 w0 = (u64)P.ws[0], w1 = (u64)P.ws[1], w2 = (u64)P.ws[2], w3 = (u64)P.ws[3];
  u64 w4 = (u64)P.ws[4], w5 = (u64)P.ws[5], w6 = (u64)P.ws[6], w7 = (u64)P.ws[7];
  asm volatile("" : "+s"(xp), "+s"(wp), "+s"(w0), "+s"(w1), "+s"(w2), "+s"(w3), "+s"(w4), "+s"(w5), "+s"(w6), "+s"(w7));
  auto ws_of = [=](int p) {
    const u64 a = (p & 1) ? w1 : w0, b = (p & 1) ? w3 : w2, c = (p & 1) ? w5 : w4, d = (p & 1) ? w7 : w6;
    const u64 e = (p & 2) ? b : a, f = (p & 2) ? d : c;
    return (unsigned char*)((p & 4) ? f : e);
  };
  const int xcd = blockIdx.x & 7, slot = blockIdx.x >> 3;
  gu32* const tmo = (gu32*)(me + OFF_TMO);
  const int xcc = __builtin_amdgcn_s_getreg((20 << 0) | (0 << 6) | (3 << 11)) & 7;
  const unsigned quad = ((TOPO_MASK >> rank) & 1) ? TOPO_MASK : (~TOPO_MASK & 0xffu);
  unsigned cand = (xcc < 4) ? (quad & ~(1u << rank)) : (~quad & 0xffu);
  for (int i = 0; i < (slot & 3); ++i) cand &= cand - 1;
  const int peer = __builtin_ctz(cand | 0x80u) & 7;
  const int jb = wave * 224 + (slot - 4) * 8 + xcd;
  const int tgt = (slot < 4) ? peer : (jb < 576) ? (((jb - 64) >> 4) & 7) : (jb >= 688) ? ((rank + 1 + ((jb - 688) >> 4)) & 7) : rank;
  unsigned char* const tws = ws_of(tgt);
  unsigned char* const dws = ws_of(lane & 7);
  STAMP(0);

  gu32* const cnt = (gu32*)(me + OFF_CNT + (blockIdx.x * 4 + wave) * 64);
  const unsigned prev_v = __hip_atomic_load(cnt, __ATOMIC_RELAXED, __HIP_MEMORY_SCOPE_AGENT);
#define CALL_NOW() ((unsigned)__builtin_amdgcn_readfirstlane(prev_v) + 1u)
  bool dead = false;
  const int h = lane >> 5, r = lane & 31;

  if (slot < 4) {
    const int w = (xcc & 3) * 4 + wave;
    unsigned fl0 = __hip_atomic_load((const gu32*)(me + OFF_SFLAG) + (peer * 16 + w) * 16 + (lane & 15), __ATOMIC_RELAXED, __HIP_MEMORY_SCOPE_SYSTEM);
    asm volatile("" : "+v"(fl0));
    const unsigned call = CALL_NOW();
    if (lane == 0) __hip_atomic_store(cnt, call, __ATOMIC_RELAXED, __HIP_MEMORY_SCOPE_AGENT);
    if (xcc < 4 && slot == 3) {
      if (xcc != 0 || wave != 0) return;
      const gu64* const ts = (const gu64*)(me + OFF_TS);
      const u64 t0 = __builtin_amdgcn_s_memrealtime();
      u64 ta, tb;
      for (;;) {
        __builtin_amdgcn_s_sleep(40);
        ta = __hip_atomic_load(ts + lane, __ATOMIC_RELAXED, __HIP_MEMORY_SCOPE_AGENT);
        tb = __hip_atomic_load(ts + 64 + lane, __ATOMIC_RELAXED, __HIP_MEMORY_SCOPE_AGENT);
        const bool ok = ((unsigned)(ta >> 32) >= call || (lane >> 4) == rank) && ((unsigned)(tb >> 32) >= call || (lane >> 4) + 4 == rank);
        if (__ballot(ok) == ~0ull) break;
        if (give_up(t0, tmo, call)) { dead = true; break; }
      }
      auto row_sum = [&](u64 t) {
        unsigned x = min((unsigned)t - (unsigned)t0 + 4096u, 1u << 20);
        x += __builtin_amdgcn_update_dpp(0u, x, 0x111, 0xf, 0xf, false);
        x += __builtin_amdgcn_update_dpp(0u, x, 0x112, 0xf, 0xf, false);
        x += __builtin_amdgcn_update_dpp(0u, x, 0x114, 0xf, 0xf, false);
        x += __builtin_amdgcn_update_dpp(0u, x, 0x118, 0xf, 0xf, false);
        return x;
      };
      const unsigned sa = row_sum(ta), sb = row_sum(tb);
      const unsigned fa = __shfl(sa, (lane & 3) * 16 + 15), fb = __shfl(sb, (lane & 3) * 16 + 15);
      unsigned F = (lane & 4) ? fb : fa;
      unsigned may = 0, busy = 0, pa = 0, pb = 0;
      for (int c = 0; c < 16; ++c) may |= (unsigned)(x3_class(rank, c) != x3_class(lane & 7, c)) << c;
      constexpr unsigned UNIT = 435;
      const bool live = lane < 8 && lane != rank;
      for (int step = 0; RELAY && !dead && step < 64; ++step) {
        unsigned k = live ? (F << 3) | lane : 0u;
        k = max(k, (unsigned)__builtin_amdgcn_update_dpp(0u, k, 0x111, 0xf, 0xf, false));
        k = max(k, (unsigned)__builtin_amdgcn_update_dpp(0u, k, 0x112, 0xf, 0xf, false));
        k = max(k, (unsigned)__builtin_amdgcn_update_dpp(0u, k, 0x114, 0xf, 0xf, false));
        const unsigned late = __builtin_amdgcn_readlane(k, 7);
        const int a = late & 7;
        const unsigned cols = may & ~busy & ~(unsigned)__builtin_amdgcn_readlane(busy, a);
        k = (live && lane != a && cols) ? (F << 3) | lane : ~0u;
        k = min(k, (unsigned)__builtin_amdgcn_update_dpp(~0u, k, 0x111, 0xf, 0xf, false));
        k = min(k, (unsigned)__builtin_amdgcn_update_dpp(~0u, k, 0x112, 0xf, 0xf, false));
        k = min(k, (unsigned)__builtin_amdgcn_update_dpp(~0u, k, 0x114, 0xf, 0xf, false));
        const unsigned early = __builtin_amdgcn_readlane(k, 7);
        if (early == ~0u || (late >> 3) < (early >> 3) + RELAY_GAP) break;
        const int b = early & 7, c = __builtin_ctz(__builtin_amdgcn_readlane(cols, b));
        if (lane == a) { F -= UNIT; busy |= 1u << c; }
        if (lane == b) { F += UNIT; busy |= 1u << c; }
        if (lane == (a & 3) * 16 + c) { if (a & 4) pb = 0x80u; else pa = 0x80u; }
        if (lane == (b & 3) * 16 + c) { if (b & 4) pb = 0x08u | a; else pa = 0x08u | a; }
      }
      gu32* const pl = (gu32*)(me + OFF_SFLAG) + lane * 16;
      __hip_atomic_store(pl, call * 256u + pa, __ATOMIC_RELAXED, __HIP_MEMORY_SCOPE_AGENT);
      __hip_atomic_store(pl + 1024, call * 256u + pb, __ATOMIC_RELAXED, __HIP_MEMORY_SCOPE_AGENT);
      return;
    }
    unsigned char* const pw = tws;
    const auto rsrc_peer = mk_rsrc(pw, WS_USED);
    const auto rsrc_me = mk_rsrc(me, WS_USED);
    const auto rsrc_out = mk_rsrc(P.out, OUT_BYTES);
    const gu32* const sfl = (const gu32*)(me + OFF_SFLAG) + (peer * 16 + w) * 16;
    const gu32* const ofl = (const gu32*)(me + OFF_OFLAG) + peer * 512;
    const auto rsrc_sfl = mk_rsrc((const void*)sfl, 64);
    const auto rsrc_ofl = mk_rsrc((const void*)ofl, 2048);
    gu32* const prog = (gu32*)(me + OFF_PROG) + (peer * 16 + w) * 32;
    const unsigned vlin = lane * 16;
    unsigned char* const ubuf = lds + wave * (2 * OSLOT);
    unsigned char* const fbuf = lds + LDS_FLAGS + wave * 1024;
    const unsigned vout = h * 4 * (GN * 4) + r * 16;
    const unsigned zaddr = lds_addr(lds + LDS_FLAGS + 4096 + wave * 256);
    *(LAS unsigned*)(size_t)(zaddr + lane * 4) = 0u;

    unsigned ops = 64;
    u64 hist = 0;
    unsigned pend_op = 0, pend_round = 0;
    gu32* last_word = prog;
    unsigned fl = fl0;
    auto settle = [&](unsigned done_below) {
      if (pend_round && pend_op < done_below) {
        if (lane < 2) __hip_atomic_store(lane ? last_word : prog, call * 16u + 15u, __ATOMIC_RELAXED, __HIP_MEMORY_SCOPE_AGENT);
        ++ops; pend_round = 0;
      }
    };
    auto throttle = [&](int win) {
      const unsigned n = (ops - 1u - (unsigned)(hist >> (8 * win - 8))) & 255u;
      wait_vm_dyn(n);
      settle(ops - n);
    };
    auto piece = [&](const __amdgpu_buffer_rsrc_t rs, unsigned char* dst, unsigned voff, unsigned soff) {
      __builtin_amdgcn_raw_ptr_buffer_load_lds(rs, (LAS void*)dst, 16, voff, soff, 0, AUX_SYS);
      hist = (hist << 8) | (ops & 255u); ++ops;
    };
    auto await_flag = [&](const gu32* f, int idx, unsigned thr) {
      const u64 t0 = __builtin_amdgcn_s_memrealtime();
      while (!dead) {
        fl = __hip_atomic_load(f + (lane & 15), __ATOMIC_RELAXED, __HIP_MEMORY_SCOPE_SYSTEM);
        if ((unsigned)__builtin_amdgcn_readlane(fl, idx) >= thr) break;
        if (give_up(t0, tmo, call)) dead = true;
      }
      __builtin_amdgcn_s_waitcnt(0x0F70);
      __builtin_amdgcn_fence(__ATOMIC_ACQUIRE, "wavefront");
      settle(ops);
    };
    unsigned fbase = 0;
    auto refresh_out = [&](unsigned from) {
      fbase = from & ~1u;
      __builtin_amdgcn_raw_ptr_buffer_load_lds(rsrc_ofl, (LAS void*)fbuf, 16, vlin + fbase * 8u, 0, 0, AUX_SYS); ++ops;
    };

    unsigned char* const ring = ubuf + OSLOT;
    unsigned dprev = 0;
    auto put = [&](int c) {
      const v4u t = *(const v4u*)(ring + (c & 15) * 1024 + vlin);
      STORE16(t, rsrc_me, vlin, dprev + (c & 15) * 1024u, AUX_DEV);
      if ((c & 15) == 15) { pend_op = ops; pend_round = 1; }
      ++ops;
    };
    {
      v4u hold = {0u, 0u, 0u, 0u};
      int tend = 12, relay_src = 0;
      const int cls = x3_class(rank, w);
      constexpr int T0 = PUSHQ == 4 ? 1 : 0, PQ = PUSHQ & 3;
      for (int t = T0; t < tend; ++t) {
        const int it = (t == 9) ? 6 + cls : (t >= 11) ? 7 - cls : item_of_round(t);
        const unsigned xo = x_item_off(t == 12 ? relay_src : peer, it & 7, call);
        const unsigned src = ((it >= 8) ? OFF_WS + rank * MiB + (it - 8) * (256 * KiB) : xo) + w * (16 * KiB);
        const unsigned dcur = ((it >= 8) ? OFF_WG + peer * MiB + (it - 8) * (256 * KiB) : xo) + w * (16 * KiB);
        const gu32* const rprog = (const gu32*)(pw + OFF_PROG) + (relay_src * 16 + w) * 32;
        const auto rs_early = mk_rsrc(me, t > T0 ? WS_USED : 0u);
        static_for<4>([&](auto qc) {
          constexpr int jq = decltype(qc)::v;
          if (jq < PQ && t == 0) return;
          static_for<4>([&](auto uc) {
            constexpr int j = jq * 4 + decltype(uc)::v;
            constexpr int REFRESH_AT = 15 - IN_WIN - 1, PUBLISH_AT = 2 * IN_WIN + 1 < 16 ? 2 * IN_WIN + 1 : 15;
            static_assert(2 * IN_WIN + 1 <= 15 && REFRESH_AT >= 4 * PQ, "window");
            wait_vm<2 * IN_WIN - 1 + ((j > REFRESH_AT && j - IN_WIN <= REFRESH_AT) ? 1 : 0)>();
            const v4u got = *(const v4u*)(ring + ((j - IN_WIN) & 15) * 1024 + vlin);
            if (j == 0 || (j & 3) == 0) {
              if (j == 0 || t == 0) {
                const int idx = (t == 12) ? 0 : t ? round_of_item(it) : 12 + jq;
                const unsigned thr = (t == 12) ? call * 16u + 10u : call;
                if (t > T0) fl = *(const unsigned*)(fbuf + (lane & 15) * 4);
                if ((unsigned)__builtin_amdgcn_readlane(fl, idx) < thr) await_flag(t == 12 ? rprog : sfl, idx, thr);
                if (PUSHQ && t == 2 && (unsigned)__builtin_amdgcn_readlane(fl, 12) < call) await_flag(sfl, 12, call);
                if (t == TS_ROUND && lane == 0)
                  __hip_atomic_store((gu64*)(me + OFF_TS) + peer * 16 + w, ((u64)call << 32) | (unsigned)__builtin_amdgcn_s_memrealtime(), __ATOMIC_RELAXED, __HIP_MEMORY_SCOPE_AGENT);
                if (t == 10) {
                  if ((unsigned)__builtin_amdgcn_readlane(fl, 0) < call * 256u) await_flag(sfl, 0, call * 256u);
                  const unsigned plan = dead ? 0u : __builtin_amdgcn_readlane(fl, 0);
                  tend = (plan & 0x80u) ? 11 : (plan & 0x08u) ? 13 : 12;
                  relay_src = plan & 7u;
                }
              }
            }
            if (j == PUBLISH_AT && t > 1 && lane == 0) __hip_atomic_store(prog + (t == 12), call * 16u + t, __ATOMIC_RELAXED, __HIP_MEMORY_SCOPE_AGENT);
            __builtin_amdgcn_raw_ptr_buffer_load_lds(rsrc_peer, (LAS void*)(ring + jq * 4096), 16, vlin, src + jq * 4096, (j & 3) * 1024, AUX_SYS);
            asm volatile("" ::: "memory");
            if (j <= IN_WIN) STORE16(hold, rs_early, vlin, dprev + (j - IN_WIN - 1 + 16) * 1024, AUX_DEV);
            else if (j - IN_WIN - 1 < 4 * PQ) STORE16(hold, rs_early, vlin, dcur + (j - IN_WIN - 1) * 1024, AUX_DEV);
            else STORE16(hold, rsrc_me, vlin, dcur + (j - IN_WIN - 1) * 1024, AUX_DEV);
            hold = got;
            if (j == REFRESH_AT) {
              if (t + 1 == tend) refresh_out(0);
              else __builtin_amdgcn_raw_ptr_buffer_load_lds(t == 11 ? mk_rsrc((const void*)rprog, 64) : rsrc_sfl, (LAS void*)fbuf, 4, lane * 4, 0, 0, AUX_SYS);
            }
          });
        });
        dprev = dcur;
      }
      if (tend == 12) last_word = prog + 1;
      if (tend == 13) last_word = (gu32*)(me + OFF_PROG) + (relay_src * 16 + w) * 32 + 1;
      ops = 128;
#pragma unroll
      for (int i = 7; i >= 0; --i) hist = (hist << 8) | (ops - 2u - 2u * i);
      STORE16(hold, rsrc_me, vlin, dprev + (15 - IN_WIN) * 1024, AUX_DEV);
      ++ops;
    }

    const gu32* const tick = (const gu32*)(me + OFF_TICK + peer * 128);
    int c = 192 - IN_WIN;
    unsigned last_op = 0;
    unsigned nx = w, par = 0;
    for (int cur = -1;;) {
      const int win = (cur < 0) ? IN_WIN : (nx >= TAIL_FROM ? TAIL_WIN : OUT_WIN);
      unsigned nb = 0;
      if (nx < 256u) {
        unsigned cn = 0, tag = 0;
        if (nx - fbase < 128u) {
          const v2u f = *(const v2u*)(fbuf + (nx - fbase) * 8u);
          cn = __builtin_amdgcn_readfirstlane(f[0]); tag = __builtin_amdgcn_readfirstlane(f[1]);
        }
        if (tag < call) {
          if (c < 192) { WAIT_VM(0); for (; c < 192; ++c) put(c); }
          await_flag(ofl + (nx & ~7u) * 2u, (nx & 7u) * 2u + 1u, call);
          cn = __builtin_amdgcn_readlane(fl, (nx & 7u) * 2u);
        }
        nb = dead ? 0u : ((512u + (cn > 4096u ? 4096u : cn) * 4u + 63u) & ~63u);
      }
      const int n = (int)((nb + 1023u) >> 10);
      const auto rs = mk_rsrc(pw + OFF_OS + rank * OS_OWNER + (nx & 255u) * OSLOT, nb);
      unsigned char* const nbuf = ubuf + par * OSLOT;
      int i = 0;
      unsigned draw_op = 0;
      auto draw = [&]() {
        if (nx < 256u && nx + 16u >= DYN_FROM) {
          asm volatile("s_mov_b64 exec, 1\n\tglobal_atomic_add v250, %0, %1, %2 sc0\n\ts_mov_b64 exec, -1"
                       :: "v"(0u), "v"(1u), "s"(tick) : "memory", "v250");
          draw_op = ops; ++ops;
        }
      };
      refresh_out(nx);
      draw_op = ops - 1u;
      if (cur >= 0) {
        for (; i < n && i < win; ++i) { throttle(win); piece(rs, nbuf + i * 1024, vlin + i * 1024, 0); }
        wait_vm_dyn(ops - 1u - last_op);
        const unsigned char* const cbuf = ubuf + (par ^ 1u) * OSLOT;
        v2u mk = *(const v2u*)(cbuf + lane * 8);
        const unsigned so = (cur >> 4) * 32u * (GN * 4) + (peer * NS + (cur & 15) * 128) * 4u;
        if (dead) { mk = v2u{~0u, ~0u}; *(LAS unsigned*)(size_t)(zaddr + lane * 4) = 0x7fc00000u; }
        const unsigned np = 64u - __popc(mk[0]) - __popc(mk[1]);
        unsigned qa = lds_addr(cbuf) + 512u + (wave_scan(np) - np) * 4u;
        static_for<4>([&](auto qc) {
          constexpr int q = decltype(qc)::v;
          v4u o[4];
          static_for<4>([&](auto ec) {
            constexpr int e = decltype(ec)::v, idx = (q * 4 + e) * 4;
            unsigned a[4];
            run_addr4<31 - (idx & 31)>(mk[idx >> 5], zaddr, qa, a);
#pragma unroll
            for (int f = 0; f < 4; ++f) o[e][f] = *(const LAS unsigned*)(size_t)(a[f] + (idx + f) * 4);
          });
          if (q == DRAW_AT) draw();
#pragma unroll
          for (int e = 0; e < 4; ++e) STORE16(o[e], rsrc_out, vout, so + (q * 8 + e) * (GN * 4), OUT_AUX);
          ops += 4;
          if (i < n) { throttle(win); piece(rs, nbuf + i * 1024, vlin + i * 1024, 0); ++i; }
        });
      } else {
        draw();
      }
      if (nx >= 256u) break;
      for (; i < n; ++i) {
        throttle(win); piece(rs, nbuf + i * 1024, vlin + i * 1024, 0);
        if (c < 192) put(c++);
      }
      last_op = (unsigned)(ops - 1u - ((ops - 1u - (unsigned)hist) & 255u));
      if (cur < 0) {
        if (c < 192) { WAIT_VM(0); for (; c < 192; ++c) put(c); }
        if (pend_round) { wait_vm_dyn(ops - 1u - pend_op); settle(pend_op + 1u); }
        WAIT_LGKM0();
      }
      wait_vm_dyn(ops - 1u - draw_op);
      cur = (int)nx;
      if (nx + 16u >= DYN_FROM) {
        asm volatile("v_readfirstlane_b32 %0, v250" : "=s"(nx) :: "memory");
        nx = nx - (call - 1u) * (272u - DYN_FROM) + DYN_FROM;
      } else nx += 16u;
      par ^= 1u;
    }
    WAIT_VM(0);
    STAMP(5);
    return;
  }

  const int g = slot - 4;
  const auto rsrc_me = mk_rsrc(me, WS_USED);
  {
    if (jb >= 64 && jb < 576) {
      const int s = (jb - 64) >> 7, q = ((jb - 64) >> 4) & 7, j = jb & 15;
      STAGE_DELAY(s == 0 ? DELAY1 : DELAY2);
      const auto rw = mk_rsrc(wp, KS * GN);
      const unsigned voff = h * 16 * GN + r * 4;
      const unsigned so = s * 128 * GN + q * NS + j * 128;
      unsigned d[64];
#pragma unroll
      for (int e = 0; e < 64; ++e)
        d[e] = __builtin_amdgcn_raw_buffer_load_b32(rw, voff, so + ((e >> 4) * 32 + (e & 15)) * GN, 0);
      const unsigned dst = ((q == rank) ? OFF_WG + rank * MiB : OFF_WS + q * MiB) + s * (256 * KiB) + j * (16 * KiB);
#pragma unroll
      for (int ki = 0; ki < 4; ++ki) {
        v4u o[4];
#pragma unroll
        for (int gq = 0; gq < 4; ++gq) {
          unsigned a = d[ki * 16 + gq * 4], b = d[ki * 16 + gq * 4 + 1], c = d[ki * 16 + gq * 4 + 2], e = d[ki * 16 + gq * 4 + 3];
          unsigned t0 = __builtin_amdgcn_perm(b, a, 0x05010400u), t1 = __builtin_amdgcn_perm(b, a, 0x07030602u);
          unsigned t2 = __builtin_amdgcn_perm(e, c, 0x05010400u), t3 = __builtin_amdgcn_perm(e, c, 0x07030602u);
          o[0][gq] = __builtin_amdgcn_perm(t2, t0, 0x05040100u);
          o[1][gq] = __builtin_amdgcn_perm(t2, t0, 0x07060302u);
          o[2][gq] = __builtin_amdgcn_perm(t3, t1, 0x05040100u);
          o[3][gq] = __builtin_amdgcn_perm(t3, t1, 0x07060302u);
        }
#pragma unroll
        for (int f = 0; f < 4; ++f)
          STORE16(o[f], rsrc_me, lane * 16, dst + f * 4096 + ki * 1024, AUX_SYS);
      }
      WAIT_VM(0);
      if (lane == 0)
        __hip_atomic_store((gu32*)(tws + OFF_SFLAG) + (rank * 16 + j) * 16 + round_of_w(s), CALL_NOW(), __ATOMIC_RELAXED, __HIP_MEMORY_SCOPE_SYSTEM);
    } else if (jb < 688 + (PUSHQ ? 112 : 0)) {
      const bool quarter = jb < 64, push = jb >= 688;
      const int un = (quarter || push) ? (jb & 15) : jb - 576 + 16;
      const int rb0 = quarter ? (jb >> 4) : 0, nrb = quarter ? 1 : (push ? PUSHQ : 4);
      const int dest = push ? tgt : rank;
      if (!quarter && !push) STAGE_DELAY(DELAY2);
      const auto rx = mk_rsrc(xp, GM * KS);
      const auto rd = mk_rsrc(push ? tws : me, WS_USED);
      const unsigned voff = r * KS + h * 16;
      const unsigned so = ((un & 7) * MS + (un >> 5) * 128) * KS + ((un >> 3) & 3) * 128;
      v4u d[16];
#pragma unroll
      for (int gq = 0; gq < 4; ++gq)
        if (gq < nrb) {
#pragma unroll
          for (int e = 0; e < 4; ++e) d[gq * 4 + e] = __builtin_amdgcn_raw_buffer_load_b128(rx, voff + e * 32, so + (rb0 + gq) * 32 * KS, 0);
        }
      __builtin_amdgcn_s_waitcnt(0x0F70);
      const unsigned dst = x_item_off(rank, un >> 4, CALL_NOW()) + (un & 15) * (16 * KiB) + rb0 * 4096;
#pragma unroll
      for (int gq = 0; gq < 4; ++gq)
        if (gq < nrb) {
#pragma unroll
          for (int e = 0; e < 4; ++e) {
            if (PUSH_WIN && push) wait_vm<(PUSH_WIN > 0 ? PUSH_WIN - 1 : 0)>();
            STORE16(d[gq * 4 + e], rd, lane * 16 + e * 1024, dst + gq * 4096, AUX_SYS);
          }
        }
      WAIT_VM(0);
      const bool tell = push ? (lane == dest) : ((quarter && rb0 < PUSHQ) ? (lane == rank) : (lane < NR));
      if (tell)
        __hip_atomic_store((gu32*)(dws + OFF_SFLAG) + (rank * 16 + (un & 15)) * 16 + ((quarter || push) ? 12 + rb0 : round_of_x(un >> 4)), CALL_NOW(), __ATOMIC_RELAXED, __HIP_MEMORY_SCOPE_SYSTEM);
    }
  }
  const unsigned call = CALL_NOW();
  if (lane == 0) __hip_atomic_store(cnt, call, __ATOMIC_RELAXED, __HIP_MEMORY_SCOPE_AGENT);
  STAMP(1);

  const float scale = P.sx[0] * P.sw[0];
  const int ntiles = (g < 4) ? 2 : 1;
  const int wm = wave >> 1, wn = wave & 1;
  for (int it = 0; it < ntiles; ++it) {
    const int t14 = (g >= 14) ? g - 14 : g;
    const int ch = (xcd & 1) + (it ? (g & 2) : (g >= 14 ? 2 : 0));
    const int own = it ? rank : ((rank + 1 + (t14 >> 1)) & 7);
    const int jj = (xcd >> 1) * 2 + (g & 1);
    const int wh = lane & 3;
    const int wsel = (wh < 2) ? 2 * jj + wh : own + (wh - 2) * 8;
    const gu64* paddr = (const gu64*)(me + OFF_PROG) + (((rank + 1 + ((lane >> 2) & 7)) & 7) * 16 + wsel) * 16;
    v2u* const order = (v2u*)(lds + LDS_ORDER);
    unsigned taken = 0;
    int nq = 4;
    auto refill = [&](int want) {
      const u64 t0 = __builtin_amdgcn_s_memrealtime();
      for (;;) {
        const u64 pv = dead ? ~0ull : __hip_atomic_load(paddr, __ATOMIC_RELAXED, __HIP_MEMORY_SCOPE_AGENT);
        for (int ks = 0; ks < 4; ++ks) {
          const bool final_unit = wh >= 2 && ch == 3 && (ks >> 1) != x3_class(rank, wsel);
          const int rnd = (wh < 2) ? round_of_w(ks) : (ch == 3) ? 9 : round_of_x(ch * 2 + (ks >> 1));
          const bool ok = (final_unit ? (unsigned)(pv >> 32) >= call * 16u : (unsigned)pv >= call * 16u + 1u + rnd) || (wh >= 2 && (wh - 2) != (ks & 1));
          const unsigned m = (unsigned)__ballot(ok);
          const unsigned rdy = m & (m >> 1) & (m >> 2) & (m >> 3) & 0x01111111u & ~(taken >> ks);
          if (lane < 32 && ((rdy >> lane) & 1u)) {
            const unsigned p = (rank + 1 + (lane >> 2)) & 7;
            order[nq + __builtin_amdgcn_mbcnt_lo(rdy, 0u)] = v2u{x_item_off(p, ch * 2 + (ks >> 1), call) + (ks & 1u) * (128 * KiB), p * MiB + ks * (256 * KiB)};
          }
          taken |= rdy << ks;
          nq += __popc(rdy);
        }
        if (nq > want) break;
        if (give_up(t0, tmo, call)) dead = true;
        __builtin_amdgcn_s_sleep(GEMM_POLL_SLEEP);
      }
      __builtin_amdgcn_s_waitcnt(0x0F70);
      __builtin_amdgcn_fence(__ATOMIC_ACQUIRE, "wavefront");
    };

    const unsigned sa0 = own * (16 * KiB) + wave * 4096;
    const unsigned sb0 = OFF_WG + jj * (32 * KiB) + wave * 8192;
    v16f acc[2][4];
#pragma unroll
    for (int mi = 0; mi < 2; ++mi)
#pragma unroll
      for (int f = 0; f < 4; ++f)
#pragma unroll
        for (int e = 0; e < 16; ++e) acc[mi][f][e] = 0.f;

    if (wave == 0) {
      if (lane < 8) order[32 + lane] = v2u{0u, 0u};
      const int wq = lane >> 4;
      const gu32* a = (const gu32*)(me + OFF_SFLAG) + (rank * 16 + ((wq < 2) ? 2 * jj + wq : own + (wq - 2) * 8)) * 16 + (lane & 15);
      const u64 wbits = (1u << round_of_w(0)) | (1u << round_of_w(1)) | (1u << round_of_w(2)) | (1u << round_of_w(3));
      const u64 xbits = (ch ? (1u << round_of_x(ch * 2)) : 0xf000u) | (1u << round_of_x(ch * 2 + 1));
      unsigned dummy;
      wait_words(a, call, wbits | (wbits << 16) | (xbits << 32) | (xbits << 48), tmo, call, dead, dummy);
    }
    __builtin_amdgcn_s_barrier();
#define MF(A, B, mi, f) acc[mi][f] = __builtin_amdgcn_mfma_scale_f32_32x32x64_f8f6f4(A[mi], B[f], acc[mi][f], 0, 0, 0, 0, 0, 0)
#define DM(l, so, o) __builtin_amdgcn_raw_ptr_buffer_load_lds(rs, (LAS void*)(l), 16, lane * 16, (so), (o), GEMM_AUX)
#define FR(dst, ptr) dst = __builtin_shufflevector(*(const v4i*)(ptr), *(const v4i*)((ptr) + 1024), 0, 1, 2, 3, 4, 5, 6, 7)
#define SB() __builtin_amdgcn_sched_barrier(0)
#pragma unroll
    for (int s = 0; s < 3; ++s) {
      const auto rs = rsrc_me;
      const unsigned sa = sa0 + x_item_off(rank, ch * 2 + (s >> 1), call) + (s & 1) * (128 * KiB), sb = sb0 + rank * MiB + s * (256 * KiB);
      unsigned char* const la = lds + s * STAGE_BYTES + wave * 4096;
      unsigned char* const lb = lds + s * STAGE_BYTES + STAGE_A + wave * 8192;
      DM(la, sa, 0); DM(la, sa, 1024); DM(lb, sb, 0); DM(lb, sb, 1024); DM(lb + 4096, sb + 4096, 0); DM(lb + 4096, sb + 4096, 1024); SB();
      DM(la, sa, 2048); DM(la, sa, 3072); DM(lb, sb, 2048); DM(lb, sb, 3072); DM(lb + 4096, sb + 4096, 2048); DM(lb + 4096, sb + 4096, 3072); SB();
    }
    WAIT_VM(30);
    __builtin_amdgcn_s_barrier();
    v8i A0[2], B0[4], A1[2], B1[4];
    STAMP(6 + it);
    const unsigned char* const ra0 = lds + lane * 16 + wm * 8192;
    const unsigned char* const rb0 = lds + lane * 16 + STAGE_A + wn * 16384;
    FR(A0[0], ra0); FR(A0[1], ra0 + 4096); FR(B0[0], rb0); FR(B0[1], rb0 + 4096); FR(B0[2], rb0 + 8192); FR(B0[3], rb0 + 12288);
    int buf = 0;
    v2u ent = {x_item_off(rank, ch * 2 + 1, call) + 128 * KiB, rank * MiB + 3 * (256 * KiB)};
    for (int k = 0; k < 32; ++k) {
      const auto rs = mk_rsrc(me, k < 29 ? WS_USED : 0u);
      const unsigned sa = sa0 + __builtin_amdgcn_readfirstlane(ent[0]);
      const unsigned sb = sb0 + __builtin_amdgcn_readfirstlane(ent[1]);
      unsigned char* const la = lds + buf + wave * 4096;
      unsigned char* const lb = lds + buf + STAGE_A + wave * 8192;
      const unsigned char* ra = ra0 + buf + 2048;
      const unsigned char* rb = rb0 + buf + 2048;
      WAIT_LGKM0();
      WAIT_VM(24);
      __builtin_amdgcn_s_barrier();
      SB();
      MF(A0, B0, 0, 0); SB(); FR(A1[0], ra); DM(la, sa, 0); SB();
      MF(A0, B0, 0, 1); SB(); FR(B1[0], rb); DM(la, sa, 1024); SB();
      MF(A0, B0, 0, 2); SB(); FR(B1[1], rb + 4096); DM(lb, sb, 0); SB();
      MF(A0, B0, 0, 3); SB(); FR(B1[2], rb + 8192); DM(lb, sb, 1024); SB();
      MF(A0, B0, 1, 0); SB(); FR(B1[3], rb + 12288); DM(lb + 4096, sb + 4096, 0); SB();
      MF(A0, B0, 1, 1); SB(); FR(A1[1], ra + 4096); DM(lb + 4096, sb + 4096, 1024); SB();
      MF(A0, B0, 1, 2); SB();
      buf = (buf == 2 * STAGE_BYTES) ? 0 : buf + STAGE_BYTES;
      ra = ra0 + buf; rb = rb0 + buf;
      MF(A0, B0, 1, 3); SB();
      if (wave == 0 && k < 28 && nq <= k + 4) refill(k + 4);
      WAIT_LGKM0();
      WAIT_VM(24);
      __builtin_amdgcn_s_barrier();
      SB();
      MF(A1, B1, 0, 0); SB(); ent = order[k + 4]; FR(A0[0], ra); DM(la, sa, 2048); SB();
      MF(A1, B1, 0, 1); SB(); FR(B0[0], rb); DM(la, sa, 3072); SB();
      MF(A1, B1, 0, 2); SB(); FR(B0[1], rb + 4096); DM(lb, sb, 2048); SB();
      MF(A1, B1, 0, 3); SB(); FR(B0[2], rb + 8192); DM(lb, sb, 3072); SB();
      MF(A1, B1, 1, 0); SB(); FR(B0[3], rb + 12288); DM(lb + 4096, sb + 4096, 2048); SB();
      MF(A1, B1, 1, 1); SB(); FR(A0[1], ra + 4096); DM(lb + 4096, sb + 4096, 3072); SB();
      MF(A1, B1, 1, 2); SB();
      MF(A1, B1, 1, 3); SB();
    }
    WAIT_VM(0);
    WAIT_LGKM0();
    __builtin_amdgcn_s_barrier();
    STAMP(2 + 2 * it);
    if (ch == 3 && own != rank) {
      unsigned dummy;
      wait_words((const gu32*)(me + OFF_PROG) + (own * 16 + (lane & 15)) * 32, call * 16u + 15u, ~0ull, tmo, call, dead, dummy);
    }
    const bool poison = __builtin_amdgcn_readfirstlane(__hip_atomic_load(tmo, __ATOMIC_RELAXED, __HIP_MEMORY_SCOPE_AGENT)) == call;
    const float sc = poison ? __uint_as_float(0x7fc00000u) : scale;
    if (own == rank) {
      const auto rdst = mk_rsrc(P.out, OUT_BYTES);
      const unsigned vo = h * 4 * (GN * 4) + r * 16;
      const unsigned sbase = (ch * 128 + wm * 64) * (GN * 4) + (rank * NS + jj * 256 + wn * 128) * 4;
#pragma unroll
      for (int mi = 0; mi < 2; ++mi)
#pragma unroll
        for (int e = 0; e < 16; ++e) {
          v4u o;
#pragma unroll
          for (int f = 0; f < 4; ++f) { const float y = acc[mi][f][e] * sc; const unsigned b = __float_as_uint(y); o[f] = b & ~(unsigned)((int)b >> 31); }
          STORE16(o, rdst, vo, sbase + (mi * 32 + (e >> 2) * 8 + (e & 3)) * (GN * 4), AUX_DEV);
        }
    } else {
      unsigned cnt0 = 0;
#pragma unroll
      for (int mi = 0; mi < 2; ++mi) {
        unsigned char* const cb = lds + (wave * 2 + mi) * OSLOT;
        float y[64];
        v2u mk = {0u, 0u};
#pragma unroll
        for (int i = 0; i < 64; ++i) {
          y[i] = acc[mi][i & 3][i >> 2] * sc;
          mk[i >> 5] = __builtin_amdgcn_alignbit(mk[i >> 5], __float_as_uint(y[i]), 31);
        }
        *(v2u*)(cb + lane * 8) = mk;
        const unsigned np = 64u - __popc(mk[0]) - __popc(mk[1]);
        const unsigned inc = wave_scan(np);
        const unsigned base = __builtin_amdgcn_readlane(inc, 63);
        unsigned qa = lds_addr(cb) + 512u + (inc - np) * 4u;
        const unsigned junk = lds_addr(lds + LDS_FLAGS + wave * 512) + lane * 4;
        static_for<16>([&](auto ec) {
          constexpr int idx = decltype(ec)::v * 4;
          unsigned a[4];
          run_addr4<31 - (idx & 31)>(mk[idx >> 5], junk, qa, a);
#pragma unroll
          for (int f = 0; f < 4; ++f) *(LAS float*)(size_t)(a[f] + (idx + f) * 4) = y[idx + f];
        });
        gu64* const oflag = (gu64*)(P.ws[own] + OFF_OFLAG) + rank * 256 + (ch * 4 + wm * 2) * 16 + jj * 2 + wn;
        if (mi == 1) {
          WAIT_VM(0);
          if (lane == 0) __hip_atomic_store(oflag, ((u64)call << 32) | cnt0, __ATOMIC_RELAXED, __HIP_MEMORY_SCOPE_SYSTEM);
        }
        const unsigned dst = OFF_OS + own * OS_OWNER + ((ch * 4 + wm * 2 + mi) * 16 + jj * 2 + wn) * OSLOT;
        const int n = (int)((512u + base * 4u + 1023u) >> 10);
        for (int i = 0; i < n; i += 3) {
          const auto rslot = mk_rsrc(me + dst, OSLOT);
          v4u t[3];
#pragma unroll
          for (int u = 0; u < 3; ++u) t[u] = *(const v4u*)(cb + (i + u) * 1024 + lane * 16);
#pragma unroll
          for (int u = 0; u < 3; ++u) STORE16(t[u], rslot, lane * 16 + (i + u) * 1024, 0, AUX_SYS);
        }
        if (mi == 0) cnt0 = base;
        else {
          WAIT_VM(0);
          if (lane == 0) __hip_atomic_store(oflag + 16, ((u64)call << 32) | base, __ATOMIC_RELAXED, __HIP_MEMORY_SCOPE_SYSTEM);
        }
      }
      WAIT_LGKM0();
    }
    STAMP(3 + 2 * it);
  }
}

}

extern "C" void kernel_launch(int rank, int nranks, ncclComm_t comm,
                              hipStream_t stream,
                              void* const* d_in, const int* in_sizes, int n_in,
                              void* d_out, int out_size,
                              const int* in_dtypes, int out_dtype,
                              void* const* d_out_all,
                              void* const* d_ws_all, size_t ws_bytes) {
  if (nranks != NR || n_in < 4 || ws_bytes < WS_USED || out_size != MS * GN ||
      in_sizes[0] != GM * KS || in_sizes[1] != KS * GN) {
    fprintf(stderr, "unexpected problem shape\n");
    return;
  }
  Params P;
  P.x = (const unsigned char*)d_in[0];
  P.w = (const unsigned char*)d_in[1];
  P.sx = (const float*)d_in[2];
  P.sw = (const float*)d_in[3];
  P.out = (float*)d_out;
  for (int p = 0; p < NR; ++p) P.ws[p] = (unsigned char*)d_ws_all[p];
  P.rank = rank; P.pad = 0;
  P.me = P.ws[rank];
  hipLaunchKernelGGL(fused_kernel, dim3(256), dim3(256), 0, stream, P);
  HIP_CHECK(hipGetLastError());
}
```

```cpp
#include <hip/hip_runtime.h>
#include <rccl/rccl.h>
#include <cstdio>
#include <cstdint>
#include <cstddef>

#define HIP_CHECK(expr) do { hipError_t _e = (expr); if (_e != hipSuccess) { \
    fprintf(stderr, "HIP error %s at %s:%d\n", hipGetErrorString(_e), __FILE__, __LINE__); \
    return; } } while (0)

namespace {

typedef int v8i __attribute__((ext_vector_type(8)));
typedef int v4i __attribute__((ext_vector_type(4)));
typedef unsigned v4u __attribute__((ext_vector_type(4)));
typedef unsigned v2u __attribute__((ext_vector_type(2)));
typedef float v16f __attribute__((ext_vector_type(16)));
typedef unsigned v8u __attribute__((ext_vector_type(8)));
typedef unsigned v16u __attribute__((ext_vector_type(16)));
typedef unsigned long long u64;
#define LAS __attribute__((address_space(3)))
#define GAS __attribute__((address_space(1)))
typedef GAS unsigned gu32;
typedef GAS u64 gu64;

constexpr int NR = 8;
constexpr int GM = 4096, GK = 4096, GN = 16384;
constexpr int KS = GK / NR;
constexpr int MS = GM / NR;

constexpr unsigned KiB = 1024u, MiB = 1024u * 1024u;
constexpr unsigned OFF_CNT   = 0;
constexpr unsigned OFF_TMO   = 64 * KiB;
constexpr unsigned OFF_SCNT  = 65 * KiB;
constexpr unsigned OFF_IFLAG = 71 * KiB;
constexpr unsigned OFF_TICK  = 72 * KiB;
constexpr unsigned OFF_TS    = 76 * KiB;
constexpr unsigned OFF_SFLAG = 80 * KiB;
constexpr unsigned OFF_OFLAG = 96 * KiB;
constexpr unsigned OFF_PROG  = 112 * KiB;
constexpr unsigned OFF_XG    = 1 * MiB;
constexpr unsigned OFF_WG    = OFF_XG + 16 * MiB;
constexpr unsigned OFF_WS    = OFF_WG + 16 * MiB;
constexpr unsigned OFF_OS    = OFF_WS + 8 * MiB;
constexpr unsigned OSLOT     = 17 * KiB;
constexpr unsigned PLAIN_FROM = 3968;
constexpr unsigned OS_OWNER  = 256 * OSLOT;
constexpr unsigned OFF_XALT  = OFF_OS + 8 * OS_OWNER;
constexpr unsigned WS_USED   = OFF_XALT + 4 * MiB;
constexpr unsigned OUT_BYTES = MS * GN * 4;

constexpr unsigned long long DEADLINE = 50000000ull;
constexpr unsigned TOPO_MASK = 0x4Du;
constexpr int AUX_SYS = 17;
constexpr int AUX_DEV = 16;

constexpr int IN_WIN = 3, OUT_WIN = 3;
constexpr int PUSHQ = 1, PUSH_WIN = 2;
constexpr int DELAY1 = 8, DELAY2 = 72, DELAY3 = 120;
constexpr int TS_ROUND = 2;
constexpr int PLAN_POS = 24;
constexpr unsigned UNIT = 432;
constexpr unsigned SAME_BIAS = 13200;
constexpr unsigned RELAY_GAP = 800;

constexpr int STAGE_A = 16 * 1024, STAGE_B = 32 * 1024, STAGE_BYTES = STAGE_A + STAGE_B;
constexpr int LDS_ORDER = 3 * STAGE_BYTES;
constexpr int LDS_BYTES = LDS_ORDER + 512 + 256;
constexpr int LDS_FLAGS = 8 * OSLOT;
static_assert(LDS_FLAGS + 5120 <= LDS_BYTES, "lds");

struct Params {
  const unsigned char* x;
  const unsigned char* w;
  unsigned char* me;
  int rank, pad;
  const float* sx;
  const float* sw;
  float* out;
  unsigned char* ws[NR];
};

#define WAIT_VM(n) asm volatile("s_waitcnt vmcnt(" #n ")" ::: "memory")
#define STORE16(val, rs, voff, soff, aux) do { const v4u sv_ = (val); \
    __builtin_amdgcn_raw_buffer_store_b128(sv_, rs, voff, soff, aux); \
    asm volatile("s_nop 1" :: "v"(sv_) : "memory"); } while (0)
template <int N> __device__ __forceinline__ void wait_vm() { asm volatile("s_waitcnt vmcnt(%0)" :: "i"(N) : "memory"); }
__device__ __forceinline__ void wait_vm_dyn(unsigned n) {
  n = n > 31u ? 31u : n;
#define WV1(k) "s_waitcnt vmcnt(" #k ")\n\ts_branch 1f\n\t"
  asm volatile("s_getpc_b64 vcc\n\t"
               "s_lshl3_add_u32 %0, %0, 16\n\t"
               "s_add_u32 vcc_lo, vcc_lo, %0\n\t"
               "s_addc_u32 vcc_hi, vcc_hi, 0\n\t"
               "s_setpc_b64 vcc\n\t"
               WV1(0) WV1(1) WV1(2) WV1(3) WV1(4) WV1(5) WV1(6) WV1(7) WV1(8) WV1(9) WV1(10) WV1(11) WV1(12) WV1(13) WV1(14) WV1(15)
               WV1(16) WV1(17) WV1(18) WV1(19) WV1(20) WV1(21) WV1(22) WV1(23) WV1(24) WV1(25) WV1(26) WV1(27) WV1(28) WV1(29) WV1(30) WV1(31)
               "1:"
               : "+s"(n) :: "memory", "scc", "vcc");
#undef WV1
}
#define BARRIER() do { __builtin_amdgcn_s_barrier(); asm volatile("" ::: "memory"); } while (0)
#define WAIT_LGKM0() asm volatile("s_waitcnt lgkmcnt(0)" ::: "memory")

__device__ __forceinline__ __amdgpu_buffer_rsrc_t mk_rsrc(const void* p, unsigned bytes) {
  return __builtin_amdgcn_make_buffer_rsrc((void*)p, 0, bytes, 0x00020000);
}
__device__ __forceinline__ unsigned wave_scan(unsigned x) {
  x += __builtin_amdgcn_update_dpp(0u, x, 0x111, 0xf, 0xf, false);
  x += __builtin_amdgcn_update_dpp(0u, x, 0x112, 0xf, 0xf, false);
  x += __builtin_amdgcn_update_dpp(0u, x, 0x114, 0xf, 0xf, false);
  x += __builtin_amdgcn_update_dpp(0u, x, 0x118, 0xf, 0xf, false);
  x += __builtin_amdgcn_update_dpp(0u, x, 0x142, 0xa, 0xf, false);
  x += __builtin_amdgcn_update_dpp(0u, x, 0x143, 0xc, 0xf, false);
  return x;
}
template <int P>
__device__ __forceinline__ void run_addr4(unsigned m, unsigned other, unsigned& q, unsigned (&a)[4]) {
  unsigned t;
  asm("v_bfe_i32 %5, %6, %8, 1\n\tv_bfi_b32 %0, %5, %7, %4\n\tv_lshl_add_u32 %4, %5, 2, %4\n\t"
      "v_bfe_i32 %5, %6, %9, 1\n\tv_bfi_b32 %1, %5, %7, %4\n\tv_lshl_add_u32 %4, %5, 2, %4\n\t"
      "v_bfe_i32 %5, %6, %10, 1\n\tv_bfi_b32 %2, %5, %7, %4\n\tv_lshl_add_u32 %4, %5, 2, %4\n\t"
      "v_bfe_i32 %5, %6, %11, 1\n\tv_bfi_b32 %3, %5, %7, %4\n\tv_lshl_add_u32 %4, %5, 2, %4"
      : "=&v"(a[0]), "=&v"(a[1]), "=&v"(a[2]), "=&v"(a[3]), "+v"(q), "=&v"(t)
      : "v"(m), "v"(other), "i"(P), "i"(P - 1), "i"(P - 2), "i"(P - 3));
}
template <int E> struct IC { static constexpr int v = E; };
template <int N, int I = 0, class F>
__device__ __forceinline__ void static_for(F&& f) { if constexpr (I < N) { f(IC<I>{}); static_for<N, I + 1>(f); } }
__device__ __forceinline__ unsigned lds_addr(const void* p) { return (unsigned)(size_t)(const LAS unsigned char*)p; }

__device__ __forceinline__ bool give_up(u64 t0, gu32* tmo, unsigned call, unsigned seen) {
  if (__builtin_amdgcn_readfirstlane(seen) == call || __builtin_amdgcn_s_memrealtime() - t0 > DEADLINE) {
    __hip_atomic_store(tmo, call, __ATOMIC_RELAXED, __HIP_MEMORY_SCOPE_AGENT);
    return true;
  }
  __builtin_amdgcn_s_sleep(4);
  return false;
}
__device__ __forceinline__ bool give_up(u64 t0, gu32* tmo, unsigned call) {
  return give_up(t0, tmo, call, __hip_atomic_load(tmo, __ATOMIC_RELAXED, __HIP_MEMORY_SCOPE_AGENT));
}

__device__ __forceinline__ u64 wait_words(const gu32* addr, unsigned thr, u64 need,
                                          gu32* tmo, unsigned call, bool& dead, unsigned& val) {
  u64 t0 = __builtin_amdgcn_s_memrealtime();
  u64 ok;
  for (;;) {
    val = __hip_atomic_load(addr, __ATOMIC_RELAXED, __HIP_MEMORY_SCOPE_SYSTEM);
    ok = __ballot(val >= thr);
    if ((ok & need) == need) break;
    if (give_up(t0, tmo, call)) { dead = true; ok = ~0ull; val = 0xffffffffu; break; }
  }
  __builtin_amdgcn_fence(__ATOMIC_ACQUIRE, "wavefront");
  return ok;
}

__device__ __forceinline__ int pos_of(int rk) { return (0x73652140u >> (4 * rk)) & 7; }
__device__ __forceinline__ int rank_at(int pos) { return (0x75416320u >> (4 * pos)) & 7; }
__device__ __forceinline__ int partner_of(int rk, int ph) { const int p = pos_of(rk); return rank_at(((p & 4) ^ 4) | ((ph - p) & 3)); }
__device__ __forceinline__ int block_of(int rk, int ph) { const int p = pos_of(rk), o = (ph - p) & 3; return (p & 4) ? o * 4 + (p & 3) : (p & 3) * 4 + o; }
constexpr u64 ROUND_CODES = 0xfdecb3a29180ull, CODE_ROUNDS = 0xb9a8753100006420ull;
__device__ __forceinline__ int code_of_round(int t) { return (int)((ROUND_CODES >> (4 * t)) & 15u); }
__device__ __forceinline__ int round_of_code(int c) { return (int)((CODE_ROUNDS >> (4 * c)) & 15u); }

__device__ __forceinline__ unsigned x_item_off(int p, int it, unsigned call) {
  return ((it == 0 || it == 5) && (call & 1u)) ? OFF_XALT + (p * 2 + (it & 1)) * (256 * KiB) : OFF_XG + p * (2 * MiB) + it * (256 * KiB);
}

__global__ void __launch_bounds__(256, 1) fused_kernel(Params P) {
  __shared__ __attribute__((aligned(1024))) unsigned char lds[LDS_BYTES];
  v8u ka; v16u kb;
  asm volatile("s_load_dwordx8 %0, %2, 0x0\n\ts_load_dwordx16 %1, %2, 0x38" : "=&s"(ka), "=&s"(kb) : "s"(__builtin_amdgcn_kernarg_segment_ptr()));
  static_assert(offsetof(Params, me) == 16 && offsetof(Params, rank) == 24 && offsetof(Params, ws) == 0x38, "layout");
  u64 code_pc; unsigned code_len;
  asm volatile("s_getpc_b64 %0\n1:\n\ts_mov_b32 %1, .Lcode_mark_a-1b\n\ts_max_u32 %1, %1, .Lcode_mark_b-1b\n\ts_max_u32 %1, %1, .Lcode_mark_c-1b"
               : "=s"(code_pc), "=s"(code_len) :: "scc");
  {
    const auto rc = mk_rsrc((const void*)code_pc, code_len & ~3u);
    __builtin_amdgcn_raw_ptr_buffer_load_lds(rc, (LAS void*)(lds + LDS_ORDER + 512), 4, threadIdx.x * 128, 0, 0, 0);
    __builtin_amdgcn_raw_ptr_buffer_load_lds(rc, (LAS void*)(lds + LDS_ORDER + 512), 4, threadIdx.x * 128 + 32768, 0, 0, 0);
    asm volatile("" ::: "memory");
  }
  const int lane = threadIdx.x & 63;
  const int wave = __builtin_amdgcn_readfirstlane(threadIdx.x >> 6);
  const int h = lane >> 5, r = lane & 31;
  const int xcd = blockIdx.x & 7, slot = blockIdx.x >> 3;
  const int jb = wave * 224 + (slot - 4) * 8 + xcd;
  const bool express = slot >= 4 && (jb < 128 || jb >= 768);
  const int fg = (jb >> 6) & 1, rb0 = (jb < 128) ? (jb >> 4) & 3 : 0;
  unsigned xoff = ((jb & 7) * MS + fg * 256 + rb0 * 32) * KS + fg * 256 + ((jb >> 3) & 1) * 128;
  asm volatile("s_waitcnt lgkmcnt(0)" : "+s"(ka), "+s"(kb), "+s"(xoff));
  auto pair = [](unsigned lo, unsigned hi) { return ((u64)hi << 32) | lo; };
  const unsigned char* const xp = (const unsigned char*)pair(ka[0], ka[1]);
  unsigned char* const me = (unsigned char*)pair(ka[4], ka[5]);
  v4u xe[4];
  if (express) {
    const auto rx = mk_rsrc(xp, GM * KS);
#pragma unroll
    for (int e = 0; e < 4; ++e)
      xe[e] = __builtin_amdgcn_raw_buffer_load_b128(rx, r * KS + h * 16 + e * 32, xoff, 0);
  }
  gu32* const cnt = (gu32*)(me + OFF_CNT + (blockIdx.x * 4 + wave) * 64);
  const unsigned prev_v = __hip_atomic_load(cnt, __ATOMIC_RELAXED, __HIP_MEMORY_SCOPE_AGENT);
#define CALL_NOW() ((unsigned)__builtin_amdgcn_readfirstlane(prev_v) + 1u)
  const int rank = ka[6] & 7;
  const int xcc = __builtin_amdgcn_s_getreg((20 << 0) | (0 << 6) | (3 << 11)) & 7;
  const int grp = pos_of(rank) >> 2;
  const int peer = rank_at((xcc < 4) ? grp * 4 + (((slot & 3) + ((slot & 3) >= (pos_of(rank) & 3))) & 3) : (grp ^ 1) * 4 + (slot & 3));
  const int w = (xcc & 3) * 4 + wave;
  const unsigned foff = ((lane & 16) ? OFF_SFLAG + (peer * 16 + w) * 128 : OFF_IFLAG + peer * 128) + (lane & 31) * 4;
  unsigned fl0 = 0;
  if (slot < 4) fl0 = __hip_atomic_load((const gu32*)(me + foff), __ATOMIC_RELAXED, __HIP_MEMORY_SCOPE_SYSTEM);
  const unsigned char* const wp = (const unsigned char*)pair(ka[2], ka[3]);
  const u64 w0 = pair(kb[0], kb[1]), w1 = pair(kb[2], kb[3]), w2 = pair(kb[4], kb[5]), w3 = pair(kb[6], kb[7]);
  const u64 w4 = pair(kb[8], kb[9]), w5 = pair(kb[10], kb[11]), w6 = pair(kb[12], kb[13]), w7 = pair(kb[14], kb[15]);
  auto ws_of = [=](int p) {
    const u64 a = (p & 1) ? w1 : w0, b = (p & 1) ? w3 : w2, c = (p & 1) ? w5 : w4, d = (p & 1) ? w7 : w6;
    const u64 e = (p & 2) ? b : a, f = (p & 2) ? d : c;
    return (unsigned char*)((p & 4) ? f : e);
  };
  gu32* const tmo = (gu32*)(me + OFF_TMO);
  const unsigned quad = ((TOPO_MASK >> rank) & 1) ? TOPO_MASK : (~TOPO_MASK & 0xffu);
  const int tgt = (slot < 4) ? peer : rank_at((jb >> 4) & 7);
  unsigned char* const tws = ws_of(tgt);
  unsigned char* const dws = ws_of(lane & 7);
  bool dead = false;

  if (slot < 4) {
    if (xcc < 4 && slot == 3) {
      const unsigned call = CALL_NOW();
      if (lane == 0) __hip_atomic_store(cnt, call, __ATOMIC_RELAXED, __HIP_MEMORY_SCOPE_AGENT);
      if (xcc != 0 || wave != 0) return;
      const gu64* const ts = (const gu64*)(me + OFF_TS);
      const u64 t0 = __builtin_amdgcn_s_memrealtime();
      u64 ta, tb;
      for (;;) {
        __builtin_amdgcn_s_sleep(20);
        ta = __hip_atomic_load(ts + lane, __ATOMIC_RELAXED, __HIP_MEMORY_SCOPE_AGENT);
        tb = __hip_atomic_load(ts + 64 + lane, __ATOMIC_RELAXED, __HIP_MEMORY_SCOPE_AGENT);
        const bool ok = ((unsigned)(ta >> 32) >= call || (lane >> 4) == rank) && ((unsigned)(tb >> 32) >= call || (lane >> 4) + 4 == rank);
        if (__ballot(ok) == ~0ull) break;
        if (give_up(t0, tmo, call)) { dead = true; break; }
      }
      auto row_sum = [&](u64 t) {
        unsigned x = min((unsigned)t - (unsigned)t0 + 4096u, 1u << 20);
        x += __builtin_amdgcn_update_dpp(0u, x, 0x111, 0xf, 0xf, false);
        x += __builtin_amdgcn_update_dpp(0u, x, 0x112, 0xf, 0xf, false);
        x += __builtin_amdgcn_update_dpp(0u, x, 0x114, 0xf, 0xf, false);
        x += __builtin_amdgcn_update_dpp(0u, x, 0x118, 0xf, 0xf, false);
        return x;
      };
      const unsigned sa = row_sum(ta), sb = row_sum(tb);
      const unsigned fa = __shfl(sa, (lane & 3) * 16 + 15), fb = __shfl(sb, (lane & 3) * 16 + 15);
      const int lp = pos_of(lane & 7), lph = (lp + pos_of(rank)) & 3;
      const bool lcross = (lp >> 2) != grp;
      unsigned F = ((lane & 4) ? fb : fa) + (lcross ? (lph == 0 ? 15u * UNIT : 0u) : SAME_BIAS);
      unsigned gone = lcross ? 0u : 5u * __popc(quad & ~(1u << rank) & ((2u << (lane & 7)) - 1u));
      unsigned took = 0;
      const bool live = lane < 8 && lane != rank;
      for (int step = 0; !dead && step < 48; ++step) {
        unsigned k = live ? (F << 3) | lane : 0u;
        k = max(k, (unsigned)__builtin_amdgcn_update_dpp(0u, k, 0x111, 0xf, 0xf, false));
        k = max(k, (unsigned)__builtin_amdgcn_update_dpp(0u, k, 0x112, 0xf, 0xf, false));
        k = max(k, (unsigned)__builtin_amdgcn_update_dpp(0u, k, 0x114, 0xf, 0xf, false));
        const unsigned late = __builtin_amdgcn_readlane(k, 7);
        const int a = late & 7;
        const unsigned room = min(16u - (((unsigned)__builtin_amdgcn_readlane(gone, a) >> (8 * lph)) & 255u), 32u - took);
        k = (live && lcross && lph != 0 && lane != a && room) ? (F << 3) | lane : ~0u;
        k = min(k, (unsigned)__builtin_amdgcn_update_dpp(~0u, k, 0x111, 0xf, 0xf, false));
        k = min(k, (unsigned)__builtin_amdgcn_update_dpp(~0u, k, 0x112, 0xf, 0xf, false));
        k = min(k, (unsigned)__builtin_amdgcn_update_dpp(~0u, k, 0x114, 0xf, 0xf, false));
        const unsigned early = __builtin_amdgcn_readlane(k, 7);
        if (early == ~0u || (late >> 3) < (early >> 3) + RELAY_GAP) break;
        const int b = early & 7;
        const unsigned n = min(max((late - early) >> 14, 1u), (unsigned)__builtin_amdgcn_readlane(room, b));
        if (lane == a) { F -= n * UNIT; gone += n << (8 * __builtin_amdgcn_readlane(lph, b)); }
        if (lane == b) { F += n * UNIT; took += n; }
      }
      auto word_for = [&](int link) {
        const unsigned c = lane & 15, gl = __shfl(gone, link);
        const int ph = __shfl(lcross ? lph : 0, link);
        unsigned at = gl & 255u, word = 0;
        for (int p = 1; p < 4; ++p) { const unsigned n = (gl >> (8 * p)) & 255u; word |= (unsigned)(((c - at) & 15u) < n) << p; at += n; }
        unsigned base = 0;
        for (int a = 0; a < 8; ++a) {
          const unsigned ga = __builtin_amdgcn_readlane(gone, a), n = ph ? (ga >> (8 * ph)) & 255u : 0u;
          const unsigned first = (ga & 255u) + (ph > 1 ? (ga >> 8) & 255u : 0u) + (ph > 2 ? (ga >> 16) & 255u : 0u);
          for (int sl = 0; sl < 2; ++sl) {
            const unsigned i = 15u - c + 16u * sl - base;
            if (i < n) word |= (0x80u | (a << 4) | ((first + i) & 15u)) << (8 + 8 * sl);
          }
          base += n;
        }
        return word;
      };
      const unsigned pa = word_for(lane >> 4), pb = word_for(4 + (lane >> 4));
      asm volatile(".Lcode_mark_b:");
      gu64* const pl = (gu64*)(me + OFF_SFLAG) + lane * 16 + 12;
      __hip_atomic_store(pl, ((u64)call << 32) | pa, __ATOMIC_RELAXED, __HIP_MEMORY_SCOPE_AGENT);
      __hip_atomic_store(pl + 1024, ((u64)call << 32) | pb, __ATOMIC_RELAXED, __HIP_MEMORY_SCOPE_AGENT);
      return;
    }
    unsigned call = 0;
    unsigned char* const pw = tws;
    const auto rsrc_peer = mk_rsrc(pw, WS_USED);
    const auto rsrc_me = mk_rsrc(me, WS_USED);
    const auto rsrc_out = mk_rsrc(P.out, OUT_BYTES);
    const gu32* const ofl = (const gu32*)(me + OFF_OFLAG) + peer * 512;
    const auto rsrc_ofl = mk_rsrc((const void*)ofl, 2048);
    gu32* const prog = (gu32*)(me + OFF_PROG) + (peer * 16 + w) * 32;
    const unsigned vlin = lane * 16;
    unsigned char* const ubuf = lds + wave * (2 * OSLOT);
    unsigned char* const fbuf = lds + LDS_FLAGS + wave * 1024;
    const unsigned vout = h * 4 * (GN * 4) + r * 16;
    const unsigned zaddr = lds_addr(lds + LDS_FLAGS + 4096 + wave * 256);
    *(LAS unsigned*)(size_t)(zaddr + lane * 4) = 0u;

    unsigned ops = 64;
    u64 hist = 0;
    unsigned pend_op = 0, pend_round = 0;
    gu32* last_word = prog;
    unsigned fl = 0;
    auto settle = [&](unsigned done_below) {
      if (pend_round && pend_op < done_below) {
        if (lane < 2) __hip_atomic_store(lane ? last_word : prog, call * 16u + 15u, __ATOMIC_RELAXED, __HIP_MEMORY_SCOPE_SYSTEM);
        asm volatile("" ::: "memory");
        ++ops; pend_round = 0;
      }
    };
    auto throttle = [&](int win) {
      const unsigned n = (ops - 1u - (unsigned)(hist >> (8 * win - 8))) & 255u;
      wait_vm_dyn(n);
      settle(ops - n);
    };
    auto piece = [&](const __amdgpu_buffer_rsrc_t rs, unsigned char* dst, unsigned voff, unsigned soff) {
      __builtin_amdgcn_raw_ptr_buffer_load_lds(rs, (LAS void*)dst, 16, voff, soff, 0, AUX_SYS);
      asm volatile("" ::: "memory");
      hist = (hist << 8) | (ops & 255u); ++ops;
    };
    auto reached = [&](unsigned need, unsigned thr) { return ((unsigned)__ballot(fl >= thr) & need) == need; };
    auto await_flag = [&](const gu32* f, unsigned need, unsigned thr) {
      const u64 t0 = __builtin_amdgcn_s_memrealtime();
      while (!dead) {
        const unsigned seen = __hip_atomic_load(tmo, __ATOMIC_RELAXED, __HIP_MEMORY_SCOPE_AGENT);
        fl = __hip_atomic_load(f, __ATOMIC_RELAXED, __HIP_MEMORY_SCOPE_SYSTEM);
        if (reached(need, thr)) break;
        settle(ops);
        if (give_up(t0, tmo, call, seen)) dead = true;
      }
      __builtin_amdgcn_s_waitcnt(0x0F70);
      __builtin_amdgcn_fence(__ATOMIC_ACQUIRE, "wavefront");
      settle(ops);
    };
    unsigned fbase = 0;
    auto refresh_out = [&](unsigned from) {
      fbase = from & ~1u;
      __builtin_amdgcn_raw_ptr_buffer_load_lds(rsrc_ofl, (LAS void*)fbuf, 16, vlin + fbase * 8u, 0, 0, AUX_SYS);
      asm volatile("" ::: "memory");
      ++ops;
    };

    unsigned char* const ring = ubuf + OSLOT;
    unsigned dprev = 0;
    auto put = [&](int c) {
      const v4u t = *(const v4u*)(ring + (c & 15) * 1024 + vlin);
      STORE16(t, rsrc_me, vlin, dprev + (c & 15) * 1024u, AUX_SYS);
      if ((c & 15) == 15) { pend_op = ops; pend_round = 1; }
      ++ops;
    };
    {
      v4u hold = {0u, 0u, 0u, 0u};
      const bool cross = xcc >= 4;
      const int lph = (pos_of(peer) + pos_of(rank)) & 3;
      const int help_round = cross ? round_of_code(9 + 2 * lph) : -1;
      const int ex = cross ? w : (w - 5 * slot) & 15;
      unsigned plan;
      {
        unsigned same = quad & ~(1u << rank);
        for (int i = 0; i < w / 5; ++i) same &= same - 1;
        plan = cross ? ((lph == 0 && w < 15) ? (0x80u | (__builtin_ctz(same | 0x80u) << 4) | w) << 8 : 0u) : (unsigned)(w / 5 == slot);
      }
      unsigned t_src, t_dst, t_odd, t_flag, t_need, t_pub;
      u64 todo;
      auto tabulate = [&]() {
        const int t = lane >> 2, sub = lane & 3, code = code_of_round(t < 12 ? t : 0), kh = (code & 1) ^ grp, n = (code >> 1) & 3;
        const unsigned line = OFF_PROG + (peer * 16 + w) * 128;
        t_flag = 0; t_pub = line;
        bool run;
        t_odd = 0;
        if (lane >= 48) {
          const int item = (2 * (grp ^ 1) + ((ex >> 2) & 1)) * 2 + ((ex >> 1) & 1), col = rank + 8 * (ex & 1);
          t_src = t_dst = x_item_off(peer, item, 0u) + col * (16 * KiB);
          t_odd = x_item_off(peer, item, 1u) - x_item_off(peer, item, 0u);
          t_need = 1u << item;
          t_pub = OFF_PROG + (peer * 16 + ex) * 128 + 20;
          run = (lane == 48 && ex < 8) || lane == 49;
        } else if (sub) {
          const unsigned job = plan >> (8 * sub), from = (job >> 4) & 7u, col = job & 15u;
          t_src = t_dst = OFF_WG + from * (2 * MiB) + (lph * 2 + (grp ^ 1)) * (256 * KiB) + col * (16 * KiB);
          t_flag = OFF_PROG + (from * 16 + col) * 128;
          t_need = 1u;
          t_pub = t_flag + 4 + 4 * lph;
          run = sub < 3 && t == help_round && (job & 0x80u);
        } else if (code < 8) {
          const int item = (2 * grp + n) * 2 + kh;
          t_src = t_dst = x_item_off(peer, item, 0u) + w * (16 * KiB);
          t_odd = x_item_off(peer, item, 1u) - x_item_off(peer, item, 0u);
          t_need = 1u << item;
          run = true;
        } else {
          t_src = OFF_WS + (block_of(rank, n) * 2 + kh) * (256 * KiB) + w * (16 * KiB);
          t_dst = OFF_WG + peer * (2 * MiB) + (n * 2 + kh) * (256 * KiB) + w * (16 * KiB);
          t_need = 0x100u << (n * 2 + kh);
          if (code & 1) t_pub = line + 4 + 4 * n;
          run = !((code & 1) && ((plan >> n) & 1u));
        }
        if (lane == 8) t_need |= 0x10000u;
        if (lane == PLAN_POS) t_need |= 1u << 25;
        todo = __ballot(run);
      };
      tabulate();
      asm volatile("" : "+v"(t_src), "+v"(t_dst), "+v"(t_odd), "+v"(t_flag), "+v"(t_need), "+v"(t_pub), "+s"(todo));
      asm volatile("" : "+v"(fl0));
      fl = fl0;
      call = CALL_NOW();
      if (lane == 0) __hip_atomic_store(cnt, call, __ATOMIC_RELAXED, __HIP_MEMORY_SCOPE_AGENT);
      const unsigned thr_help = call * 16u + 1u + round_of_code(8 + 2 * lph);
      int pos = 0;
      gu32* ppub = prog; unsigned pval = 0;
      while (pos < 49) {
        int npos = 49;
        const unsigned codd = (call & 1u) ? __builtin_amdgcn_readlane(t_odd, pos) : 0u;
        const unsigned csrc = __builtin_amdgcn_readlane(t_src, pos) + codd, cdst = __builtin_amdgcn_readlane(t_dst, pos) + codd;
        const auto rs_early = mk_rsrc(me, pos ? WS_USED : 0u);
        static_for<4>([&](auto qc) {
          constexpr int jq = decltype(qc)::v;
          if (jq < PUSHQ && pos == 0) return;
          static_for<4>([&](auto uc) {
            constexpr int j = jq * 4 + decltype(uc)::v;
            constexpr int REFRESH_AT = 15 - IN_WIN - 1, PUBLISH_AT = 2 * IN_WIN + 1;
            static_assert(2 * IN_WIN + 1 <= 15 && REFRESH_AT >= 4 * PUSHQ, "window");
            wait_vm<2 * IN_WIN - 1 + ((j > REFRESH_AT && j - IN_WIN <= REFRESH_AT) ? 1 : 0)>();
            const v4u got = *(const v4u*)(ring + ((j - IN_WIN) & 15) * 1024 + vlin);
            if ((j & 3) == 0) {
              if (j == 0 || pos == 0) {
                const unsigned need = pos ? __builtin_amdgcn_readlane(t_need, pos) : 0x10000u << jq;
                const unsigned thr = (pos & 3) ? thr_help : call;
                if (pos) fl = *(const unsigned*)(fbuf + (lane & 31) * 4);
                if (!reached(need, thr)) await_flag((const gu32*)((pos & 3) ? pw + (unsigned)__builtin_amdgcn_readlane(t_flag, pos) : me + foff), need, thr);
                if (pos == 4 * TS_ROUND && lane == 0)
                  __hip_atomic_store((gu64*)(me + OFF_TS) + peer * 16 + w, ((u64)call << 32) | (unsigned)__builtin_amdgcn_s_memrealtime(), __ATOMIC_RELAXED, __HIP_MEMORY_SCOPE_AGENT);
                if (pos == PLAN_POS) plan = dead ? 0u : __builtin_amdgcn_readlane(fl, 24);
              }
            }
            if (j == PUBLISH_AT && pos > 4 && lane == 0) __hip_atomic_store(ppub, pval, __ATOMIC_RELAXED, __HIP_MEMORY_SCOPE_SYSTEM);
            __builtin_amdgcn_raw_ptr_buffer_load_lds(rsrc_peer, (LAS void*)(ring + jq * 4096), 16, vlin, csrc + jq * 4096, (j & 3) * 1024, AUX_SYS);
            asm volatile("" ::: "memory");
            if (j <= IN_WIN) STORE16(hold, rs_early, vlin, dprev + (j - IN_WIN - 1 + 16) * 1024, AUX_SYS);
            else if (j - IN_WIN - 1 < 4 * PUSHQ) STORE16(hold, rs_early, vlin, cdst + (j - IN_WIN - 1) * 1024, AUX_SYS);
            else STORE16(hold, rsrc_me, vlin, cdst + (j - IN_WIN - 1) * 1024, AUX_SYS);
            hold = got;
            if (j == 1 && pos == PLAN_POS) tabulate();
            if (j == REFRESH_AT) {
              npos = pos + 1 + __builtin_ctzll(todo >> (pos + 1));
              if (npos >= 49) refresh_out(0);
              else __builtin_amdgcn_raw_ptr_buffer_load_lds((npos & 3) ? rsrc_peer : rsrc_me, (LAS void*)fbuf, 4, (npos & 3) ? 0u : foff, __builtin_amdgcn_readlane(t_flag, npos), 0, AUX_SYS);
            }
          });
        });
        dprev = cdst; ppub = (gu32*)(me + (unsigned)__builtin_amdgcn_readlane(t_pub, pos)); pval = call * 16u + (pos >> 2) + 1u;
        pos = npos;
      }
      last_word = ppub;
      ops = 128;
#pragma unroll
      for (int i = 7; i >= 0; --i) hist = (hist << 8) | (ops - 2u - 2u * i);
      STORE16(hold, rsrc_me, vlin, dprev + (15 - IN_WIN) * 1024, AUX_SYS);
      ++ops;
    }

    const gu32* const tick = (const gu32*)(me + OFF_TICK + peer * 128);
    const unsigned nunits = (xcc >= 4) ? 192u : 256u;
    const int skip_ph = (xcc >= 4) ? ((pos_of(peer) + pos_of(rank)) & 3) : 4;
    int c = 192 - IN_WIN;
    unsigned last_op = 0;
    unsigned nx = w, par = 0;
    bool plain_cur = false, plain_nx = false;
    for (int cur = -1;;) {
      const int win = (cur < 0) ? IN_WIN : OUT_WIN;
      unsigned nb = 0;
      if (nx < nunits) {
        unsigned cn = 0, tag = 0;
        if (nx - fbase < 128u) {
          const v2u f = *(const v2u*)(fbuf + (nx - fbase) * 8u);
          cn = __builtin_amdgcn_readfirstlane(f[0]); tag = __builtin_amdgcn_readfirstlane(f[1]);
        }
        if (tag < call) {
          if (c < 192) { WAIT_VM(0); for (; c < 192; ++c) put(c); }
          await_flag(ofl + (nx & ~15u) * 2u + (lane & 31), 2u << ((nx & 15u) * 2u), call);
          cn = __builtin_amdgcn_readlane(fl, (nx & 15u) * 2u);
        }
        plain_nx = cn >= PLAIN_FROM;
        nb = dead ? 0u : plain_nx ? 16384u : ((512u + cn * 4u + 63u) & ~63u);
      }
      const int n = (int)((nb + 1023u) >> 10);
      const auto rs = mk_rsrc(pw + OFF_OS + rank * OS_OWNER + (nx & 255u) * OSLOT, nb);
      unsigned char* const nbuf = ubuf + par * OSLOT;
      int i = 0;
      unsigned draw_op = 0;
      auto draw = [&]() {
        if (nx < nunits) {
          refresh_out(nx);
          asm volatile("s_mov_b64 exec, 1\n\tglobal_atomic_add v250, %0, %1, %2 sc0\n\ts_mov_b64 exec, -1"
                       :: "v"(0u), "v"(1u), "s"(tick) : "memory", "v250");
          draw_op = ops; ++ops;
        }
      };
      if (cur >= 0) {
        for (; i < n && i < win; ++i) { throttle(win); piece(rs, nbuf + i * 1024, vlin + i * 1024, 0); }
        wait_vm_dyn(ops - 1u - last_op);
        const unsigned char* const cbuf = ubuf + (par ^ 1u) * OSLOT;
        const int oph = (cur >> 6) + ((cur >> 6) >= skip_ph);
        const int hi = (cur >> 5) & 1, mid = (cur >> 4) & 1;
        const int rblk = (oph ? mid : hi) * 4 + (cur & 2) + (oph ? hi : mid);
        const unsigned so = ((pos_of(peer) >> 2) * 8 + rblk) * 32u * (GN * 4) + (block_of(peer, oph) * 1024 + ((cur >> 2) & 3) * 256 + (cur & 1) * 128) * 4u;
        if (__builtin_expect(plain_cur && !dead, 0)) {
          draw();
#pragma unroll 1
          for (int e = 0; e < 16; ++e) {
            const v4u o = *(const v4u*)(cbuf + e * 1024 + vlin);
            STORE16(o, rsrc_out, vout, so + ((e >> 2) * 8 + (e & 3)) * (GN * 4), AUX_DEV);
            ++ops;
            if (i < n) { throttle(win); piece(rs, nbuf + i * 1024, vlin + i * 1024, 0); ++i; }
          }
        } else {
          v2u mk = *(const v2u*)(cbuf + lane * 8);
          if (dead) { mk = v2u{~0u, ~0u}; *(LAS unsigned*)(size_t)(zaddr + lane * 4) = 0x7fc00000u; }
          const unsigned np = 64u - __popc(mk[0]) - __popc(mk[1]);
          unsigned qa = lds_addr(cbuf) + 512u + (wave_scan(np) - np) * 4u;
          static_for<4>([&](auto qc) {
            constexpr int q = decltype(qc)::v;
            v4u o[4];
            static_for<4>([&](auto ec) {
              constexpr int e = decltype(ec)::v, idx = (q * 4 + e) * 4;
              unsigned a[4];
              run_addr4<31 - (idx & 31)>(mk[idx >> 5], zaddr, qa, a);
#pragma unroll
              for (int f = 0; f < 4; ++f) o[e][f] = *(const LAS unsigned*)(size_t)(a[f] + (idx + f) * 4);
            });
            if (q == 1) draw();
#pragma unroll
            for (int e = 0; e < 4; ++e) STORE16(o[e], rsrc_out, vout, so + (q * 8 + e) * (GN * 4), AUX_DEV);
            ops += 4;
            if (i < n) { throttle(win); piece(rs, nbuf + i * 1024, vlin + i * 1024, 0); ++i; }
          });
        }
      } else {
        draw();
      }
      if (nx >= nunits) break;
      for (; i < n; ++i) {
        throttle(win); piece(rs, nbuf + i * 1024, vlin + i * 1024, 0);
        if (c < 192) put(c++);
      }
      last_op = (unsigned)(ops - 1u - ((ops - 1u - (unsigned)hist) & 255u));
      if (cur < 0) {
        if (c < 192) { WAIT_VM(0); for (; c < 192; ++c) put(c); }
        if (pend_round) { wait_vm_dyn(ops - 1u - pend_op); settle(pend_op + 1u); }
        WAIT_LGKM0();
      }
      wait_vm_dyn(ops - 1u - draw_op);
      cur = (int)nx;
      plain_cur = plain_nx;
      asm volatile("v_readfirstlane_b32 %0, v250" : "=s"(nx) :: "memory");
      nx = nx - (call - 1u) * nunits + 16u;
      par ^= 1u;
    }
    asm volatile(".Lcode_mark_a:\n\ts_waitcnt vmcnt(0)" ::: "memory");
    return;
  }

  const int g = slot - 4;
  const auto rsrc_me = mk_rsrc(me, WS_USED);
  {
    auto item_done = [&](int item, unsigned n, int word, bool tell) {
      WAIT_VM(0);
      unsigned old = 0;
      if (lane == 0) old = __hip_atomic_fetch_add((gu32*)(me + OFF_SCNT + item * 128), 1u, __ATOMIC_RELAXED, __HIP_MEMORY_SCOPE_AGENT);
      if ((unsigned)__builtin_amdgcn_readfirstlane(old) + 1u == n * CALL_NOW() && tell)
        __hip_atomic_store((gu32*)(dws + OFF_IFLAG) + rank * 32 + word, CALL_NOW(), __ATOMIC_RELAXED, __HIP_MEMORY_SCOPE_SYSTEM);
    };
    if (jb >= 128 && jb < 640) {
      const int ph = (jb - 128) >> 7, ia = (jb >> 5) & 3, s = (jb >> 3) & 3, j = jb & 7;
      const int blk = ia * 4 + ((ph - ia) & 3);
      if (ph == 0) __builtin_amdgcn_s_sleep(DELAY1); else { __builtin_amdgcn_s_sleep(DELAY2); for (int i = 0; i < ph; ++i) __builtin_amdgcn_s_sleep(DELAY3); }
      const auto rw = mk_rsrc(wp, KS * GN);
      const unsigned voff = h * 16 * GN + r * 4;
      const unsigned so = s * 128 * GN + blk * 1024 + j * 128;
      unsigned d[64];
#pragma unroll
      for (int e = 0; e < 64; ++e)
        d[e] = __builtin_amdgcn_raw_buffer_load_b32(rw, voff, so + ((e >> 4) * 32 + (e & 15)) * GN, 0);
      const unsigned dst = OFF_WS + blk * (512 * KiB) + (s * 8 + j) * (16 * KiB);
#pragma unroll
      for (int ki = 0; ki < 4; ++ki) {
        v4u o[4];
#pragma unroll
        for (int gq = 0; gq < 4; ++gq) {
          unsigned a = d[ki * 16 + gq * 4], b = d[ki * 16 + gq * 4 + 1], c = d[ki * 16 + gq * 4 + 2], e = d[ki * 16 + gq * 4 + 3];
          unsigned t0 = __builtin_amdgcn_perm(b, a, 0x05010400u), t1 = __builtin_amdgcn_perm(b, a, 0x07030602u);
          unsigned t2 = __builtin_amdgcn_perm(e, c, 0x05010400u), t3 = __builtin_amdgcn_perm(e, c, 0x07030602u);
          o[0][gq] = __builtin_amdgcn_perm(t2, t0, 0x05040100u);
          o[1][gq] = __builtin_amdgcn_perm(t2, t0, 0x07060302u);
          o[2][gq] = __builtin_amdgcn_perm(t3, t1, 0x05040100u);
          o[3][gq] = __builtin_amdgcn_perm(t3, t1, 0x07060302u);
        }
#pragma unroll
        for (int f = 0; f < 4; ++f)
          STORE16(o[f], rsrc_me, lane * 16, dst + f * 4096 + ki * 1024, AUX_SYS);
      }
      item_done(8 + blk * 2 + (s >> 1), 16u, 8 + ph * 2 + (s >> 1), lane == rank_at(ia) || lane == rank_at(4 + (blk & 3)));
    } else if (express) {
      const bool push = jb >= 768;
      if (!push || tgt != rank) {
        const auto rd = mk_rsrc(push ? tws : me, WS_USED);
        const unsigned dst = x_item_off(rank, fg * 5, CALL_NOW()) + (jb & 15) * (16 * KiB) + rb0 * 4096;
#pragma unroll
        for (int e = 0; e < 4; ++e) {
          if (push && e >= PUSH_WIN) wait_vm<PUSH_WIN - 1>();
          STORE16(xe[e], rd, lane * 16 + e * 1024, dst, AUX_SYS);
        }
        WAIT_VM(0);
        const bool tell = push ? (lane == tgt) : (lane < NR && rb0 > 0 && (pos_of(lane & 7) >> 2) == fg);
        if (tell)
          __hip_atomic_store((gu32*)(dws + OFF_SFLAG) + (rank * 16 + (jb & 15)) * 32 + 16 + rb0, CALL_NOW(), __ATOMIC_RELAXED, __HIP_MEMORY_SCOPE_SYSTEM);
        if (!push) item_done(fg * 5, 64u, fg * 5, lane < NR && (lane == rank || (pos_of(lane & 7) >> 2) != fg));
      }
    } else if (jb < 736) {
      const int rest = (jb - 640) >> 4;
      const int un = (rest + 1 + (rest >= 4)) * 16 + (jb & 15);
      __builtin_amdgcn_s_sleep(DELAY2);
      const auto rx = mk_rsrc(xp, GM * KS);
      const unsigned so = ((un & 7) * MS + (un >> 5) * 128) * KS + ((un >> 3) & 3) * 128;
      v4u d[16];
#pragma unroll
      for (int e = 0; e < 16; ++e) d[e] = __builtin_amdgcn_raw_buffer_load_b128(rx, r * KS + h * 16 + (e & 3) * 32, so + (e >> 2) * 32 * KS, 0);
#pragma unroll
      for (int e = 0; e < 16; ++e) STORE16(d[e], rsrc_me, lane * 16 + (e & 3) * 1024, OFF_XG + rank * (2 * MiB) + un * (16 * KiB) + (e >> 2) * 4096, AUX_SYS);
      item_done(un >> 4, 16u, un >> 4, lane < NR);
    }
  }
  const unsigned call = CALL_NOW();
  if (lane == 0) __hip_atomic_store(cnt, call, __ATOMIC_RELAXED, __HIP_MEMORY_SCOPE_AGENT);

  const float scale = P.sx[0] * P.sw[0];
  const int ntiles = (g < 4) ? 2 : 1;
  const int wm = wave >> 1, wn = wave & 1;
  for (int it = 0; it < ntiles; ++it) {
    const bool mine = it || g >= 24;
    const int ph = (xcd & 1) + ((it || (g >= 12 && g < 24)) ? 2 : 0);
    const int jj = (xcd & 2) + (g & 1);
    const bool extra = mine && (g & 2);
    const int ch = 2 * (grp ^ (int)extra) + (xcd >> 2);
    int own = rank;
    if (!mine) {
      unsigned others = 0xffu & ~(1u << rank) & ~(1u << partner_of(rank, ph));
      for (int i = 0; i < ((g >= 12 ? g - 12 : g) >> 1); ++i) others &= others - 1;
      own = __builtin_ctz(others);
    }
    const int blk = block_of(rank, ph);
    const int sl = lane & 7;
    const int wsel = (sl < 4) ? (sl >> 1) * 8 + 2 * jj + (sl & 1) : extra ? (xcd >> 2) * 4 + (sl - 4) : own + 8 * (sl & 1);
    const gu32* paddr = (const gu32*)(me + OFF_PROG) + (((rank + 1 + (lane >> 3)) & 7) * 16 + wsel) * 32;
    const unsigned thr1 = call * 16u + ((sl >= 4 && extra) ? 0u : 1u + round_of_code((sl < 4) ? 8 + 2 * ph : 2 * (xcd >> 2) + (((sl >> 1) & 1) ^ grp)));
    v2u* const order = (v2u*)(lds + LDS_ORDER);
    u64 taken = 0;
    int nq = 4;
    auto refill = [&](int want) {
      const u64 t0 = __builtin_amdgcn_s_memrealtime();
      for (;;) {
        const unsigned p1 = dead ? ~0u : __hip_atomic_load(paddr + ((sl >= 4 && extra) ? 5 : 0), __ATOMIC_RELAXED, __HIP_MEMORY_SCOPE_AGENT);
        const unsigned p2 = dead ? ~0u : __hip_atomic_load(paddr + 1 + ph, __ATOMIC_RELAXED, __HIP_MEMORY_SCOPE_AGENT);
        const u64 m1 = __ballot(p1 >= thr1), m2 = __ballot(p2 >= call * 16u);
        for (int ks = 0; ks < 4; ++ks) {
          const u64 mw = (((ks >> 1) == grp) ? m1 : m2) >> (2 * (ks & 1));
          const u64 rdy = mw & (mw >> 1) & (m1 >> (4 + ks)) & 0x0001010101010101ull & ~(taken >> ks);
          if ((rdy >> lane) & 1) {
            const unsigned p = (rank + 1 + (lane >> 3)) & 7;
            order[nq + __builtin_amdgcn_mbcnt_hi((unsigned)(rdy >> 32), __builtin_amdgcn_mbcnt_lo((unsigned)rdy, 0u))] =
                v2u{x_item_off(p, ch * 2 + (ks >> 1), call) + (ks & 1u) * (128 * KiB), OFF_WG + p * (2 * MiB) + (ph * 4 + ks) * (128 * KiB)};
          }
          taken |= rdy << ks;
          nq += __popcll(rdy);
        }
        if (nq > want) break;
        if (give_up(t0, tmo, call)) dead = true;
        __builtin_amdgcn_s_sleep(16);
      }
      __builtin_amdgcn_s_waitcnt(0x0F70);
      __builtin_amdgcn_fence(__ATOMIC_ACQUIRE, "wavefront");
    };

    const unsigned sa0 = own * (16 * KiB) + wave * 4096;
    const unsigned sb0 = jj * (32 * KiB) + wave * 8192;
    v16f acc[2][4];
#pragma unroll
    for (int mi = 0; mi < 2; ++mi)
#pragma unroll
      for (int f = 0; f < 4; ++f)
#pragma unroll
        for (int e = 0; e < 16; ++e) acc[mi][f][e] = 0.f;

    if (wave == 0) {
      if (lane < 8) order[32 + lane] = v2u{0u, 0u};
      unsigned dummy;
      wait_words((const gu32*)(me + OFF_IFLAG) + rank * 32 + (lane & 15), call, (3ull << (ch * 2)) | (0x300ull << (ph * 2)), tmo, call, dead, dummy);
    }
    BARRIER();
#define MF(A, B, mi, f) acc[mi][f] = __builtin_amdgcn_mfma_scale_f32_32x32x64_f8f6f4(A[mi], B[f], acc[mi][f], 0, 0, 0, 0, 0, 0)
#define DM(l, so, o) __builtin_amdgcn_raw_ptr_buffer_load_lds(rs, (LAS void*)(l), 16, lane * 16, (so), (o), AUX_DEV)
#define FR(dst, ptr) dst = __builtin_shufflevector(*(const v4i*)(ptr), *(const v4i*)((ptr) + 1024), 0, 1, 2, 3, 4, 5, 6, 7)
#define SB() __builtin_amdgcn_sched_barrier(0)
#pragma unroll
    for (int s = 0; s < 3; ++s) {
      const auto rs = rsrc_me;
      const unsigned sa = sa0 + x_item_off(rank, ch * 2 + (s >> 1), call) + (s & 1) * (128 * KiB), sb = sb0 + OFF_WS + (blk * 4 + s) * (128 * KiB);
      unsigned char* const la = lds + s * STAGE_BYTES + wave * 4096;
      unsigned char* const lb = lds + s * STAGE_BYTES + STAGE_A + wave * 8192;
      DM(la, sa, 0); DM(la, sa, 1024); DM(lb, sb, 0); DM(lb, sb, 1024); DM(lb + 4096, sb + 4096, 0); DM(lb + 4096, sb + 4096, 1024); SB();
      DM(la, sa, 2048); DM(la, sa, 3072); DM(lb, sb, 2048); DM(lb, sb, 3072); DM(lb + 4096, sb + 4096, 2048); DM(lb + 4096, sb + 4096, 3072); SB();
    }
    WAIT_VM(30);
    BARRIER();
    v8i A0[2], B0[4], A1[2], B1[4];
    const unsigned char* const ra0 = lds + lane * 16 + wm * 8192;
    const unsigned char* const rb0 = lds + lane * 16 + STAGE_A + wn * 16384;
    FR(A0[0], ra0); FR(A0[1], ra0 + 4096); FR(B0[0], rb0); FR(B0[1], rb0 + 4096); FR(B0[2], rb0 + 8192); FR(B0[3], rb0 + 12288);
    int buf = 0;
    v2u ent = {x_item_off(rank, ch * 2 + 1, call) + 128 * KiB, OFF_WS + (blk * 4 + 3) * (128 * KiB)};
    for (int k = 0; k < 32; ++k) {
      const auto rs = mk_rsrc(me, k < 29 ? WS_USED : 0u);
      const unsigned sa = sa0 + __builtin_amdgcn_readfirstlane(ent[0]);
      const unsigned sb = sb0 + __builtin_amdgcn_readfirstlane(ent[1]);
      unsigned char* const la = lds + buf + wave * 4096;
      unsigned char* const lb = lds + buf + STAGE_A + wave * 8192;
      const unsigned char* ra = ra0 + buf + 2048;
      const unsigned char* rb = rb0 + buf + 2048;
      WAIT_LGKM0();
      WAIT_VM(24);
      BARRIER();
      SB();
      MF(A0, B0, 0, 0); SB(); FR(A1[0], ra); DM(la, sa, 0); SB();
      MF(A0, B0, 0, 1); SB(); FR(B1[0], rb); DM(la, sa, 1024); SB();
      MF(A0, B0, 0, 2); SB(); FR(B1[1], rb + 4096); DM(lb, sb, 0); SB();
      MF(A0, B0, 0, 3); SB(); FR(B1[2], rb + 8192); DM(lb, sb, 1024); SB();
      MF(A0, B0, 1, 0); SB(); FR(B1[3], rb + 12288); DM(lb + 4096, sb + 4096, 0); SB();
      MF(A0, B0, 1, 1); SB(); FR(A1[1], ra + 4096); DM(lb + 4096, sb + 4096, 1024); SB();
      MF(A0, B0, 1, 2); SB();
      buf = (buf == 2 * STAGE_BYTES) ? 0 : buf + STAGE_BYTES;
      ra = ra0 + buf; rb = rb0 + buf;
      MF(A0, B0, 1, 3); SB();
      if (wave == 0 && k < 28 && nq <= k + 4) refill(k + 4);
      WAIT_LGKM0();
      WAIT_VM(24);
      BARRIER();
      SB();
      MF(A1, B1, 0, 0); SB(); ent = order[k + 4]; FR(A0[0], ra); DM(la, sa, 2048); SB();
      MF(A1, B1, 0, 1); SB(); FR(B0[0], rb); DM(la, sa, 3072); SB();
      MF(A1, B1, 0, 2); SB(); FR(B0[1], rb + 4096); DM(lb, sb, 2048); SB();
      MF(A1, B1, 0, 3); SB(); FR(B0[2], rb + 8192); DM(lb, sb, 3072); SB();
      MF(A1, B1, 1, 0); SB(); FR(B0[3], rb + 12288); DM(lb + 4096, sb + 4096, 2048); SB();
      MF(A1, B1, 1, 1); SB(); FR(A0[1], ra + 4096); DM(lb + 4096, sb + 4096, 3072); SB();
      MF(A1, B1, 1, 2); SB();
      MF(A1, B1, 1, 3); SB();
    }
    const unsigned tseen = __hip_atomic_load(tmo, __ATOMIC_RELAXED, __HIP_MEMORY_SCOPE_AGENT);
    WAIT_VM(0);
    WAIT_LGKM0();
    BARRIER();
    const bool poison = __builtin_amdgcn_readfirstlane(tseen) == call;
    const float sc = poison ? __uint_as_float(0x7fc00000u) : scale;
    if (mine) {
      const auto rdst = mk_rsrc(P.out, OUT_BYTES);
      const unsigned vo = h * 4 * (GN * 4) + r * 16;
      const unsigned sbase = (ch * 128 + wm * 64) * (GN * 4) + (blk * 1024 + jj * 256 + wn * 128) * 4;
#pragma unroll
      for (int mi = 0; mi < 2; ++mi)
#pragma unroll
        for (int e = 0; e < 16; ++e) {
          v4u o;
#pragma unroll
          for (int f = 0; f < 4; ++f) { const float y = acc[mi][f][e] * sc; const unsigned b = __float_as_uint(y); o[f] = b & ~(unsigned)((int)b >> 31); }
          STORE16(o, rdst, vo, sbase + (mi * 32 + (e >> 2) * 8 + (e & 3)) * (GN * 4), AUX_DEV);
        }
    } else {
      unsigned* const cnts = (unsigned*)(lds + LDS_ORDER);
      const bool ocross = (pos_of(own) >> 2) != grp;
      const int unit0 = (ph - (ocross && ph > ((pos_of(own) + pos_of(rank)) & 3))) * 64 + (ch & 1) * (ph ? 16 : 32) + jj * 4;
      const int mstep = ph ? 32 : 16;
#pragma unroll
      for (int mi = 0; mi < 2; ++mi) {
        unsigned char* const cb = lds + (wave * 2 + mi) * OSLOT;
        float y[64];
        v2u mk = {0u, 0u};
#pragma unroll
        for (int i = 0; i < 64; ++i) {
          y[i] = acc[mi][i & 3][i >> 2] * sc;
          mk[i >> 5] = __builtin_amdgcn_alignbit(mk[i >> 5], __float_as_uint(y[i]), 31);
        }
        const unsigned np = 64u - __popc(mk[0]) - __popc(mk[1]);
        const unsigned inc = wave_scan(np);
        const unsigned base = __builtin_amdgcn_readlane(inc, 63);
        gu64* const oflag = (gu64*)(P.ws[own] + OFF_OFLAG) + rank * 256 + unit0;
        if (lane == 0) cnts[mi * 4 + wave] = base;
        if (mi == 1) {
          WAIT_VM(0);
          WAIT_LGKM0();
          BARRIER();
          if (wave == 0 && lane < 4) __hip_atomic_store(oflag + lane, ((u64)call << 32) | cnts[lane], __ATOMIC_RELAXED, __HIP_MEMORY_SCOPE_SYSTEM);
        }
        if (__builtin_expect(base >= PLAIN_FROM, 0)) {
#pragma unroll
          for (int e = 0; e < 16; ++e) {
            v4u o;
#pragma unroll
            for (int f = 0; f < 4; ++f) { const unsigned b = __float_as_uint(y[e * 4 + f]); o[f] = b & ~(unsigned)((int)b >> 31); }
            *(v4u*)(cb + e * 1024 + lane * 16) = o;
          }
        } else {
          *(v2u*)(cb + lane * 8) = mk;
          unsigned qa = lds_addr(cb) + 512u + (inc - np) * 4u;
          const unsigned junk = lds_addr(lds + LDS_FLAGS + wave * 512) + lane * 4;
          static_for<16>([&](auto ec) {
            constexpr int idx = decltype(ec)::v * 4;
            unsigned a[4];
            run_addr4<31 - (idx & 31)>(mk[idx >> 5], junk, qa, a);
#pragma unroll
            for (int f = 0; f < 4; ++f) *(LAS float*)(size_t)(a[f] + (idx + f) * 4) = y[idx + f];
          });
        }
        asm volatile("" ::: "memory");
        const unsigned dst = OFF_OS + own * OS_OWNER + (unit0 + wave + mi * mstep) * OSLOT;
        const int n = base >= PLAIN_FROM ? 16 : (int)((512u + base * 4u + 1023u) >> 10);
        for (int i = 0; i < n; i += 3) {
          const auto rslot = mk_rsrc(me + dst, OSLOT);
          v4u t[3];
#pragma unroll
          for (int u = 0; u < 3; ++u) t[u] = *(const v4u*)(cb + (i + u) * 1024 + lane * 16);
#pragma unroll
          for (int u = 0; u < 3; ++u) STORE16(t[u], rslot, lane * 16 + (i + u) * 1024, 0, AUX_SYS);
        }
        if (mi == 1) {
          bool stuck = false;
          if (wave == 0 && (ph == 3 || (ph == 2 && own == partner_of(rank, 3)))) {
            unsigned dummy;
            wait_words((const gu32*)(me + OFF_PROG) + (own * 16 + (lane & 15)) * 32, call * 16u + 15u, ~0ull, tmo, call, stuck, dummy);
          }
          WAIT_VM(0);
          BARRIER();
          if (wave == 0 && lane < 4 && (!stuck || poison))
            __hip_atomic_store(oflag + mstep + lane, ((u64)call << 32) | cnts[4 + lane], __ATOMIC_RELAXED, __HIP_MEMORY_SCOPE_SYSTEM);
        }
      }
      asm volatile(".Lcode_mark_c:\n\ts_waitcnt lgkmcnt(0)" ::: "memory");
    }
  }
}

}

extern "C" void kernel_launch(int rank, int nranks, ncclComm_t comm,
                              hipStream_t stream,
                              void* const* d_in, const int* in_sizes, int n_in,
                              void* d_out, int out_size,
                              const int* in_dtypes, int out_dtype,
                              void* const* d_out_all,
                              void* const* d_ws_all, size_t ws_bytes) {
  if (nranks != NR || (unsigned)rank >= (unsigned)NR || n_in < 4 || ws_bytes < WS_USED || out_size != MS * GN ||
      in_sizes[0] != GM * KS || in_sizes[1] != KS * GN) {
    fprintf(stderr, "unexpected problem shape\n");
    return;
  }
  Params P;
  P.x = (const unsigned char*)d_in[0];
  P.w = (const unsigned char*)d_in[1];
  P.sx = (const float*)d_in[2];
  P.sw = (const float*)d_in[3];
  P.out = (float*)d_out;
  for (int p = 0; p < NR; ++p) P.ws[p] = (unsigned char*)d_ws_all[p];
  P.rank = rank; P.pad = 0;
  P.me = P.ws[rank];
  hipLaunchKernelGGL(fused_kernel, dim3(256), dim3(256), 0, stream, P);
  HIP_CHECK(hipGetLastError());
}
```
